```python
import numpy as np

kernel_name = "gpu_gemm_rs_m4096_k4096_n16384_e4m3e5m2_dequant"


ABSMAX_RTOL = 0.05

COMPUTE_DTYPE = "float32"
EPILOGUE = "dequant"
IN_QUANT = None
OUT_QUANT = None
IN_FP8 = "e4m3e5m2"
IN_DTYPES = {"x": "e4m3", "w_mat": "e5m2",
             "scale_x": "float32", "scale_w": "float32"}
WORLD_SIZE = 8

def _snap_fp8(x, mant_bits, max_abs, min_normal):
    a = np.abs(x.astype(np.float64))
    _m, _e = np.frexp(a)
    q = np.where(a >= min_normal, np.ldexp(1.0, _e - 1 - mant_bits),
                 min_normal * 2.0 ** -mant_bits)
    snapped = np.minimum(np.round(a / q) * q, max_abs)
    return (np.sign(x) * snapped).astype(np.float32)

def setup_inputs():

    x = _snap_fp8(np.random.randn(4096, 4096).astype(np.float32), 3, 448.0, 2.0 ** -6)
    w_mat = _snap_fp8(np.random.randn(4096, 16384).astype(np.float32), 2, 57344.0, 2.0 ** -14)
    scale_x = (0.02 * (1.0 + np.random.rand(1))).astype(np.float32)
    scale_w = (0.02 * (1.0 + np.random.rand(1))).astype(np.float32)
    return {"x": x, "w_mat": w_mat, "scale_x": scale_x, "scale_w": scale_w}

def reference(x, w_mat, scale_x, scale_w):
    acc = (x @ w_mat).astype(np.float32)
    y = (acc * (scale_x * scale_w)).astype(np.float32)
    return y
```

```cpp
#include <hip/hip_runtime.h>
#include <rccl/rccl.h>
#include <cstdio>
#include <cstdint>

#define HIP_CHECK(expr) do { hipError_t _e = (expr); if (_e != hipSuccess) { \
    fprintf(stderr, "HIP error %s at %s:%d\n", hipGetErrorString(_e), __FILE__, __LINE__); \
    return; } } while (0)
#define NCCL_CHECK(expr) do { ncclResult_t _r = (expr); if (_r != ncclSuccess) { \
    fprintf(stderr, "RCCL error %s at %s:%d\n", ncclGetErrorString(_r), __FILE__, __LINE__); \
    return; } } while (0)

typedef unsigned u32;
typedef unsigned long long u64;
typedef int      v8i  __attribute__((ext_vector_type(8)));
typedef unsigned v4u  __attribute__((ext_vector_type(4)));
typedef float    v4f  __attribute__((ext_vector_type(4)));
typedef float    v16f __attribute__((ext_vector_type(16)));

constexpr int NR    = 8;
constexpr int MM    = 4096;
constexpr int KK    = 4096;
constexpr int NN    = 16384;
constexpr int KS    = KK / NR;
constexpr int NC    = NN / NR;
constexpr int MR    = MM / NR;
constexpr int TM    = 128;
constexpr int TN    = 128;
constexpr int NSL   = MR / TM;
constexpr int NT    = NC / TN;
constexpr int NXI   = NSL * NR;
constexpr int NTILE = NXI * NT;
constexpr u32 ITEM  = 65536;
constexpr u32 UNIT  = 16384;

constexpr int GRID  = 256;
constexpr int TPB   = 256;
#define PWG 5
constexpr int NPW   = 7 * PWG;
constexpr int NGW   = GRID - NPW;
constexpr int NPREP = NXI + NR * NT;

constexpr u32 OFF_TICKET = 0;
constexpr u32 OFF_TMO    = 2048;
constexpr u32 OFF_UCTR   = 3072;
constexpr u32 OFF_INFLAG = 4096;
constexpr u32 OFF_CNTX   = 8192;
constexpr u32 OFF_CNTW   = 16384;
constexpr u32 OFF_CNTP   = 24576;
constexpr u32 OFF_OFLAG  = 65536;
constexpr u32 OFF_XT     = 1u << 20;
constexpr u32 OFF_WT     = OFF_XT + 2u * NXI * ITEM;
constexpr u32 OFF_XL     = OFF_WT + 2u * NR * NT * ITEM;
constexpr u32 OFF_WL     = OFF_XL + (u32)NR * NXI * ITEM;
constexpr u32 OFF_OUT    = OFF_WL + (u32)NR * NT * ITEM;
#define NPUSH 3
#define TAILT 8
constexpr u32 OFF_XP     = OFF_OUT + 2u * NR * (NSL * NT) * ITEM;
constexpr u32 WS_USED    = OFF_XP + 2u * NR * PWG * ITEM;
static_assert(NPUSH <= PWG && NPUSH <= NR, "pushed x items: row blocks of slice 0, one per puller workgroup");

constexpr u32 IN_PER_CALL  = NXI + NT;
constexpr u32 CNT_PER_CALL = NR * 4;
constexpr u64 DEADLINE_TICKS = 100000000ull;

constexpr int EPI_LD   = TN + 4;
constexpr int SM_BYTES = TM * EPI_LD * 4;
constexpr int SM_TOTAL = SM_BYTES + 64;

constexpr int AUX_SYS   = 17;
constexpr int AUX_AGENT = 16;
#define AUX_OUT 16
constexpr int NUIN  = (NT + NXI - NPUSH) * 4;
constexpr int NUBIG = NUIN + (NSL * NT - TAILT) * 4;
constexpr int NUNIT = NUBIG + TAILT * 16;

#define STAMP(slot) do {} while (0)

struct Params {
  const unsigned char* x;
  const unsigned char* w;
  const float* sx;
  const float* sw;
  float* out;
  unsigned char* ws[NR];
  int rank;
  int nranks;
};

#define WAIT_VM(n)  asm volatile("s_waitcnt vmcnt(" #n ")" ::: "memory")
#define WAIT_LGKM0  do { __builtin_amdgcn_s_waitcnt(0xC07F); asm volatile("" ::: "memory"); } while (0)
#define RAW_BAR     __builtin_amdgcn_s_barrier()
#define SCHED_FENCE __builtin_amdgcn_sched_barrier(0)

__device__ __forceinline__ u32 ld_sys(const unsigned char* p) {
  return __hip_atomic_load((const u32*)p, __ATOMIC_RELAXED, __HIP_MEMORY_SCOPE_SYSTEM);
}
__device__ __forceinline__ u32 ld_agent(const unsigned char* p) {
  return __hip_atomic_load((const u32*)p, __ATOMIC_RELAXED, __HIP_MEMORY_SCOPE_AGENT);
}

__device__ __forceinline__ bool wait_ge(const unsigned char* flag, u32 target, unsigned char* tmo, u32 call) {
  if ((int)(ld_sys(flag) - target) >= 0) return true;
  const u64 t0 = __builtin_amdgcn_s_memrealtime();
  for (;;) {
    if ((int)(ld_sys(flag) - target) >= 0) return true;
    if (ld_agent(tmo) == call || __builtin_amdgcn_s_memrealtime() - t0 > DEADLINE_TICKS) {
      __hip_atomic_store((u32*)tmo, call, __ATOMIC_RELAXED, __HIP_MEMORY_SCOPE_AGENT);
      return false;
    }
    __builtin_amdgcn_s_sleep(2);
  }
}

__device__ __forceinline__ bool wait_flags(const unsigned char* base, int n, int lane, unsigned char* tmo, u32 call) {
  const u64 t0 = __builtin_amdgcn_s_memrealtime();
  for (;;) {
    const u32 f = lane < n ? ld_sys(base + lane * 128) : call;
    if (__builtin_amdgcn_ballot_w64((int)(f - call) < 0) == 0ull) return true;
    if (ld_agent(tmo) == call || __builtin_amdgcn_s_memrealtime() - t0 > DEADLINE_TICKS) {
      __hip_atomic_store((u32*)tmo, call, __ATOMIC_RELAXED, __HIP_MEMORY_SCOPE_AGENT);
      return false;
    }
    __builtin_amdgcn_s_sleep(2);
  }
}

__device__ __forceinline__ __amdgpu_buffer_rsrc_t mk_rsrc(const void* p, u32 bytes) {
  return __builtin_amdgcn_make_buffer_rsrc((void*)p, (short)0, (int)bytes, 0x00020000);
}

__global__ void __launch_bounds__(TPB, 1) fused_gemm_rs(Params P) {
  __shared__ __attribute__((aligned(16))) unsigned char smem[SM_TOTAL];
  const int tid  = threadIdx.x;
  const int lane = tid & 63;
  const int wave = tid >> 6;
  const int me   = P.rank;
  const int bid  = blockIdx.x;
  unsigned char* const my = P.ws[me];
  unsigned char* const tmo = my + OFF_TMO;
  volatile u32* const sm_word = (volatile u32*)(smem + SM_BYTES);
  const __amdgpu_buffer_rsrc_t rs_my = mk_rsrc(my, WS_USED);

  STAMP(0);
  if (tid == 0) {
    const u32 ticket = __hip_atomic_fetch_add((u32*)(my + OFF_TICKET + (bid & 15) * 128), 1u, __ATOMIC_RELAXED, __HIP_MEMORY_SCOPE_AGENT);
    sm_word[0] = ticket / (u32)(GRID / 16) + 1u;
  }

  if (bid < NPW) {
    const int p    = (me + 1 + bid % 7) & (NR - 1);
    const int pw   = bid / 7;
    v4u pv[16];
    if (pw < NPUSH) {
      const unsigned char* src = P.x + (size_t)(pw * MR + wave * 32 + (lane & 31)) * KS + 32 * (lane >> 5);
      #pragma unroll
      for (int j = 0; j < 16; ++j) pv[j] = *(const v4u*)(src + (j >> 1) * 64 + (j & 1) * 16);
    }
    __syncthreads();
    const u32 call = sm_word[0];
    const u32 par  = call & 1u;
    if ((unsigned)p >= (unsigned)P.nranks) return;
    const __amdgpu_buffer_rsrc_t rs_rp  = mk_rsrc(P.ws[p], WS_USED);
    const __amdgpu_buffer_rsrc_t rs_out = mk_rsrc(P.out, (u32)MR * NN * 4u);
    const u32 l16 = (u32)lane * 16u;

    u32* const uctr = (u32*)(my + OFF_UCTR + p * 128);
    const u32 ubase = (call - 1u) * (u32)(NUNIT + 4 * PWG);
    const u32 fl_base = OFF_OFLAG + (u32)p * 64u * 128u;
    u32 tk = 0;
    if (lane == 0) tk = __hip_atomic_fetch_add(uctr, 1u, __ATOMIC_RELAXED, __HIP_MEMORY_SCOPE_AGENT);
    if (pw < NPUSH) {
      const u32 dof = OFF_XP + ((par * NR + (u32)me) * PWG + (u32)pw) * ITEM + (u32)(wave * 2 * 64 + lane) * 16u;
      #pragma unroll
      for (int j = 0; j < 16; ++j)
        __builtin_amdgcn_raw_buffer_store_b128(pv[j], rs_rp, (int)(dof + (u32)((j >> 1) * 8192 + (j & 1) * 1024)), 0, AUX_SYS);
      WAIT_VM(0);
      if (lane == 0) __hip_atomic_fetch_add((u32*)(P.ws[p] + OFF_CNTP + (par * NR + (u32)pw) * 128u), 1u, __ATOMIC_RELAXED, __HIP_MEMORY_SCOPE_SYSTEM);
    }
    bool ok = wait_ge(my + OFF_INFLAG + p * 128, IN_PER_CALL * call, tmo, call);
    u32 u = __builtin_amdgcn_readfirstlane(tk) - ubase;
    STAMP(1);
    unsigned char* pend = nullptr;
    u32 fl = 0;
    bool pf = false;
    const u32 ftl = (u32)(NSL * NT - TAILT) + (u32)(lane < TAILT ? lane : 0);
    while (u < (u32)NUBIG) {
      u32 un;
      if (u < (u32)NUIN) {
        u32 so, dof; unsigned char* cnt;
        if (u < 64u) {
          const u32 t = u >> 2, q = u & 3u;
          so  = OFF_WT + ((par * NR + (u32)me) * NT + t) * ITEM + q * UNIT;
          dof = OFF_WL + ((u32)p * NT + t) * ITEM + q * UNIT;
          cnt = my + OFF_CNTW + t * 128u;
        } else {
          const u32 it = (u32)NPUSH + ((u - 64u) >> 2), q = u & 3u;
          so  = OFF_XT + (par * NXI + it) * ITEM + q * UNIT;
          dof = OFF_XL + ((u32)p * NXI + it) * ITEM + q * UNIT;
          cnt = my + OFF_CNTX + it * 128u;
        }
        if (ok) {
          #pragma unroll
          for (int r = 0; r < 4; ++r) {
            v4u v[4];
            #pragma unroll
            for (int j = 0; j < 4; ++j)
              v[j] = __builtin_amdgcn_raw_buffer_load_b128(rs_rp, (int)(so + r * 4096 + j * 1024 + l16), 0, AUX_SYS);
            if (r == 0 && lane == 0) tk = __hip_atomic_fetch_add(uctr, 1u, __ATOMIC_RELAXED, __HIP_MEMORY_SCOPE_AGENT);
            if (r == 1 && un >= (u32)NUIN && un < (u32)NUNIT) {
              pf = un >= (u32)NUBIG;
              fl = __builtin_amdgcn_raw_buffer_load_b32(rs_my, (int)(fl_base + (pf ? ftl : ((un - NUIN) >> 2)) * 128u), 0, AUX_SYS);
            }
            WAIT_VM(0);
            if (r == 0) {
              un = __builtin_amdgcn_readfirstlane(tk) - ubase;
              if (pend != nullptr && lane == 0) __hip_atomic_fetch_add((u32*)pend, 1u, __ATOMIC_RELAXED, __HIP_MEMORY_SCOPE_AGENT);
            }
            #pragma unroll
            for (int j = 0; j < 4; ++j)
              __builtin_amdgcn_raw_buffer_store_b128(v[j], rs_my, (int)(dof + r * 4096 + j * 1024 + l16), 0, AUX_AGENT);
          }
          pend = cnt;
        } else {
          if (lane == 0) tk = __hip_atomic_fetch_add(uctr, 1u, __ATOMIC_RELAXED, __HIP_MEMORY_SCOPE_AGENT);
          un = __builtin_amdgcn_readfirstlane(tk) - ubase;
        }
      } else {
        const u32 tile = (u - NUIN) >> 2, q = u & 3u;
        const u32 s = tile >> 4, t = tile & 15u;
        if (ok && (int)(fl - call) < 0) {
          if (pend != nullptr) {
            WAIT_VM(0);
            if (lane == 0) __hip_atomic_fetch_add((u32*)pend, 1u, __ATOMIC_RELAXED, __HIP_MEMORY_SCOPE_AGENT);
            pend = nullptr;
          }
          ok = wait_ge(my + fl_base + tile * 128u, call, tmo, call);
        }
        const u32 so = OFF_OUT + ((par * NR + (u32)me) * 64u + tile) * ITEM + q * UNIT;
        const u32 dof = (s * TM + q * 32u + (u32)(lane >> 5)) * (u32)(NN * 4) + ((u32)(p * NC) + t * TN) * 4u + (u32)(lane & 31) * 16u;
        if (ok) {
          #pragma unroll
          for (int r = 0; r < 4; ++r) {
            v4u v[4];
            #pragma unroll
            for (int j = 0; j < 4; ++j)
              v[j] = __builtin_amdgcn_raw_buffer_load_b128(rs_rp, (int)(so + r * 4096 + j * 1024 + l16), 0, AUX_SYS);
            if (r == 0 && lane == 0) tk = __hip_atomic_fetch_add(uctr, 1u, __ATOMIC_RELAXED, __HIP_MEMORY_SCOPE_AGENT);
            if (r == 1 && un >= (u32)NUIN && un < (u32)NUNIT) {
              pf = un >= (u32)NUBIG;
              fl = __builtin_amdgcn_raw_buffer_load_b32(rs_my, (int)(fl_base + (pf ? ftl : ((un - NUIN) >> 2)) * 128u), 0, AUX_SYS);
            }
            WAIT_VM(0);
            if (r == 0) {
              un = __builtin_amdgcn_readfirstlane(tk) - ubase;
              if (pend != nullptr && lane == 0) __hip_atomic_fetch_add((u32*)pend, 1u, __ATOMIC_RELAXED, __HIP_MEMORY_SCOPE_AGENT);
              pend = nullptr;
            }
            #pragma unroll
            for (int j = 0; j < 4; ++j)
              __builtin_amdgcn_raw_buffer_store_b128(v[j], rs_out, (int)(dof + (u32)(r * 8 + j * 2) * (u32)(NN * 4)), 0, AUX_OUT);
          }
        } else {
          if (lane == 0) tk = __hip_atomic_fetch_add(uctr, 1u, __ATOMIC_RELAXED, __HIP_MEMORY_SCOPE_AGENT);
          un = __builtin_amdgcn_readfirstlane(tk) - ubase;
          const v4u nanv = {0x7fc00000u, 0x7fc00000u, 0x7fc00000u, 0x7fc00000u};
          for (int r = 0; r < 16; ++r)
            __builtin_amdgcn_raw_buffer_store_b128(nanv, rs_out, (int)(dof + (u32)(r * 2) * (u32)(NN * 4)), 0, 0);
        }
      }
      u = un;
    }
    if (pend != nullptr) {
      WAIT_VM(0);
      if (lane == 0) __hip_atomic_fetch_add((u32*)pend, 1u, __ATOMIC_RELAXED, __HIP_MEMORY_SCOPE_AGENT);
    }
    if (TAILT > 0 && u < (u32)NUNIT) {
      if (ok && !(pf && __builtin_amdgcn_ballot_w64((int)(fl - call) < 0) == 0ull)) ok = wait_flags(my + fl_base + (u32)(NSL * NT - TAILT) * 128u, TAILT, lane, tmo, call);
      while (u < (u32)NUNIT) {
        const u32 tile = (u32)(NSL * NT - TAILT) + ((u - NUBIG) >> 4), sub = (u - NUBIG) & 15u;
        const u32 s = tile >> 4, t = tile & 15u;
        const u32 so = OFF_OUT + ((par * NR + (u32)me) * 64u + tile) * ITEM + sub * 4096u;
        const u32 dof = (s * TM + sub * 8u + (u32)(lane >> 5)) * (u32)(NN * 4) + ((u32)(p * NC) + t * TN) * 4u + (u32)(lane & 31) * 16u;
        v4u v[4];
        #pragma unroll
        for (int j = 0; j < 4; ++j) {
          if (ok) v[j] = __builtin_amdgcn_raw_buffer_load_b128(rs_rp, (int)(so + j * 1024 + l16), 0, AUX_SYS);
          else    v[j] = (v4u){0x7fc00000u, 0x7fc00000u, 0x7fc00000u, 0x7fc00000u};
        }
        if (lane == 0) tk = __hip_atomic_fetch_add(uctr, 1u, __ATOMIC_RELAXED, __HIP_MEMORY_SCOPE_AGENT);
        WAIT_VM(0);
        u = __builtin_amdgcn_readfirstlane(tk) - ubase;
        #pragma unroll
        for (int j = 0; j < 4; ++j)
          __builtin_amdgcn_raw_buffer_store_b128(v[j], rs_out, (int)(dof + (u32)(j * 2) * (u32)(NN * 4)), 0, AUX_OUT);
      }
    }
    STAMP(10);
    return;
  }

  const int g = bid - NPW;
  u32 call;
  if (g < NXI) {
    const int s = g >> 3, m = g & 7;
    const unsigned char* src = P.x + (size_t)(m * MR + s * TM + wave * 32 + (lane & 31)) * KS + 32 * (lane >> 5);
    v4u v[16];
    #pragma unroll
    for (int j = 0; j < 16; ++j) v[j] = *(const v4u*)(src + (j >> 1) * 64 + (j & 1) * 16);
    __syncthreads();
    call = sm_word[0];
    const u32 dof = OFF_XT + ((call & 1u) * NXI + (u32)g) * ITEM + (u32)(wave * 2 * 64 + lane) * 16u;
    #pragma unroll
    for (int j = 0; j < 16; ++j)
      __builtin_amdgcn_raw_buffer_store_b128(v[j], rs_my, (int)(dof + (u32)((j >> 1) * 8192 + (j & 1) * 1024)), 0, AUX_SYS);
    WAIT_VM(0);
    __syncthreads();
    if (tid < NR && tid < P.nranks) {
      if (tid == me) __hip_atomic_fetch_add((u32*)(my + OFF_CNTX + g * 128), 4u, __ATOMIC_RELAXED, __HIP_MEMORY_SCOPE_AGENT);
      else __hip_atomic_fetch_add((u32*)(P.ws[tid] + OFF_INFLAG + me * 128), 1u, __ATOMIC_RELAXED, __HIP_MEMORY_SCOPE_SYSTEM);
    }
  } else if (g < NPREP) {
    const int it = g - NXI;
    const int j = it >> 4, t = it & 15;
    const int q = lane & 31;
    const unsigned char* src = P.w + (size_t)(j * NC + t * TN + q * 4);
    u32 r[4][16];
    #pragma unroll
    for (int a = 0; a < 4; ++a) {
      const int kg = a * 8 + wave * 2 + (lane >> 5);
      #pragma unroll
      for (int i = 0; i < 16; ++i) r[a][i] = *(const u32*)(src + (size_t)(kg * 16 + i) * NN);
    }
    #pragma unroll
    for (int a = 0; a < 4; ++a) {
      const int kg = a * 8 + wave * 2 + (lane >> 5);
      u32 o[4][4];
      #pragma unroll
      for (int b = 0; b < 4; ++b) {
        const u32 r0 = r[a][4 * b], r1 = r[a][4 * b + 1], r2 = r[a][4 * b + 2], r3 = r[a][4 * b + 3];
        const u32 t0 = __builtin_amdgcn_perm(r1, r0, 0x05010400u), t1 = __builtin_amdgcn_perm(r1, r0, 0x07030602u);
        const u32 t2 = __builtin_amdgcn_perm(r3, r2, 0x05010400u), t3 = __builtin_amdgcn_perm(r3, r2, 0x07030602u);
        o[0][b] = __builtin_amdgcn_perm(t2, t0, 0x05040100u);
        o[1][b] = __builtin_amdgcn_perm(t2, t0, 0x07060302u);
        o[2][b] = __builtin_amdgcn_perm(t3, t1, 0x05040100u);
        o[3][b] = __builtin_amdgcn_perm(t3, t1, 0x07060302u);
      }
      const int kb = kg >> 2, half = (kg >> 1) & 1, part = kg & 1, blk = q >> 3;
      const int c0 = ((kb * 4 + blk) * 2 + part) * 64 + half * 32 + (q & 7) * 4;
      #pragma unroll
      for (int c = 0; c < 4; ++c) {
        const int ch = c0 + c;
        const v4u ov = {o[c][0], o[c][1], o[c][2], o[c][3]};
        *(v4u*)(smem + (ch ^ ((ch >> 3) & 3)) * 16) = ov;
      }
    }
    __syncthreads();
    call = sm_word[0];
    const u32 dof = OFF_WT + (((call & 1u) * NR + (u32)j) * NT + (u32)t) * ITEM;
    #pragma unroll
    for (int i = 0; i < 16; ++i) {
      const int ch = i * 256 + tid;
      const v4u ov = *(const v4u*)(smem + (ch ^ ((ch >> 3) & 3)) * 16);
      __builtin_amdgcn_raw_buffer_store_b128(ov, rs_my, (int)(dof + (u32)ch * 16u), 0, AUX_SYS);
    }
    WAIT_VM(0);
    __syncthreads();
    if (tid == 0 && j < P.nranks) {
      if (j == me) __hip_atomic_fetch_add((u32*)(my + OFF_CNTW + t * 128), 4u, __ATOMIC_RELAXED, __HIP_MEMORY_SCOPE_AGENT);
      else __hip_atomic_fetch_add((u32*)(P.ws[j] + OFF_INFLAG + me * 128), 1u, __ATOMIC_RELAXED, __HIP_MEMORY_SCOPE_SYSTEM);
    }
  } else {
    __syncthreads();
    call = sm_word[0];
  }
  STAMP(1);
  const u32 par = call & 1u;
  const float scale = P.sx[0] * P.sw[0];
  const int wm = wave & 1, wn = wave >> 1;

  const int grp = bid & 7;
  const int gfirst = NPW + ((grp - NPW) & 7);
  const int gcnt = (GRID - 1 - gfirst) / 8 + 1;
  for (int T = (bid - gfirst) >> 3; T < NTILE / 8; T += gcnt) {
    const int s = T >> 4, t = (grp >> 1) * 4 + ((T >> 2) & 3), m = (grp & 1) * 4 + (T & 3);
    const int xi = s * 8 + m;
    __syncthreads();
    if (tid == 0) {
      bool ok = wait_ge(my + OFF_CNTW + t * 128, CNT_PER_CALL * call, tmo, call);
      if (xi < NPUSH) {
        if (ok) ok = wait_ge(my + OFF_CNTX + xi * 128, 4u * call, tmo, call);
        if (ok) ok = wait_ge(my + OFF_CNTP + (par * NR + (u32)xi) * 128u, (CNT_PER_CALL - 4u) * ((call + 1u) >> 1), tmo, call);
        __builtin_amdgcn_fence(__ATOMIC_ACQUIRE, "");
      } else {
        if (ok) ok = wait_ge(my + OFF_CNTX + xi * 128, CNT_PER_CALL * call, tmo, call);
        __builtin_amdgcn_fence(__ATOMIC_ACQUIRE, "agent");
      }
      WAIT_VM(0);
      if (ok) ok = (ld_agent(tmo) != call);
      sm_word[1] = ok ? 1u : 0u;
    }
    __syncthreads();
    const bool ok = sm_word[1] != 0u;
    STAMP(2 + 3 * (T / gcnt));

    v16f acc[2][2];
    #pragma unroll
    for (int i = 0; i < 2; ++i)
      #pragma unroll
      for (int j = 0; j < 2; ++j)
        #pragma unroll
        for (int e = 0; e < 16; ++e) acc[i][j][e] = 0.f;

    if (ok) {
      const unsigned char* xo = my + OFF_XT + (par * NXI + (u32)xi) * ITEM + tid * 16;
      const unsigned char* xl = (xi < NPUSH ? my + OFF_XP + (par * NR * PWG + (u32)xi) * ITEM : my + OFF_XL + (u32)xi * ITEM) + tid * 16;
      const u32 xstr = xi < NPUSH ? (u32)PWG * ITEM : (u32)NXI * ITEM;
      const unsigned char* wo = my + OFF_WT + ((par * NR + (u32)me) * NT + (u32)t) * ITEM + tid * 16;
      const unsigned char* wl = my + OFF_WL + (u32)t * ITEM + tid * 16;
      unsigned char* const ldst = smem + wave * 1024;
      auto issue = [&](int ks) {
        const int sr = ks >> 3, kb = ks & 7;
        const unsigned char* xa = (sr == me ? xo : xl + (size_t)((u32)sr * xstr)) + kb * 8192;
        const unsigned char* wa = (sr == me ? wo : wl + (size_t)sr * (NT * ITEM)) + kb * 8192;
        unsigned char* l = ldst + (ks & 3) * 16384;
        __builtin_amdgcn_global_load_lds((const __attribute__((address_space(1))) void*)(xa),        (__attribute__((address_space(3))) void*)(l),         16, 0, 0);
        __builtin_amdgcn_global_load_lds((const __attribute__((address_space(1))) void*)(xa + 4096), (__attribute__((address_space(3))) void*)(l + 4096),  16, 0, 0);
        __builtin_amdgcn_global_load_lds((const __attribute__((address_space(1))) void*)(wa),        (__attribute__((address_space(3))) void*)(l + 8192),  16, 0, 0);
        __builtin_amdgcn_global_load_lds((const __attribute__((address_space(1))) void*)(wa + 4096), (__attribute__((address_space(3))) void*)(l + 12288), 16, 0, 0);
      };
      const unsigned char* const fx = smem + (wm * 2) * 2048 + lane * 16;
      const unsigned char* const fw = smem + 8192 + (wn * 2) * 2048 + lane * 16;
      auto rdfrag = [&](int ks, v8i (&X)[2], v8i (&W)[2]) {
        const int so = (ks & 3) * 16384;
        #pragma unroll
        for (int b = 0; b < 2; ++b) {
          const v4u x0 = *(const v4u*)(fx + so + b * 2048), x1 = *(const v4u*)(fx + so + b * 2048 + 1024);
          const v4u w0 = *(const v4u*)(fw + so + b * 2048), w1 = *(const v4u*)(fw + so + b * 2048 + 1024);
          X[b] = (v8i){(int)x0[0], (int)x0[1], (int)x0[2], (int)x0[3], (int)x1[0], (int)x1[1], (int)x1[2], (int)x1[3]};
          W[b] = (v8i){(int)w0[0], (int)w0[1], (int)w0[2], (int)w0[3], (int)w1[0], (int)w1[1], (int)w1[2], (int)w1[3]};
        }
      };
      auto mma = [&](v8i (&X)[2], v8i (&W)[2]) {
        #pragma unroll
        for (int i = 0; i < 2; ++i)
          #pragma unroll
          for (int j = 0; j < 2; ++j)
            acc[i][j] = __builtin_amdgcn_mfma_scale_f32_32x32x64_f8f6f4(W[i], X[j], acc[i][j], 1, 0, 0, 0x7F7F7F7F, 0, 0x7F7F7F7F);
      };
      v8i XA[2], WA[2], XB[2], WB[2];
      issue(0); issue(1); issue(2);
      WAIT_VM(8); RAW_BAR;
      issue(3);
      rdfrag(0, XA, WA);
      for (int ks = 0; ks < 60; ks += 2) {
        WAIT_VM(8); WAIT_LGKM0; RAW_BAR;
        issue(ks + 4);
        rdfrag(ks + 1, XB, WB);
        SCHED_FENCE;
        mma(XA, WA);
        SCHED_FENCE;
        WAIT_VM(8); WAIT_LGKM0; RAW_BAR;
        issue(ks + 5);
        rdfrag(ks + 2, XA, WA);
        SCHED_FENCE;
        mma(XB, WB);
        SCHED_FENCE;
      }
      WAIT_VM(8); WAIT_LGKM0; RAW_BAR;
      rdfrag(61, XB, WB); SCHED_FENCE; mma(XA, WA); SCHED_FENCE;
      WAIT_VM(4); WAIT_LGKM0; RAW_BAR;
      rdfrag(62, XA, WA); SCHED_FENCE; mma(XB, WB); SCHED_FENCE;
      WAIT_VM(0); WAIT_LGKM0; RAW_BAR;
      rdfrag(63, XB, WB); SCHED_FENCE; mma(XA, WA); SCHED_FENCE;
      mma(XB, WB);
    }
    STAMP(3 + 3 * (T / gcnt));
    __syncthreads();
    {
      float* img = (float*)smem;
      const float nanf_ = __builtin_nanf("");
      #pragma unroll
      for (int i = 0; i < 2; ++i)
        #pragma unroll
        for (int j = 0; j < 2; ++j)
          #pragma unroll
          for (int q = 0; q < 4; ++q) {
            v4f o;
            #pragma unroll
            for (int e = 0; e < 4; ++e) o[e] = ok ? acc[i][j][4 * q + e] * scale : nanf_;
            const int row = wm * 64 + j * 32 + (lane & 31);
            const int col = wn * 64 + i * 32 + 8 * q + 4 * (lane >> 5);
            *(v4f*)(img + row * EPI_LD + col) = o;
          }
      __syncthreads();
      const int r0 = wave * 2 + (lane >> 5), c4 = (lane & 31) * 4;
      if (m == me) {
        float* dst = P.out + (size_t)(s * TM + r0) * NN + me * NC + t * TN + c4;
        #pragma unroll
        for (int i = 0; i < 16; ++i) {
          const v4f o = *(const v4f*)(img + (r0 + i * 8) * EPI_LD + c4);
          *(v4f*)(dst + (size_t)(i * 8) * NN) = o;
        }
      } else {
        const u32 dof = OFF_OUT + ((par * NR + (u32)m) * 64u + (u32)(s * NT + t)) * ITEM + (u32)(r0 * 512 + c4 * 4);
        #pragma unroll
        for (int i = 0; i < 16; ++i) {
          const v4f o = *(const v4f*)(img + (r0 + i * 8) * EPI_LD + c4);
          __builtin_amdgcn_raw_buffer_store_b128(__builtin_bit_cast(v4u, o), rs_my, (int)(dof + (u32)(i * 8 * 512)), 0, AUX_SYS);
        }
        WAIT_VM(0);
        __syncthreads();
        if (tid == 0 && m < P.nranks)
          __hip_atomic_store((u32*)(P.ws[m] + OFF_OFLAG + (u32)(me * 64 + s * NT + t) * 128u), call, __ATOMIC_RELAXED, __HIP_MEMORY_SCOPE_SYSTEM);
      }
    }
    STAMP(4 + 3 * (T / gcnt));
  }
}

extern "C" void kernel_launch(int rank, int nranks, ncclComm_t comm,
                              hipStream_t stream,
                              void* const* d_in, const int* in_sizes, int n_in,
                              void* d_out, int out_size,
                              const int* in_dtypes, int out_dtype,
                              void* const* d_out_all,
                              void* const* d_ws_all, size_t ws_bytes) {
  (void)comm; (void)in_sizes; (void)in_dtypes; (void)out_dtype; (void)d_out_all;
  if (nranks != NR || n_in < 4 || ws_bytes < (size_t)WS_USED || out_size != MR * NN || rank < 0 || rank >= NR) {
    fprintf(stderr, "unexpected configuration\n");
    return;
  }
  Params P;
  P.x  = (const unsigned char*)d_in[0];
  P.w  = (const unsigned char*)d_in[1];
  P.sx = (const float*)d_in[2];
  P.sw = (const float*)d_in[3];
  P.out = (float*)d_out;
  for (int p = 0; p < NR; ++p) P.ws[p] = (unsigned char*)d_ws_all[p];
  P.rank = rank;
  P.nranks = nranks;
  fused_gemm_rs<<<GRID, TPB, 0, stream>>>(P);
  HIP_CHECK(hipGetLastError());
}
```

```cpp
#include <hip/hip_runtime.h>
#include <rccl/rccl.h>
#include <cstdio>
#include <cstdint>

#define HIP_CHECK(expr) do { hipError_t _e = (expr); if (_e != hipSuccess) { \
    fprintf(stderr, "HIP error %s at %s:%d\n", hipGetErrorString(_e), __FILE__, __LINE__); \
    return; } } while (0)
#define NCCL_CHECK(expr) do { ncclResult_t _r = (expr); if (_r != ncclSuccess) { \
    fprintf(stderr, "RCCL error %s at %s:%d\n", ncclGetErrorString(_r), __FILE__, __LINE__); \
    return; } } while (0)

typedef unsigned u32;
typedef unsigned long long u64;
typedef int      v8i  __attribute__((ext_vector_type(8)));
typedef unsigned v4u  __attribute__((ext_vector_type(4)));
typedef float    v4f  __attribute__((ext_vector_type(4)));
typedef float    v16f __attribute__((ext_vector_type(16)));

constexpr int NR    = 8;
constexpr int MM    = 4096;
constexpr int KK    = 4096;
constexpr int NN    = 16384;
constexpr int KS    = KK / NR;
constexpr int NC    = NN / NR;
constexpr int MR    = MM / NR;
constexpr int TM    = 128;
constexpr int TN    = 128;
constexpr int NSL   = MR / TM;
constexpr int NT    = NC / TN;
constexpr int NXI   = NSL * NR;
constexpr int NTILE = NXI * NT;
constexpr u32 ITEM  = 65536;
constexpr u32 UNIT  = 16384;

constexpr int GRID  = 256;
constexpr int TPB   = 256;
#define PWG 6
constexpr int NPW   = 7 * PWG;
constexpr int NGW   = GRID - NPW;
constexpr int NPREP = NXI + NR * NT;

constexpr u32 OFF_TICKET = 0;
constexpr u32 OFF_TMO    = 2048;
constexpr u32 OFF_PDONE  = 2560;
constexpr u32 OFF_PTK    = 2816;
constexpr u32 OFF_UCTR   = 3072;
constexpr u32 OFF_INFLAG = 4096;
constexpr u32 OFF_CNTX   = 8192;
constexpr u32 OFF_CNTW   = 16384;
constexpr u32 OFF_CNTP   = 24576;
constexpr u32 OFF_PCNT   = 28672;
constexpr u32 OFF_OFLAG  = 65536;
constexpr u32 OFF_XT     = 1u << 20;
constexpr u32 OFF_WT     = OFF_XT + 2u * NXI * ITEM;
constexpr u32 OFF_XL     = OFF_WT + 2u * NR * NT * ITEM;
constexpr u32 OFF_WL     = OFF_XL + (u32)NR * NXI * ITEM;
constexpr u32 OFF_OUT    = OFF_WL + (u32)NR * NT * ITEM;
#define NPUSH 4
#define TAILT 4
constexpr u32 OFF_XP     = OFF_OUT + 2u * NR * (NSL * NT) * ITEM;
constexpr u32 WS_USED    = OFF_XP + 2u * NR * NR * ITEM;
static_assert(NPUSH <= NR - 1 && NPREP + 7 * NPUSH <= NGW, "pushed x items: row blocks of slice 0, one per otherwise idle workgroup");

constexpr u32 CNT_PER_CALL = NR * 4;
constexpr u64 DEADLINE_TICKS = 100000000ull;

constexpr int EPI_LD   = TN + 4;
constexpr int SM_BYTES = TM * EPI_LD * 4;
constexpr int SM_TOTAL = SM_BYTES + 64;

constexpr int AUX_SYS   = 17;
constexpr int AUX_AGENT = 16;
#define AUX_OUT 16
#define AUX_PLD 16
#define AUX_PST 16
constexpr int NPT   = NSL * (NR - 1) * NT;
constexpr int NUX0  = (NR - 1 - NPUSH) * 4;
constexpr int NUW   = NUX0 + NT * 4;
constexpr int NUIN  = (NT + NXI - NSL - NPUSH) * 4;
constexpr int NUBIG = NUIN + (NSL * NT - TAILT) * 4;
constexpr int NUNIT = NUBIG + TAILT * 16;

#define STAMP(slot) do {} while (0)
#define USTAMP(g, u) do {} while (0)

struct Params {
  const unsigned char* x;
  const unsigned char* w;
  const float* sx;
  const float* sw;
  float* out;
  unsigned char* ws[NR];
  int rank;
  int nranks;
};

#define WAIT_VM(n)  asm volatile("s_waitcnt vmcnt(" #n ")" ::: "memory")
#define WAIT_LGKM0  do { __builtin_amdgcn_s_waitcnt(0xC07F); asm volatile("" ::: "memory"); } while (0)
#define RAW_BAR     __builtin_amdgcn_s_barrier()
#define SCHED_FENCE __builtin_amdgcn_sched_barrier(0)

__device__ __forceinline__ u32 ld_sys(const unsigned char* p) {
  return __hip_atomic_load((const u32*)p, __ATOMIC_RELAXED, __HIP_MEMORY_SCOPE_SYSTEM);
}
__device__ __forceinline__ u32 ld_agent(const unsigned char* p) {
  return __hip_atomic_load((const u32*)p, __ATOMIC_RELAXED, __HIP_MEMORY_SCOPE_AGENT);
}

__device__ __forceinline__ bool wait_ge(const unsigned char* flag, u32 target, unsigned char* tmo, u32 call) {
  if ((int)(ld_sys(flag) - target) >= 0) return true;
  const u64 t0 = __builtin_amdgcn_s_memrealtime();
  for (;;) {
    if ((int)(ld_sys(flag) - target) >= 0) return true;
    if (ld_agent(tmo) == call || __builtin_amdgcn_s_memrealtime() - t0 > DEADLINE_TICKS) {
      __hip_atomic_store((u32*)tmo, call, __ATOMIC_RELAXED, __HIP_MEMORY_SCOPE_AGENT);
      return false;
    }
    __builtin_amdgcn_s_sleep(2);
  }
}

__device__ __forceinline__ bool wait_flags(const unsigned char* base, int n, int lane, u32 target, unsigned char* tmo, u32 call) {
  const u64 t0 = __builtin_amdgcn_s_memrealtime();
  for (;;) {
    const u32 f = lane < n ? ld_sys(base + lane * 128) : target;
    if (__builtin_amdgcn_ballot_w64((int)(f - target) < 0) == 0ull) return true;
    if (ld_agent(tmo) == call || __builtin_amdgcn_s_memrealtime() - t0 > DEADLINE_TICKS) {
      __hip_atomic_store((u32*)tmo, call, __ATOMIC_RELAXED, __HIP_MEMORY_SCOPE_AGENT);
      return false;
    }
    __builtin_amdgcn_s_sleep(2);
  }
}

__device__ __forceinline__ __amdgpu_buffer_rsrc_t mk_rsrc(const void* p, u32 bytes) {
  return __builtin_amdgcn_make_buffer_rsrc((void*)p, (short)0, (int)bytes, 0x00020000);
}

__global__ void __launch_bounds__(TPB, 1) fused_gemm_rs(Params P) {
  __shared__ __attribute__((aligned(16))) unsigned char smem[SM_TOTAL];
  const int tid  = threadIdx.x;
  const int lane = tid & 63;
  const int wave = tid >> 6;
  const int me   = P.rank;
  const int bid  = blockIdx.x;
  unsigned char* const my = P.ws[me];
  unsigned char* const tmo = my + OFF_TMO;
  volatile u32* const sm_word = (volatile u32*)(smem + SM_BYTES);
  const __amdgpu_buffer_rsrc_t rs_my = mk_rsrc(my, WS_USED);

  STAMP(0);
  if (tid == 0) {
    const u32 ticket = __hip_atomic_fetch_add((u32*)(my + OFF_TICKET + (bid & 15) * 128), 1u, __ATOMIC_RELAXED, __HIP_MEMORY_SCOPE_AGENT);
    sm_word[0] = ticket / (u32)(GRID / 16) + 1u;
  }

  if (bid < NPW) {
    const int p    = (me + 1 + bid % 7) & (NR - 1);
    __syncthreads();
    const u32 call = sm_word[0];
    const u32 par  = call & 1u;
    if ((unsigned)p >= (unsigned)P.nranks) return;
    const __amdgpu_buffer_rsrc_t rs_rp  = mk_rsrc(P.ws[p], WS_USED);
    const __amdgpu_buffer_rsrc_t rs_out = mk_rsrc(P.out, (u32)MR * NN * 4u);
    const u32 l16 = (u32)lane * 16u;

    u32* const uctr = (u32*)(my + OFF_UCTR + p * 128);
    const u32 ubase = (call - 1u) * (u32)(NUNIT + 4 * PWG);
    const u32 fl_base = OFF_OFLAG + (u32)p * 64u * 128u;
    u32 tk = 0;
    if (lane == 0) tk = __hip_atomic_fetch_add(uctr, 1u, __ATOMIC_RELAXED, __HIP_MEMORY_SCOPE_AGENT);
    bool ok = true;
    u32 u = __builtin_amdgcn_readfirstlane(tk) - ubase;
    unsigned char* pend = nullptr;
    u32 fl = 0;
    bool pf = false;
    u32 psl = 99u;
    u32 iph = 99u;
    const u32 ftl = (u32)(NSL * NT - TAILT) + (u32)(lane < TAILT ? lane : 0);
    while (u < (u32)NUBIG) {
      u32 un;
      if (u < (u32)NUIN) {
        const u32 ph = u < (u32)NUX0 ? 0u : u < (u32)NUW ? 1u : 2u;
        if (ok && ph != iph) {
          if (pend != nullptr) {
            WAIT_VM(0);
            if (lane == 0) __hip_atomic_fetch_add((u32*)pend, 1u, __ATOMIC_RELAXED, __HIP_MEMORY_SCOPE_AGENT);
            pend = nullptr;
          }
          ok = wait_ge(my + OFF_INFLAG + p * 128 + ph * 4u, (ph + 1u) * (u32)NR * call, tmo, call);
          iph = ph;
          if (ph == 0u) STAMP(1);
          if (ph == 1u) STAMP(11);
        }
        u32 so, dof; unsigned char* cnt;
        if (ph == 1u) {
          const u32 t = (u - NUX0) >> 2, q = u & 3u;
          so  = OFF_WT + ((par * NR + (u32)me) * NT + t) * ITEM + q * UNIT;
          dof = OFF_WL + ((u32)p * NT + t) * ITEM + q * UNIT;
          cnt = my + OFF_CNTW + t * 128u;
        } else {
          const u32 z = (ph == 0u ? u : u - 64u) >> 2, q = u & 3u;
          u32 it;
          if (z < (u32)(NR - 1 - NPUSH)) it = ((u32)p + 1u + NPUSH + z) & (NR - 1);
          else { const u32 y = z - (u32)(NR - 1 - NPUSH), i = y % 7u; it = (1u + y / 7u) * 8u + i + (i >= (u32)p ? 1u : 0u); }
          so  = OFF_XT + (par * NXI + it) * ITEM + q * UNIT;
          dof = OFF_XL + ((u32)p * NXI + it) * ITEM + q * UNIT;
          cnt = my + OFF_CNTX + it * 128u;
        }
        if (ok) {
          #pragma unroll
          for (int r = 0; r < 4; ++r) {
            v4u v[4];
            #pragma unroll
            for (int j = 0; j < 4; ++j)
              v[j] = __builtin_amdgcn_raw_buffer_load_b128(rs_rp, (int)(so + r * 4096 + j * 1024 + l16), 0, AUX_SYS);
            if (r == 0 && lane == 0) tk = __hip_atomic_fetch_add(uctr, 1u, __ATOMIC_RELAXED, __HIP_MEMORY_SCOPE_AGENT);
            if (r == 1 && un >= (u32)NUIN && un < (u32)NUNIT) {
              pf = un >= (u32)NUBIG;
              fl = __builtin_amdgcn_raw_buffer_load_b32(rs_my, (int)(fl_base + (pf ? ftl : ((un - NUIN) >> 2)) * 128u), 0, AUX_SYS);
            }
            WAIT_VM(0);
            if (r == 0) {
              un = __builtin_amdgcn_readfirstlane(tk) - ubase;
              if (pend != nullptr && lane == 0) __hip_atomic_fetch_add((u32*)pend, 1u, __ATOMIC_RELAXED, __HIP_MEMORY_SCOPE_AGENT);
            }
            #pragma unroll
            for (int j = 0; j < 4; ++j)
              __builtin_amdgcn_raw_buffer_store_b128(v[j], rs_my, (int)(dof + r * 4096 + j * 1024 + l16), 0, AUX_AGENT);
          }
          pend = cnt;
        } else {
          if (lane == 0) tk = __hip_atomic_fetch_add(uctr, 1u, __ATOMIC_RELAXED, __HIP_MEMORY_SCOPE_AGENT);
          un = __builtin_amdgcn_readfirstlane(tk) - ubase;
        }
      } else {
        const u32 tile = (u - NUIN) >> 2, q = u & 3u;
        const u32 s = tile >> 4, t = tile & 15u;
        if (ok && (psl != s || (int)(fl - call) < 0)) {
          if (pend != nullptr) {
            WAIT_VM(0);
            if (lane == 0) __hip_atomic_fetch_add((u32*)pend, 1u, __ATOMIC_RELAXED, __HIP_MEMORY_SCOPE_AGENT);
            pend = nullptr;
          }
          if (psl != s) ok = wait_ge(my + OFF_PCNT + s * 128u, (u32)(NPT / NSL) * call, tmo, call);
          psl = s;
          if (ok && (int)(fl - call) < 0) ok = wait_ge(my + fl_base + tile * 128u, call, tmo, call);
        }
        const u32 so = OFF_OUT + ((par * NR + (u32)me) * 64u + tile) * ITEM + q * UNIT;
        const u32 dof = (s * TM + q * 32u + (u32)(lane >> 5)) * (u32)(NN * 4) + ((u32)(p * NC) + t * TN) * 4u + (u32)(lane & 31) * 16u;
        if (ok) {
          #pragma unroll
          for (int r = 0; r < 4; ++r) {
            v4u v[4], a[4];
            #pragma unroll
            for (int j = 0; j < 4; ++j)
              v[j] = __builtin_amdgcn_raw_buffer_load_b128(rs_rp, (int)(so + r * 4096 + j * 1024 + l16), 0, AUX_SYS);
            #pragma unroll
            for (int j = 0; j < 4; ++j)
              a[j] = __builtin_amdgcn_raw_buffer_load_b128(rs_out, (int)(dof + (u32)(r * 8 + j * 2) * (u32)(NN * 4)), 0, AUX_PLD);
            if (r == 0 && lane == 0) tk = __hip_atomic_fetch_add(uctr, 1u, __ATOMIC_RELAXED, __HIP_MEMORY_SCOPE_AGENT);
            if (r == 1 && un >= (u32)NUIN && un < (u32)NUNIT) {
              pf = un >= (u32)NUBIG;
              fl = __builtin_amdgcn_raw_buffer_load_b32(rs_my, (int)(fl_base + (pf ? ftl : ((un - NUIN) >> 2)) * 128u), 0, AUX_SYS);
            }
            WAIT_VM(0);
            if (r == 0) {
              un = __builtin_amdgcn_readfirstlane(tk) - ubase;
              if (pend != nullptr && lane == 0) __hip_atomic_fetch_add((u32*)pend, 1u, __ATOMIC_RELAXED, __HIP_MEMORY_SCOPE_AGENT);
              pend = nullptr;
            }
            #pragma unroll
            for (int j = 0; j < 4; ++j) {
              const v4f o = __builtin_bit_cast(v4f, v[j]) + __builtin_bit_cast(v4f, a[j]);
              __builtin_amdgcn_raw_buffer_store_b128(__builtin_bit_cast(v4u, o), rs_out, (int)(dof + (u32)(r * 8 + j * 2) * (u32)(NN * 4)), 0, AUX_OUT);
            }
          }
        } else {
          if (lane == 0) tk = __hip_atomic_fetch_add(uctr, 1u, __ATOMIC_RELAXED, __HIP_MEMORY_SCOPE_AGENT);
          un = __builtin_amdgcn_readfirstlane(tk) - ubase;
          const v4u nanv = {0x7fc00000u, 0x7fc00000u, 0x7fc00000u, 0x7fc00000u};
          for (int r = 0; r < 16; ++r)
            __builtin_amdgcn_raw_buffer_store_b128(nanv, rs_out, (int)(dof + (u32)(r * 2) * (u32)(NN * 4)), 0, 0);
        }
      }
      USTAMP(bid % 7, u);
      u = un;
    }
    if (pend != nullptr) {
      WAIT_VM(0);
      if (lane == 0) __hip_atomic_fetch_add((u32*)pend, 1u, __ATOMIC_RELAXED, __HIP_MEMORY_SCOPE_AGENT);
    }
    if (TAILT > 0 && u < (u32)NUNIT) {
      if (ok && !(pf && __builtin_amdgcn_ballot_w64((int)(fl - call) < 0) == 0ull)) ok = wait_flags(my + fl_base + (u32)(NSL * NT - TAILT) * 128u, TAILT, lane, call, tmo, call);
      if (ok && psl != (u32)(NSL - 1)) ok = wait_ge(my + OFF_PCNT + (NSL - 1) * 128u, (u32)(NPT / NSL) * call, tmo, call);
      while (u < (u32)NUNIT) {
        const u32 tile = (u32)(NSL * NT - TAILT) + ((u - NUBIG) >> 4), sub = (u - NUBIG) & 15u;
        const u32 s = tile >> 4, t = tile & 15u;
        const u32 so = OFF_OUT + ((par * NR + (u32)me) * 64u + tile) * ITEM + sub * 4096u;
        const u32 dof = (s * TM + sub * 8u + (u32)(lane >> 5)) * (u32)(NN * 4) + ((u32)(p * NC) + t * TN) * 4u + (u32)(lane & 31) * 16u;
        v4u v[4], a[4];
        #pragma unroll
        for (int j = 0; j < 4; ++j) {
          if (ok) v[j] = __builtin_amdgcn_raw_buffer_load_b128(rs_rp, (int)(so + j * 1024 + l16), 0, AUX_SYS);
          else    v[j] = (v4u){0x7fc00000u, 0x7fc00000u, 0x7fc00000u, 0x7fc00000u};
        }
        #pragma unroll
        for (int j = 0; j < 4; ++j)
          a[j] = __builtin_amdgcn_raw_buffer_load_b128(rs_out, (int)(dof + (u32)(j * 2) * (u32)(NN * 4)), 0, AUX_PLD);
        if (lane == 0) tk = __hip_atomic_fetch_add(uctr, 1u, __ATOMIC_RELAXED, __HIP_MEMORY_SCOPE_AGENT);
        WAIT_VM(0);
        u = __builtin_amdgcn_readfirstlane(tk) - ubase;
        #pragma unroll
        for (int j = 0; j < 4; ++j) {
          const v4f o = __builtin_bit_cast(v4f, v[j]) + __builtin_bit_cast(v4f, a[j]);
          __builtin_amdgcn_raw_buffer_store_b128(__builtin_bit_cast(v4u, o), rs_out, (int)(dof + (u32)(j * 2) * (u32)(NN * 4)), 0, AUX_OUT);
        }
      }
    }
    STAMP(10);
    return;
  }

  const int g = bid - NPW;
  u32 call;
  if (g < NXI) {
    const int s = g >> 3, m = g & 7;
    const unsigned char* src = P.x + (size_t)(m * MR + s * TM + wave * 32 + (lane & 31)) * KS + 32 * (lane >> 5);
    v4u v[16];
    #pragma unroll
    for (int j = 0; j < 16; ++j) v[j] = *(const v4u*)(src + (j >> 1) * 64 + (j & 1) * 16);
    __syncthreads();
    call = sm_word[0];
    const u32 dof = OFF_XT + ((call & 1u) * NXI + (u32)g) * ITEM + (u32)(wave * 2 * 64 + lane) * 16u;
    #pragma unroll
    for (int j = 0; j < 16; ++j)
      __builtin_amdgcn_raw_buffer_store_b128(v[j], rs_my, (int)(dof + (u32)((j >> 1) * 8192 + (j & 1) * 1024)), 0, AUX_SYS);
    WAIT_VM(0);
    __syncthreads();
    if (tid < NR && tid < P.nranks) {
      if (tid == me) __hip_atomic_fetch_add((u32*)(my + OFF_CNTX + g * 128), 4u, __ATOMIC_RELAXED, __HIP_MEMORY_SCOPE_AGENT);
      else __hip_atomic_fetch_add((u32*)(P.ws[tid] + OFF_INFLAG + me * 128 + (s == 0 ? 0 : 8)), 1u, __ATOMIC_RELAXED, __HIP_MEMORY_SCOPE_SYSTEM);
      if (tid == 0) __hip_atomic_fetch_add((u32*)(my + OFF_PDONE), 1u, __ATOMIC_RELAXED, __HIP_MEMORY_SCOPE_AGENT);
    }
  } else if (g < NPREP) {
    const int it = g - NXI;
    const int j = it >> 4, t = it & 15;
    const int q = lane & 31;
    const unsigned char* src = P.w + (size_t)(j * NC + t * TN + q * 4);
    u32 r[4][16];
    #pragma unroll
    for (int a = 0; a < 4; ++a) {
      const int kg = a * 8 + wave * 2 + (lane >> 5);
      #pragma unroll
      for (int i = 0; i < 16; ++i) r[a][i] = *(const u32*)(src + (size_t)(kg * 16 + i) * NN);
    }
    #pragma unroll
    for (int a = 0; a < 4; ++a) {
      const int kg = a * 8 + wave * 2 + (lane >> 5);
      u32 o[4][4];
      #pragma unroll
      for (int b = 0; b < 4; ++b) {
        const u32 r0 = r[a][4 * b], r1 = r[a][4 * b + 1], r2 = r[a][4 * b + 2], r3 = r[a][4 * b + 3];
        const u32 t0 = __builtin_amdgcn_perm(r1, r0, 0x05010400u), t1 = __builtin_amdgcn_perm(r1, r0, 0x07030602u);
        const u32 t2 = __builtin_amdgcn_perm(r3, r2, 0x05010400u), t3 = __builtin_amdgcn_perm(r3, r2, 0x07030602u);
        o[0][b] = __builtin_amdgcn_perm(t2, t0, 0x05040100u);
        o[1][b] = __builtin_amdgcn_perm(t2, t0, 0x07060302u);
        o[2][b] = __builtin_amdgcn_perm(t3, t1, 0x05040100u);
        o[3][b] = __builtin_amdgcn_perm(t3, t1, 0x07060302u);
      }
      const int kb = kg >> 2, half = (kg >> 1) & 1, part = kg & 1, blk = q >> 3;
      const int c0 = ((kb * 4 + blk) * 2 + part) * 64 + half * 32 + (q & 7) * 4;
      #pragma unroll
      for (int c = 0; c < 4; ++c) {
        const int ch = c0 + c;
        const v4u ov = {o[c][0], o[c][1], o[c][2], o[c][3]};
        *(v4u*)(smem + (ch ^ ((ch >> 3) & 3)) * 16) = ov;
      }
    }
    __syncthreads();
    call = sm_word[0];
    const u32 dof = OFF_WT + (((call & 1u) * NR + (u32)j) * NT + (u32)t) * ITEM;
    #pragma unroll
    for (int i = 0; i < 16; ++i) {
      const int ch = i * 256 + tid;
      const v4u ov = *(const v4u*)(smem + (ch ^ ((ch >> 3) & 3)) * 16);
      __builtin_amdgcn_raw_buffer_store_b128(ov, rs_my, (int)(dof + (u32)ch * 16u), 0, AUX_SYS);
    }
    WAIT_VM(0);
    __syncthreads();
    if (tid == 0 && j < P.nranks) {
      if (j == me) __hip_atomic_fetch_add((u32*)(my + OFF_CNTW + t * 128), 4u, __ATOMIC_RELAXED, __HIP_MEMORY_SCOPE_AGENT);
      else __hip_atomic_fetch_add((u32*)(P.ws[j] + OFF_INFLAG + me * 128 + 4), 1u, __ATOMIC_RELAXED, __HIP_MEMORY_SCOPE_SYSTEM);
      __hip_atomic_fetch_add((u32*)(my + OFF_PDONE), 1u, __ATOMIC_RELAXED, __HIP_MEMORY_SCOPE_AGENT);
    }
  } else if (g < NPREP + 7 * NPUSH) {
    const int p  = (me + 1 + (g - NPREP) % 7) & (NR - 1), pw = (g - NPREP) / 7;
    const int pm = (me + 1 + pw) & (NR - 1);
    const unsigned char* src = P.x + (size_t)(pm * MR + wave * 32 + (lane & 31)) * KS + 32 * (lane >> 5);
    v4u v[16];
    #pragma unroll
    for (int j = 0; j < 16; ++j) v[j] = *(const v4u*)(src + (j >> 1) * 64 + (j & 1) * 16);
    __syncthreads();
    call = sm_word[0];
    if (p < P.nranks) {
      const __amdgpu_buffer_rsrc_t rs_rp = mk_rsrc(P.ws[p], WS_USED);
      const u32 dof = OFF_XP + (((call & 1u) * NR + (u32)me) * NR + (u32)pw) * ITEM + (u32)(wave * 2 * 64 + lane) * 16u;
      #pragma unroll
      for (int j = 0; j < 16; ++j)
        __builtin_amdgcn_raw_buffer_store_b128(v[j], rs_rp, (int)(dof + (u32)((j >> 1) * 8192 + (j & 1) * 1024)), 0, AUX_SYS);
      WAIT_VM(0);
      if (lane == 0) __hip_atomic_fetch_add((u32*)(P.ws[p] + OFF_CNTP + ((call & 1u) * NR + (u32)pm) * 128u), 1u, __ATOMIC_RELAXED, __HIP_MEMORY_SCOPE_SYSTEM);
      USTAMP((g - NPREP) % 7, 1000 + pw * 4 + wave);
    }
  } else {
    __syncthreads();
    call = sm_word[0];
  }
  STAMP(1);
  const u32 par = call & 1u;
  const float scale = P.sx[0] * P.sw[0];
  const int wm = wave & 1, wn = wave >> 1;

  const int grp = bid & 7;
  const int gfirst = NPW + ((grp - NPW) & 7);
  const int gcnt = (GRID - 1 - gfirst) / 8 + 1;
  int T = (bid - gfirst) >> 3;
  bool pdry = false;
  for (;;) {
    __syncthreads();
    if (tid == 0) {
      const bool have = T < NTILE / 8;
      const u32 ms = (u32)T >> 4, mt = (u32)((grp >> 1) * 4 + ((T >> 2) & 3)), mm = (u32)((grp & 1) * 4 + (T & 3));
      const u32 np = ms != 0u ? 0u : (u32)NPUSH - (((mm - (u32)me - 1u) & (NR - 1)) < (u32)NPUSH ? 1u : 0u);
      const u32 tw = CNT_PER_CALL * call, tx = 4u * ((mm == (u32)me ? NR : NR - 1) - np) * call, tp = 4u * np * ((call + 1u) >> 1);
      const unsigned char* const cw = my + OFF_CNTW + mt * 128u;
      const unsigned char* const cx = my + OFF_CNTX + ((ms & 3u) * 8u + mm) * 128u;
      const unsigned char* const cp = my + OFF_CNTP + (par * NR + mm) * 128u;
      bool ready = false, ok = true;
      if (have) {
        const u32 a = ld_sys(cw), b = ld_sys(cx), c = ld_sys(cp);
        ready = (int)(a - tw) >= 0 && (int)(b - tx) >= 0 && (int)(c - tp) >= 0;
      }
      u32 job = 0xffffffffu;
      if (!ready && !pdry) {
        job = __hip_atomic_fetch_add((u32*)(my + OFF_PTK), 1u, __ATOMIC_RELAXED, __HIP_MEMORY_SCOPE_AGENT) - (call - 1u) * (u32)(NPT + NGW);
        pdry = job >= (u32)NPT;
      }
      if (job < (u32)NPT) {
        ok = wait_ge(my + OFF_PDONE, (u32)NPREP * call, tmo, call);
      } else if (have) {
        job = 0xfffffffeu;
        if (!ready) {
          ok = wait_ge(cw, tw, tmo, call);
          if (ok) ok = wait_ge(cx, tx, tmo, call);
          if (ok) ok = wait_ge(cp, tp, tmo, call);
        }
      } else {
        job = 0xffffffffu;
      }
      if (job == 0xfffffffeu && ms == 0u) __builtin_amdgcn_fence(__ATOMIC_ACQUIRE, "");
      else                                __builtin_amdgcn_fence(__ATOMIC_ACQUIRE, "agent");
      WAIT_VM(0);
      if (ok) ok = (ld_agent(tmo) != call);
      sm_word[1] = ok ? 1u : 0u;
      sm_word[2] = job;
    }
    __syncthreads();
    const u32 job = __builtin_amdgcn_readfirstlane(sm_word[2]);
    if (job == 0xffffffffu) break;
    const bool part = job < (u32)NPT;
    int s, t, m, j;
    if (part) { const int r = (int)job % (NPT / NSL); s = (int)job / (NPT / NSL); j = (me + 1 + (r >> 4)) & (NR - 1); t = r & 15; m = me; }
    else      { s = T >> 4; t = (grp >> 1) * 4 + ((T >> 2) & 3); m = (grp & 1) * 4 + (T & 3); j = me; }
    const int xi = s * 8 + m;
    const bool ok = sm_word[1] != 0u;
    if (!part) STAMP(2 + 3 * (T / gcnt));

    v16f acc[2][2];
    #pragma unroll
    for (int i = 0; i < 2; ++i)
      #pragma unroll
      for (int j = 0; j < 2; ++j)
        #pragma unroll
        for (int e = 0; e < 16; ++e) acc[i][j][e] = 0.f;

    if (ok) {
      const unsigned char* xo = my + OFF_XT + (par * NXI + (u32)xi) * ITEM + tid * 16;
      const unsigned char* xl = my + OFF_XL + (u32)xi * ITEM + tid * 16;
      const unsigned char* xp = my + OFF_XP + par * NR * NR * ITEM + tid * 16;
      const unsigned char* wo = my + OFF_WT + ((par * NR + (u32)j) * NT + (u32)t) * ITEM + tid * 16;
      const int sr0 = part ? me : 0, skip = m == me ? NR : m;
      const int nst = part ? 8 : (m == me ? 64 : 56);
      const unsigned char* wl = my + OFF_WL + (u32)t * ITEM + tid * 16;
      unsigned char* const ldst = smem + wave * 1024;
      auto issue = [&](int ks) {
        int sr = sr0 + (ks >> 3); sr += sr >= skip ? 1 : 0;
        const int kb = ks & 7;
        const u32 d = (u32)(m - sr - 1) & (NR - 1);
        const unsigned char* xa = (sr == me ? xo : (s == 0 && d < (u32)NPUSH) ? xp + (size_t)(((u32)sr * NR + d) * ITEM)
                                                                            : xl + (size_t)sr * (NXI * ITEM)) + kb * 8192;
        const unsigned char* wa = (sr == me ? wo : wl + (size_t)sr * (NT * ITEM)) + kb * 8192;
        unsigned char* l = ldst + (ks & 3) * 16384;
        __builtin_amdgcn_global_load_lds((const __attribute__((address_space(1))) void*)(xa),        (__attribute__((address_space(3))) void*)(l),         16, 0, 0);
        __builtin_amdgcn_global_load_lds((const __attribute__((address_space(1))) void*)(xa + 4096), (__attribute__((address_space(3))) void*)(l + 4096),  16, 0, 0);
        __builtin_amdgcn_global_load_lds((const __attribute__((address_space(1))) void*)(wa),        (__attribute__((address_space(3))) void*)(l + 8192),  16, 0, 0);
        __builtin_amdgcn_global_load_lds((const __attribute__((address_space(1))) void*)(wa + 4096), (__attribute__((address_space(3))) void*)(l + 12288), 16, 0, 0);
      };
      const unsigned char* const fx = smem + (wm * 2) * 2048 + lane * 16;
      const unsigned char* const fw = smem + 8192 + (wn * 2) * 2048 + lane * 16;
      auto rdfrag = [&](int ks, v8i (&X)[2], v8i (&W)[2]) {
        const int so = (ks & 3) * 16384;
        #pragma unroll
        for (int b = 0; b < 2; ++b) {
          const v4u x0 = *(const v4u*)(fx + so + b * 2048), x1 = *(const v4u*)(fx + so + b * 2048 + 1024);
          const v4u w0 = *(const v4u*)(fw + so + b * 2048), w1 = *(const v4u*)(fw + so + b * 2048 + 1024);
          X[b] = (v8i){(int)x0[0], (int)x0[1], (int)x0[2], (int)x0[3], (int)x1[0], (int)x1[1], (int)x1[2], (int)x1[3]};
          W[b] = (v8i){(int)w0[0], (int)w0[1], (int)w0[2], (int)w0[3], (int)w1[0], (int)w1[1], (int)w1[2], (int)w1[3]};
        }
      };
      auto mma = [&](v8i (&X)[2], v8i (&W)[2]) {
        #pragma unroll
        for (int i = 0; i < 2; ++i)
          #pragma unroll
          for (int j = 0; j < 2; ++j)
            acc[i][j] = __builtin_amdgcn_mfma_scale_f32_32x32x64_f8f6f4(W[i], X[j], acc[i][j], 1, 0, 0, 0x7F7F7F7F, 0, 0x7F7F7F7F);
      };
      v8i XA[2], WA[2], XB[2], WB[2];
      issue(0); issue(1); issue(2);
      WAIT_VM(8); RAW_BAR;
      issue(3);
      rdfrag(0, XA, WA);
      for (int ks = 0; ks < nst - 4; ks += 2) {
        WAIT_VM(8); WAIT_LGKM0; RAW_BAR;
        issue(ks + 4);
        rdfrag(ks + 1, XB, WB);
        SCHED_FENCE;
        mma(XA, WA);
        SCHED_FENCE;
        WAIT_VM(8); WAIT_LGKM0; RAW_BAR;
        issue(ks + 5);
        rdfrag(ks + 2, XA, WA);
        SCHED_FENCE;
        mma(XB, WB);
        SCHED_FENCE;
      }
      WAIT_VM(8); WAIT_LGKM0; RAW_BAR;
      rdfrag(61, XB, WB); SCHED_FENCE; mma(XA, WA); SCHED_FENCE;
      WAIT_VM(4); WAIT_LGKM0; RAW_BAR;
      rdfrag(62, XA, WA); SCHED_FENCE; mma(XB, WB); SCHED_FENCE;
      WAIT_VM(0); WAIT_LGKM0; RAW_BAR;
      rdfrag(63, XB, WB); SCHED_FENCE; mma(XA, WA); SCHED_FENCE;
      mma(XB, WB);
    }
    if (!part) STAMP(3 + 3 * (T / gcnt));
    __syncthreads();
    {
      float* img = (float*)smem;
      const float nanf_ = __builtin_nanf("");
      #pragma unroll
      for (int i = 0; i < 2; ++i)
        #pragma unroll
        for (int j = 0; j < 2; ++j)
          #pragma unroll
          for (int q = 0; q < 4; ++q) {
            v4f o;
            #pragma unroll
            for (int e = 0; e < 4; ++e) o[e] = ok ? acc[i][j][4 * q + e] * scale : nanf_;
            const int row = wm * 64 + j * 32 + (lane & 31);
            const int col = wn * 64 + i * 32 + 8 * q + 4 * (lane >> 5);
            *(v4f*)(img + row * EPI_LD + col) = o;
          }
      __syncthreads();
      const int r0 = wave * 2 + (lane >> 5), c4 = (lane & 31) * 4;
      if (part) {
        const __amdgpu_buffer_rsrc_t rs_out = mk_rsrc(P.out, (u32)MR * NN * 4u);
        const u32 dof = ((u32)(s * TM + r0) * (u32)NN + (u32)(j * NC + t * TN + c4)) * 4u;
        #pragma unroll
        for (int i = 0; i < 16; ++i) {
          const v4f o = *(const v4f*)(img + (r0 + i * 8) * EPI_LD + c4);
          __builtin_amdgcn_raw_buffer_store_b128(__builtin_bit_cast(v4u, o), rs_out, (int)(dof + (u32)(i * 8) * (u32)(NN * 4)), 0, AUX_PST);
        }
        WAIT_VM(0);
        __syncthreads();
        if (tid == 0) __hip_atomic_fetch_add((u32*)(my + OFF_PCNT + s * 128), 1u, __ATOMIC_RELAXED, __HIP_MEMORY_SCOPE_AGENT);
      } else if (m == me) {
        float* dst = P.out + (size_t)(s * TM + r0) * NN + me * NC + t * TN + c4;
        #pragma unroll
        for (int i = 0; i < 16; ++i) {
          const v4f o = *(const v4f*)(img + (r0 + i * 8) * EPI_LD + c4);
          *(v4f*)(dst + (size_t)(i * 8) * NN) = o;
        }
      } else {
        const u32 dof = OFF_OUT + ((par * NR + (u32)m) * 64u + (u32)(s * NT + t)) * ITEM + (u32)(r0 * 512 + c4 * 4);
        #pragma unroll
        for (int i = 0; i < 16; ++i) {
          const v4f o = *(const v4f*)(img + (r0 + i * 8) * EPI_LD + c4);
          __builtin_amdgcn_raw_buffer_store_b128(__builtin_bit_cast(v4u, o), rs_my, (int)(dof + (u32)(i * 8 * 512)), 0, AUX_SYS);
        }
        WAIT_VM(0);
        __syncthreads();
        if (tid == 0 && m < P.nranks)
          __hip_atomic_store((u32*)(P.ws[m] + OFF_OFLAG + (u32)(me * 64 + s * NT + t) * 128u), call, __ATOMIC_RELAXED, __HIP_MEMORY_SCOPE_SYSTEM);
      }
    }
    if (part) { STAMP(13); }
    else      { STAMP(4 + 3 * (T / gcnt)); T += gcnt; }
  }
}

extern "C" void kernel_launch(int rank, int nranks, ncclComm_t comm,
                              hipStream_t stream,
                              void* const* d_in, const int* in_sizes, int n_in,
                              void* d_out, int out_size,
                              const int* in_dtypes, int out_dtype,
                              void* const* d_out_all,
                              void* const* d_ws_all, size_t ws_bytes) {
  (void)comm; (void)in_sizes; (void)in_dtypes; (void)out_dtype; (void)d_out_all;
  if (nranks != NR || n_in < 4 || ws_bytes < (size_t)WS_USED || out_size != MR * NN || rank < 0 || rank >= NR) {
    fprintf(stderr, "unexpected configuration\n");
    return;
  }
  Params P;
  P.x  = (const unsigned char*)d_in[0];
  P.w  = (const unsigned char*)d_in[1];
  P.sx = (const float*)d_in[2];
  P.sw = (const float*)d_in[3];
  P.out = (float*)d_out;
  for (int p = 0; p < NR; ++p) P.ws[p] = (unsigned char*)d_ws_all[p];
  P.rank = rank;
  P.nranks = nranks;
  fused_gemm_rs<<<GRID, TPB, 0, stream>>>(P);
  HIP_CHECK(hipGetLastError());
}
```

```cpp
#include <hip/hip_runtime.h>
#include <rccl/rccl.h>
#include <cstdio>
#include <cstdint>
#include <type_traits>

#define HIP_CHECK(expr) do { hipError_t _e = (expr); if (_e != hipSuccess) { \
    fprintf(stderr, "HIP error %s at %s:%d\n", hipGetErrorString(_e), __FILE__, __LINE__); \
    return; } } while (0)
#define NCCL_CHECK(expr) do { ncclResult_t _r = (expr); if (_r != ncclSuccess) { \
    fprintf(stderr, "RCCL error %s at %s:%d\n", ncclGetErrorString(_r), __FILE__, __LINE__); \
    return; } } while (0)

typedef unsigned u32;
typedef unsigned long long u64;
typedef int      v8i  __attribute__((ext_vector_type(8)));
typedef unsigned v4u  __attribute__((ext_vector_type(4)));
typedef unsigned v3u  __attribute__((ext_vector_type(3)));
typedef unsigned v2u  __attribute__((ext_vector_type(2)));
typedef float    v4f  __attribute__((ext_vector_type(4)));
typedef float    v16f __attribute__((ext_vector_type(16)));

constexpr int NR    = 8;
constexpr int MM    = 4096;
constexpr int KK    = 4096;
constexpr int NN    = 16384;
constexpr int KS    = KK / NR;
constexpr int NC    = NN / NR;
constexpr int MR    = MM / NR;
constexpr int TM    = 128;
constexpr int TN    = 128;
constexpr int NSL   = MR / TM;
constexpr int NT    = NC / TN;
constexpr int NXI   = NSL * NR;
constexpr int NTILE = NXI * NT;
constexpr u32 ITEM  = 65536;
constexpr u32 UNIT  = 16384;

constexpr int GRID  = 256;
constexpr int TPB   = 256;
#define PWG 6
constexpr int NPW   = 7 * PWG;
constexpr int NGW   = GRID - NPW;
constexpr int GW0   = NGW - NR * NT;

constexpr u32 OFF_TICKET = 0;
constexpr u32 OFF_TMO    = 2048;
constexpr u32 OFF_UCTR   = 3072;
constexpr u32 OFF_INFLAG = 4096;
constexpr u32 OFF_CNTX   = 8192;
constexpr u32 OFF_CNTW   = 16384;
constexpr u32 OFF_CNTP   = 24576;
constexpr u32 OFF_PCNT   = 28672;
constexpr u32 OFF_OFLAG  = 65536;
constexpr u32 OFF_XT     = 1u << 20;
constexpr u32 OFF_WT     = OFF_XT + 2u * NXI * ITEM;
constexpr u32 OFF_XL     = OFF_WT + 2u * NR * NT * ITEM;
constexpr u32 OFF_WL     = OFF_XL + (u32)NR * NXI * ITEM;
constexpr u32 OFF_OUT    = OFF_WL + (u32)NR * NT * ITEM;
#define NPUSH 4
#define TAILT 4
constexpr u32 OFF_XP     = OFF_OUT + 2u * NR * (NSL * NT) * ITEM;
constexpr u32 RBW        = 3072 + 128;
constexpr u32 RBX        = 3584 + 128;
constexpr u32 OFF_WC     = OFF_XP + 2u * NR * NR * ITEM;
constexpr u32 OFF_XC     = OFF_WC + 2u * NR * NT * 16u * RBW;
constexpr u32 WS_USED    = OFF_XC + 2u * NXI * 16u * RBX;
static_assert(NPUSH <= NR - 1 && NXI + 7 * NPUSH <= GW0, "pushed x items: row blocks of slice 0, one per otherwise idle workgroup");

constexpr u32 CNT_PER_CALL = NR * 4;
constexpr u64 DEADLINE_TICKS = 100000000ull;

constexpr int EPI_LD   = TN + 4;
constexpr int SM_BYTES = TM * EPI_LD * 4;
constexpr int SM_W     = SM_BYTES + 2048;
constexpr int SM_TOTAL = SM_W + 65536;

constexpr int AUX_SYS   = 17;
constexpr int AUX_AGENT = 16;
#define AUX_OUT 16
#define AUX_PLD 16
#define AUX_PST 16
constexpr int NPC   = (NR - 1) * NT;
constexpr int NUX1  = (2 * (NR - 1) - NPUSH) * 4;
#define NUA 20
static_assert(NUA % 4 == 0 && NUA <= NUX1 && NUA >= (NR - 1 - NPUSH) * 4, "x units pulled ahead of w");
constexpr int NUW   = NUA + NT * 4;
constexpr int NUIN  = (NT + NXI - NSL - NPUSH) * 4;
constexpr int NUBIG = NUIN + (NSL * NT - TAILT) * 4;
constexpr int NUNIT = NUBIG + TAILT * 16;
static_assert(NPW >= NUX1, "the first x units are packed by the pullers, one unit per workgroup");

__device__ __forceinline__ u32 x_item(u32 z, u32 p) {
  if (z < (u32)(NR - 1 - NPUSH)) return (p + 1u + NPUSH + z) & (NR - 1);
  const u32 y = z - (u32)(NR - 1 - NPUSH), i = y % 7u;
  return (1u + y / 7u) * 8u + i + (i >= p ? 1u : 0u);
}

#define STAMP(slot) do {} while (0)
#define USTAMP(g, u) do {} while (0)

struct Params {
  const unsigned char* x;
  const unsigned char* w;
  const float* sx;
  const float* sw;
  float* out;
  unsigned char* ws[NR];
  int rank;
  int nranks;
};

#define WAIT_VM(n)  asm volatile("s_waitcnt vmcnt(" #n ")" ::: "memory")
#define WAIT_VM4    do { __builtin_amdgcn_s_waitcnt(0x0F74); asm volatile("" ::: "memory"); } while (0)
#define WAIT_VM0    do { __builtin_amdgcn_s_waitcnt(0x0F70); asm volatile("" ::: "memory"); } while (0)
#define WAIT_LGKM0  do { __builtin_amdgcn_s_waitcnt(0xC07F); asm volatile("" ::: "memory"); } while (0)
#define RAW_BAR     __builtin_amdgcn_s_barrier()
#define SCHED_FENCE __builtin_amdgcn_sched_barrier(0)

__device__ __forceinline__ u32 ld_sys(const unsigned char* p) {
  return __hip_atomic_load((const u32*)p, __ATOMIC_RELAXED, __HIP_MEMORY_SCOPE_SYSTEM);
}
__device__ __forceinline__ u32 ld_sys1(const unsigned char* p) {
  return __builtin_amdgcn_readfirstlane(__hip_atomic_load((const u32*)p, __ATOMIC_RELAXED, __HIP_MEMORY_SCOPE_SYSTEM));
}
__device__ __forceinline__ u32 ld_agent(const unsigned char* p) {
  return __builtin_amdgcn_readfirstlane(__hip_atomic_load((const u32*)p, __ATOMIC_RELAXED, __HIP_MEMORY_SCOPE_AGENT));
}

__device__ __forceinline__ bool wait_ge(const unsigned char* flag, u32 target, unsigned char* tmo, u32 call) {
  if ((int)(ld_sys1(flag) - target) >= 0) return true;
  const u64 t0 = __builtin_amdgcn_s_memrealtime();
  for (;;) {
    if ((int)(ld_sys1(flag) - target) >= 0) return true;
    if (ld_agent(tmo) == call || __builtin_amdgcn_s_memrealtime() - t0 > DEADLINE_TICKS) {
      __hip_atomic_store((u32*)tmo, call, __ATOMIC_RELAXED, __HIP_MEMORY_SCOPE_AGENT);
      return false;
    }
    __builtin_amdgcn_s_sleep(2);
  }
}

__device__ __forceinline__ bool wait_flags(const unsigned char* base, int n, int lane, u32 target, unsigned char* tmo, u32 call) {
  const u64 t0 = __builtin_amdgcn_s_memrealtime();
  for (;;) {
    const u32 f = lane < n ? ld_sys(base + lane * 128) : target;
    if (__builtin_amdgcn_ballot_w64((int)(f - target) < 0) == 0ull) return true;
    if (ld_agent(tmo) == call || __builtin_amdgcn_s_memrealtime() - t0 > DEADLINE_TICKS) {
      __hip_atomic_store((u32*)tmo, call, __ATOMIC_RELAXED, __HIP_MEMORY_SCOPE_AGENT);
      return false;
    }
    __builtin_amdgcn_s_sleep(2);
  }
}

__device__ __forceinline__ __amdgpu_buffer_rsrc_t mk_rsrc(const void* p, u32 bytes) {
  return __builtin_amdgcn_make_buffer_rsrc((void*)p, (short)0, (int)bytes, 0x00020000);
}

typedef __attribute__((address_space(3))) unsigned char lds_u8;
typedef __attribute__((address_space(3))) u32 lds_u32;
typedef __attribute__((address_space(3))) v4u lds_v4u;

__device__ __forceinline__ u32 lane_prefix(u32 n) {
  u32 off = 0;
  #pragma unroll
  for (int b = 0; b < 5; ++b) {
    const u64 m = __builtin_amdgcn_ballot_w64(((n >> b) & 1u) != 0u);
    off += __builtin_amdgcn_mbcnt_hi((u32)(m >> 32), __builtin_amdgcn_mbcnt_lo((u32)m, 0u)) << b;
  }
  return off;
}
__device__ __forceinline__ u32 flag_mask(const u32* e) {
  u32 t[4];
  #pragma unroll
  for (int k = 0; k < 4; ++k) t[k] = __builtin_amdgcn_udot4(e[2 * k + 1], 0x80402010u, __builtin_amdgcn_udot4(e[2 * k], 0x08040201u, 0u, false), false);
  return (t[0] >> 7) | (t[1] << 1) | (t[2] << 9) | (t[3] << 17);
}

template <bool W, int S, u32 H>
__device__ __forceinline__ bool pack_block(const v4u* v, const lds_u8* raw, u32 (&pk)[14], lds_u32* area, int lane, u32& escw) {
  u32 c[16], e[16];
  #pragma unroll
  for (int i = 0; i < 16; ++i) {
    const u32 d = v[i >> 2][i & 3];
    const u32 y = (((d & (W ? 0x7C7C7C7Cu : 0x78787878u)) | 0x80808080u) - (W ? 0x28282828u : 0x10101010u)) & 0x7F7F7F7Fu;
    e[i] = (y + (W ? 0x64646464u : 0x48484848u)) & 0x80808080u;
    c[i] = W ? (d & 0x83838383u) | ((y | ((e[i] >> 2) - (e[i] >> 5))) & 0x1C1C1C1Cu)
             : (d & 0x87878787u) | ((y | ((e[i] >> 1) - (e[i] >> 4))) & 0x38383838u);
  }
  if (W) {
    #pragma unroll
    for (int k = 0; k < 4; ++k) {
      const u32 z = c[4 * k + 3];
      pk[3 * k]     = c[4 * k]     | ((z << 5) & 0x60606060u);
      pk[3 * k + 1] = c[4 * k + 1] | ((z << 3) & 0x60606060u);
      pk[3 * k + 2] = c[4 * k + 2] | ((z << 1) & 0x20202020u) | ((z >> 1) & 0x40404040u);
    }
    pk[12] = 0; pk[13] = 0;
  } else {
    #pragma unroll
    for (int k = 0; k < 2; ++k) {
      const u32 z = c[8 * k + 7];
      #pragma unroll
      for (int i = 0; i < 6; ++i) pk[7 * k + i] = c[8 * k + i] | ((z << (6 - i)) & 0x40404040u);
      pk[7 * k + 6] = c[8 * k + 6] | ((z >> 1) & 0x40404040u);
    }
  }
  u32 m0 = flag_mask(e), m1 = flag_mask(e + 8);
  const u32 n0 = (u32)__builtin_popcount(m0), n1 = (u32)__builtin_popcount(m1);
  u32 a0 = 0, a1 = 0, top = 0;
  bool rawb = __builtin_amdgcn_ballot_w64(n0 > 8u || n1 > 8u) != 0ull;
  if (!rawb) {
    for (u32 sh = 0; __builtin_amdgcn_ballot_w64((m0 | m1) != 0u) != 0ull; sh += 4u) {
      const u32 p0 = (u32)__builtin_ctz(m0 | 0x80000000u), p1 = (u32)__builtin_ctz(m1 | 0x80000000u);
      const u32 b0 = raw[((p0 & 16u) << S) + (p0 & 15u)], b1 = raw[H + ((p1 & 16u) << S) + (p1 & 15u)];
      u32 f0 = W ? (b0 >> 2) & 31u : (b0 >> 3) & 15u, f1 = W ? (b1 >> 2) & 31u : (b1 >> 3) & 15u;
      if (W) { f0 = f0 < 10u ? f0 : f0 - 7u; f1 = f1 < 10u ? f1 : f1 - 7u; }
      f0 = m0 != 0u ? f0 : 0u; f1 = m1 != 0u ? f1 : 0u;
      top |= f0 | f1;
      a0 |= f0 << sh; a1 |= f1 << sh;
      m0 &= m0 - 1u; m1 &= m1 - 1u;
    }
  }
  const u32 n = rawb ? 0u : n0 + n1;
  const u32 off = lane_prefix(n);
  rawb = rawb || __builtin_amdgcn_ballot_w64(top > 15u) != 0ull || __builtin_amdgcn_readlane(off + n, 63) > 254u;
  if (!rawb && n != 0u) {
    const u64 a = (u64)a0 | ((u64)a1 << (4u * n0));
    const u32 sh = (off & 7u) * 4u, l = (u32)a, h = (u32)(a >> 32);
    lds_u32* q = area + (off >> 3);
    __hip_atomic_fetch_or(q,     l << sh, __ATOMIC_RELAXED, __HIP_MEMORY_SCOPE_WORKGROUP);
    __hip_atomic_fetch_or(q + 1, (u32)(a >> (32u - sh)), __ATOMIC_RELAXED, __HIP_MEMORY_SCOPE_WORKGROUP);
    __hip_atomic_fetch_or(q + 2, (u32)((u64)h >> (32u - sh)), __ATOMIC_RELAXED, __HIP_MEMORY_SCOPE_WORKGROUP);
  }
  asm volatile("" ::: "memory");
  escw = __hip_atomic_load(area + (lane & 31), __ATOMIC_RELAXED, __HIP_MEMORY_SCOPE_WORKGROUP);
  if (rawb && (lane & 31) == 31) escw = 0xFF000000u;
  return rawb;
}

template <bool W>
__device__ __forceinline__ void unpack_block(const u32 (&pk)[14], u32 escw, lds_u8* st, v4u (&o)[4]) {
  u32 c[16], e[16];
  if (W) {
    #pragma unroll
    for (int k = 0; k < 4; ++k) {
      c[4 * k] = pk[3 * k]; c[4 * k + 1] = pk[3 * k + 1]; c[4 * k + 2] = pk[3 * k + 2];
      c[4 * k + 3] = ((pk[3 * k] >> 5) & 0x03030303u) | ((pk[3 * k + 1] >> 3) & 0x0C0C0C0Cu) |
                     ((pk[3 * k + 2] >> 1) & 0x10101010u) | ((pk[3 * k + 2] << 1) & 0x80808080u);
    }
  } else {
    #pragma unroll
    for (int k = 0; k < 2; ++k) {
      u32 t = (pk[7 * k + 6] << 1) & 0x80808080u;
      #pragma unroll
      for (int i = 0; i < 6; ++i) t |= (pk[7 * k + i] >> (6 - i)) & (0x01010101u << i);
      #pragma unroll
      for (int i = 0; i < 7; ++i) c[8 * k + i] = pk[7 * k + i];
      c[8 * k + 7] = t;
    }
  }
  #pragma unroll
  for (int i = 0; i < 16; ++i) {
    e[i] = W ? ((c[i] & 0x1F1F1F1Fu) + 0x64646464u) & 0x80808080u : ((c[i] & 0x3F3F3F3Fu) + 0x48484848u) & 0x80808080u;
    c[i] = W ? (c[i] & 0x9F9F9F9Fu) + 0x28282828u : (c[i] & 0xBFBFBFBFu) + 0x10101010u;
  }
  #pragma unroll
  for (int k = 0; k < 4; ++k) *(lds_v4u*)(st + k * 16) = (v4u){c[4 * k], c[4 * k + 1], c[4 * k + 2], c[4 * k + 3]};
  u32 m0 = flag_mask(e), m1 = flag_mask(e + 8);
  const u32 n0 = (u32)__builtin_popcount(m0);
  const u32 off = lane_prefix(n0 + (u32)__builtin_popcount(m1));
  const int ix = (int)(off >> 3) * 4;
  const u32 w0 = (u32)__builtin_amdgcn_ds_bpermute(ix, (int)escw), w1 = (u32)__builtin_amdgcn_ds_bpermute(ix + 4, (int)escw),
            w2 = (u32)__builtin_amdgcn_ds_bpermute(ix + 8, (int)escw);
  const u32 s0 = (off & 7u) * 4u;
  const u32 a0 = __builtin_amdgcn_alignbit(w1, w0, s0);
  const u32 a1 = (u32)((((u64)__builtin_amdgcn_alignbit(w2, w1, s0) << 32) | a0) >> (4u * n0));
  asm volatile("" ::: "memory");
  for (u32 sh = 0; __builtin_amdgcn_ballot_w64((m0 | m1) != 0u) != 0ull; sh += 4u) {
    const u32 p0 = (u32)__builtin_ctz(m0 | 0x80000000u), p1 = (u32)__builtin_ctz(m1 | 0x80000000u);
    u32 f0 = (a0 >> sh) & 15u, f1 = (a1 >> sh) & 15u;
    if (W) { f0 += ((f0 + 6u) >> 4) * 7u; f1 += ((f1 + 6u) >> 4) * 7u; }
    f0 = m0 != 0u ? f0 - (W ? 17u : 9u) : 0u; f1 = m1 != 0u ? f1 - (W ? 17u : 9u) : 0u;
    __hip_atomic_fetch_add((lds_u32*)(st + (p0 & 28u)),       f0 << ((p0 * 8u + (W ? 2u : 3u)) & 31u), __ATOMIC_RELAXED, __HIP_MEMORY_SCOPE_WORKGROUP);
    __hip_atomic_fetch_add((lds_u32*)(st + 32u + (p1 & 28u)), f1 << ((p1 * 8u + (W ? 2u : 3u)) & 31u), __ATOMIC_RELAXED, __HIP_MEMORY_SCOPE_WORKGROUP);
    m0 &= m0 - 1u; m1 &= m1 - 1u;
  }
  asm volatile("" ::: "memory");
  #pragma unroll
  for (int k = 0; k < 4; ++k) o[k] = *(volatile lds_v4u*)(st + k * 16);
}

__global__ void __launch_bounds__(TPB, 1) fused_gemm_rs(Params P) {
  __shared__ __attribute__((aligned(16))) unsigned char smem[SM_TOTAL];
  const int tid  = threadIdx.x;
  const int lane = tid & 63;
  const int wave = tid >> 6;
  const int me   = P.rank;
  const int bid  = blockIdx.x;
  unsigned char* const my = P.ws[me];
  unsigned char* const tmo = my + OFF_TMO;
  volatile u32* const sm_word = (volatile u32*)(smem + SM_BYTES);
  const __amdgpu_buffer_rsrc_t rs_my = mk_rsrc(my, WS_USED);

  STAMP(0);
  if (tid == 0) {
    const u32 ticket = __hip_atomic_fetch_add((u32*)(my + OFF_TICKET + (bid & 15) * 128), 1u, __ATOMIC_RELAXED, __HIP_MEMORY_SCOPE_AGENT);
    sm_word[0] = ticket / (u32)(GRID / 16) + 1u;
  }

  if (bid < NPW) {
    const int p    = (me + 1 + bid % 7) & (NR - 1);
    const bool enc = bid < NUX1;
    const int ei   = (int)x_item((u32)bid >> 2, (u32)me);
    v4u ev[4];
    if (enc) {
      const unsigned char* src = P.x + (size_t)((ei & 7) * MR + (ei >> 3) * TM + wave * 32 + (lane & 31)) * KS + 32 * (lane >> 5) + (bid & 3) * 128;
      #pragma unroll
      for (int j = 0; j < 4; ++j) ev[j] = *(const v4u*)(src + (j >> 1) * 64 + (j & 1) * 16);
    }
    __syncthreads();
    const u32 call = __builtin_amdgcn_readfirstlane(sm_word[0]);
    const u32 par  = call & 1u;
    if (enc) {
      lds_u32* const area = (lds_u32*)(smem + 65536) + wave * 32;
      lds_u8* const st = (lds_u8*)smem + (u32)tid * 80u;
      area[lane & 31] = 0u;
      #pragma unroll
      for (int j = 0; j < 4; ++j) *(lds_v4u*)(st + j * 16) = ev[j];
      asm volatile("" ::: "memory");
      u32 pk[14], escw;
      pack_block<false, 0, 32u>(ev, st, pk, area, lane, escw);
      const u32 o = OFF_XC + ((par * NXI + (u32)ei) * 16u + (u32)((bid & 3) * 4 + wave)) * RBX;
      #pragma unroll
      for (int k = 0; k < 3; ++k)
        __builtin_amdgcn_raw_buffer_store_b128((v4u){pk[4 * k], pk[4 * k + 1], pk[4 * k + 2], pk[4 * k + 3]}, rs_my, (int)(o + (u32)(k * 1024 + lane * 16)), 0, AUX_SYS);
      __builtin_amdgcn_raw_buffer_store_b64((v2u){pk[12], pk[13]}, rs_my, (int)(o + 3072u + (u32)lane * 8u), 0, AUX_SYS);
      if (lane < 32) __builtin_amdgcn_raw_buffer_store_b32(escw, rs_my, (int)(o + RBX - 128u + (u32)lane * 4u), 0, AUX_SYS);
      WAIT_VM(0);
      __syncthreads();
      if (tid < NR && tid < P.nranks && tid != me)
        __hip_atomic_fetch_add((u32*)(P.ws[tid] + OFF_INFLAG + me * 128), 1u, __ATOMIC_RELAXED, __HIP_MEMORY_SCOPE_SYSTEM);
      STAMP(8);
    }
    if ((unsigned)p >= (unsigned)P.nranks) return;
    const __amdgpu_buffer_rsrc_t rs_rp  = mk_rsrc(P.ws[p], WS_USED);
    const __amdgpu_buffer_rsrc_t rs_out = mk_rsrc(P.out, (u32)MR * NN * 4u);
    const u32 l16 = (u32)lane * 16u;

    u32 zoff = 0;
    asm volatile("" : "+v"(zoff));
    u32* const uctr = (u32*)(my + OFF_UCTR + p * 128 + zoff);
    const u32 ubase = (call - 1u) * (u32)(NUNIT + 4 * PWG);
    const u32 fl_base = OFF_OFLAG + (u32)p * 64u * 128u;
    u32 tk = 0;
    if (lane == 0) tk = __hip_atomic_fetch_add(uctr, 1u, __ATOMIC_RELAXED, __HIP_MEMORY_SCOPE_AGENT);
    bool ok = true;
    u32 u = __builtin_amdgcn_readfirstlane(tk) - ubase;
    unsigned char* pend = nullptr;
    u32 fl = 0;
    bool pf = false;
    u32 psl = 99u;
    u32 iph = 99u;
    const u32 ftl = (u32)(NSL * NT - TAILT) + (u32)(lane < TAILT ? lane : 0);
    u32 pc = 0;
    bool pre = false;
    v4u a0, a1, a2, a3;
    const u32 tlw = lane < (int)((RBW - 3072u) / 16u) ? 3072u + l16 : 0x7FFFFFF0u;
    const u32 tlx = lane < (int)((RBX - 3072u) / 16u) ? 3072u + l16 : 0x7FFFFFF0u;
    lds_u8* const ta = (lds_u8*)smem + 32768 + wave * 1024;
    auto fetch = [&](u32 o, u32 tl) {
      a0 = __builtin_amdgcn_raw_buffer_load_b128(rs_rp, (int)(o + l16), 0, AUX_SYS);
      a1 = __builtin_amdgcn_raw_buffer_load_b128(rs_rp, (int)(o + 1024u + l16), 0, AUX_SYS);
      a2 = __builtin_amdgcn_raw_buffer_load_b128(rs_rp, (int)(o + 2048u + l16), 0, AUX_SYS);
      a3 = __builtin_amdgcn_raw_buffer_load_b128(rs_rp, (int)(o + tl), 0, AUX_SYS);
    };
    auto out_flags = [&](u32 z) {
      pf = z >= (u32)NUBIG;
      fl = __builtin_amdgcn_raw_buffer_load_b32(rs_my, (int)(fl_base + (pf ? ftl : ((z - NUIN) >> 2)) * 128u), 0, AUX_SYS);
      pc = __builtin_amdgcn_raw_buffer_load_b32(rs_my, (int)(OFF_PCNT + (pf ? 1u : (z - NUIN) >> 7) * 128u), 0, AUX_SYS);
    };
    bool flok = false, pcok = false;
    auto eval_flags = [&]() {
      flok = __builtin_amdgcn_ballot_w64((int)(fl - call) < 0) == 0ull;
      pcok = (int)(__builtin_amdgcn_readfirstlane(pc) - (u32)NPC * call) >= 0;
    };
    while (u < (u32)NUBIG) {
      u32 un;
      if (u < (u32)NUIN) {
        u32 ph = u < (u32)NUA ? 0u : u < (u32)NUW ? 1u : 2u;
        if (ok && ph != iph) {
          if (pend != nullptr) {
            WAIT_VM(0);
            if (lane == 0) __hip_atomic_fetch_add((u32*)pend, 1u, __ATOMIC_RELAXED, __HIP_MEMORY_SCOPE_AGENT);
            pend = nullptr;
          }
          ok = wait_ge(my + OFF_INFLAG + p * 128 + ph * 4u, (ph == 0u ? (u32)NUX1 : ph == 1u ? (u32)NT : (u32)NXI) * call, tmo, call);
          if (ok && ph == 2u) ok = wait_ge(my + OFF_INFLAG + p * 128, (u32)NUX1 * call, tmo, call);
          iph = ph;
          if (ph == 0u) STAMP(1);
          if (ph == 1u) STAMP(11);
          if (ph == 2u) STAMP(12);
        }
        u32 so, sc, dof; unsigned char* cnt;
        auto place = [&](u32 z) {
          const u32 q = z & 3u;
          if (ph == 1u) {
            const u32 t = (z - NUA) >> 2;
            so  = OFF_WT + ((par * NR + (u32)me) * NT + t) * ITEM + q * UNIT;
            sc  = OFF_WC + (((par * NR + (u32)me) * NT + t) * 16u + q * 4u) * RBW;
            dof = OFF_WL + ((u32)p * NT + t) * ITEM + q * UNIT;
            cnt = my + OFF_CNTW + t * 128u;
          } else {
            const u32 it = x_item((ph == 2u ? z - 64u : z) >> 2, (u32)p);
            so  = OFF_XT + (par * NXI + it) * ITEM + q * UNIT;
            sc  = OFF_XC + ((par * NXI + it) * 16u + q * 4u) * RBX;
            dof = OFF_XL + ((u32)p * NXI + it) * ITEM + q * UNIT;
            cnt = my + OFF_CNTX + it * 128u;
          }
        };
        place(u);
        if (ok) {
          v3u inf = {0u, 0u, 0u};
          fetch(sc, ph == 1u ? tlw : tlx);
          WAIT_VM0;
          #pragma unroll 1
          for (u32 r = 0;; r = (r + 1u) & 3u) {
            WAIT_VM4;
            const u32 bph = ph;
            *(volatile lds_v4u*)(ta + l16) = a3;
            asm volatile("" ::: "memory");
            const v2u t3 = *(volatile __attribute__((address_space(3))) v2u*)(ta + (l16 >> 1));
            const u32 escw = *(volatile lds_u32*)(ta + (bph == 1u ? RBW - 3200u : RBX - 3200u) + (u32)(lane & 31) * 4u);
            const u32 pk[14] = {a0[0], a0[1], a0[2], a0[3], a1[0], a1[1], a1[2], a1[3], a2[0], a2[1], a2[2], a2[3], t3[0], t3[1]};
            const u32 ro = r * 2048u + l16, bso = so + ro, bdof = dof + ro;
            asm volatile("" : "+v"(a0), "+v"(a1), "+v"(a2), "+v"(a3));
            if (r == 1u) {
              un = __builtin_amdgcn_readfirstlane(tk) - ubase;
              if (pend != nullptr && lane == 0) __hip_atomic_fetch_add((u32*)pend, 1u, __ATOMIC_RELAXED, __HIP_MEMORY_SCOPE_AGENT);
              pend = cnt;
            }
            bool last = false;
            u32 tl = bph == 1u ? tlw : tlx;
            sc += bph == 1u ? RBW : RBX;
            if (r == 3u) {
              if (un < (u32)NUIN) {
                const u32 nph = un < (u32)NUA ? 0u : un < (u32)NUW ? 1u : 2u;
                if (nph != ph) {
                  const u32 i0 = __builtin_amdgcn_readfirstlane(inf[0]), i1 = __builtin_amdgcn_readfirstlane(inf[1]), i2 = __builtin_amdgcn_readfirstlane(inf[2]);
                  last = nph == 1u ? (int)(i1 - (u32)NT * call) < 0 : ((int)(i2 - (u32)NXI * call) < 0 || (int)(i0 - (u32)NUX1 * call) < 0);
                }
                if (!last) {
                  USTAMP(bid % 7, u);
                  u = un; ph = nph; iph = nph; place(u);
                  tl = nph == 1u ? tlw : tlx;
                }
              } else {
                last = true;
                eval_flags();
                pre = un < (u32)NUBIG && flok && pcok;
                sc = OFF_OUT + ((par * NR + (u32)me) * 64u) * ITEM + (un - NUIN) * UNIT;
                tl = 3072u + l16;
              }
            }
            if (!last || pre) fetch(sc, tl);
            if (r == 0u && lane == 0) tk = __hip_atomic_fetch_add(uctr, 1u, __ATOMIC_RELAXED, __HIP_MEMORY_SCOPE_AGENT);
            if (r == 2u) {
              inf = __builtin_amdgcn_raw_buffer_load_b96(rs_my, (int)(OFF_INFLAG + (u32)p * 128u), 0, AUX_SYS);
              out_flags(un < (u32)NUIN ? (u32)NUIN : un);
            }
            v4u o[4];
            if (__builtin_amdgcn_readlane(escw, 31) >> 24) {
              if (bph == 0u && ok) ok = wait_ge(my + OFF_INFLAG + p * 128 + 8u, (u32)NXI * call, tmo, call);
              #pragma unroll
              for (int j = 0; j < 4; ++j)
                o[j] = __builtin_amdgcn_raw_buffer_load_b128(rs_rp, (int)(bso + (u32)((j >> 1) * 8192 + (j & 1) * 1024)), 0, AUX_SYS);
              WAIT_VM0;
            } else if (bph == 1u) {
              unpack_block<true>(pk, escw, (lds_u8*)smem + (u32)tid * 80u, o);
            } else {
              unpack_block<false>(pk, escw, (lds_u8*)smem + (u32)tid * 80u, o);
            }
            #pragma unroll
            for (int j = 0; j < 4; ++j)
              __builtin_amdgcn_raw_buffer_store_b128(o[j], rs_my, (int)(bdof + (u32)((j >> 1) * 8192 + (j & 1) * 1024)), 0, AUX_AGENT);
            if (last) break;
          }
        } else {
          if (lane == 0) tk = __hip_atomic_fetch_add(uctr, 1u, __ATOMIC_RELAXED, __HIP_MEMORY_SCOPE_AGENT);
          un = __builtin_amdgcn_readfirstlane(tk) - ubase;
        }
      } else {
        const u32 tile = (u - NUIN) >> 2, q = u & 3u;
        const u32 s = tile >> 4, t = tile & 15u;
        if (pcok) psl = s >> 1;
        if (ok && (psl != (s >> 1) || !flok)) {
          if (pend != nullptr) {
            WAIT_VM(0);
            if (lane == 0) __hip_atomic_fetch_add((u32*)pend, 1u, __ATOMIC_RELAXED, __HIP_MEMORY_SCOPE_AGENT);
            pend = nullptr;
          }
          if (psl != (s >> 1)) ok = wait_ge(my + OFF_PCNT + (s >> 1) * 128u, (u32)NPC * call, tmo, call);
          psl = s >> 1;
          if (ok && !flok) ok = wait_ge(my + fl_base + tile * 128u, call, tmo, call);
        }
        const u32 so = OFF_OUT + ((par * NR + (u32)me) * 64u + tile) * ITEM + q * UNIT;
        const u32 dof = (s * TM + q * 32u + (u32)(lane >> 5)) * (u32)(NN * 4) + ((u32)(p * NC) + t * TN) * 4u + (u32)(lane & 31) * 16u;
        if (ok) {
          #pragma unroll
          for (int r = 0; r < 4; ++r) {
            v4u a[4];
            if (r != 0) fetch(so + r * 4096, 3072u + l16);
            #pragma unroll
            for (int j = 0; j < 4; ++j)
              a[j] = __builtin_amdgcn_raw_buffer_load_b128(rs_out, (int)(dof + (u32)(r * 8 + j * 2) * (u32)(NN * 4)), 0, AUX_PLD);
            if (r == 0 && !pre) fetch(so, 3072u + l16);
            if (r == 0 && lane == 0) tk = __hip_atomic_fetch_add(uctr, 1u, __ATOMIC_RELAXED, __HIP_MEMORY_SCOPE_AGENT);
            if (r == 1 && un < (u32)NUNIT) out_flags(un);
            WAIT_VM(0);
            const v4u v[4] = {a0, a1, a2, a3};
            if (r == 1) eval_flags();
            if (r == 0) {
              pre = false;
              un = __builtin_amdgcn_readfirstlane(tk) - ubase;
              if (pend != nullptr && lane == 0) __hip_atomic_fetch_add((u32*)pend, 1u, __ATOMIC_RELAXED, __HIP_MEMORY_SCOPE_AGENT);
              pend = nullptr;
            }
            #pragma unroll
            for (int j = 0; j < 4; ++j) {
              const v4f o = __builtin_bit_cast(v4f, v[j]) + __builtin_bit_cast(v4f, a[j]);
              __builtin_amdgcn_raw_buffer_store_b128(__builtin_bit_cast(v4u, o), rs_out, (int)(dof + (u32)(r * 8 + j * 2) * (u32)(NN * 4)), 0, AUX_OUT);
            }
          }
        } else {
          if (lane == 0) tk = __hip_atomic_fetch_add(uctr, 1u, __ATOMIC_RELAXED, __HIP_MEMORY_SCOPE_AGENT);
          un = __builtin_amdgcn_readfirstlane(tk) - ubase;
          const v4u nanv = {0x7fc00000u, 0x7fc00000u, 0x7fc00000u, 0x7fc00000u};
          for (int r = 0; r < 16; ++r)
            __builtin_amdgcn_raw_buffer_store_b128(nanv, rs_out, (int)(dof + (u32)(r * 2) * (u32)(NN * 4)), 0, 0);
        }
      }
      USTAMP(bid % 7, u);
      u = un;
    }
    if (pend != nullptr) {
      WAIT_VM(0);
      if (lane == 0) __hip_atomic_fetch_add((u32*)pend, 1u, __ATOMIC_RELAXED, __HIP_MEMORY_SCOPE_AGENT);
    }
    if (TAILT > 0 && u < (u32)NUNIT) {
      if (ok && !(pf && flok)) ok = wait_flags(my + fl_base + (u32)(NSL * NT - TAILT) * 128u, TAILT, lane, call, tmo, call);
      if (ok && psl != 1u && !(pf && pcok)) ok = wait_ge(my + OFF_PCNT + 128u, (u32)NPC * call, tmo, call);
      while (u < (u32)NUNIT) {
        const u32 tile = (u32)(NSL * NT - TAILT) + ((u - NUBIG) >> 4), sub = (u - NUBIG) & 15u;
        const u32 s = tile >> 4, t = tile & 15u;
        const u32 so = OFF_OUT + ((par * NR + (u32)me) * 64u + tile) * ITEM + sub * 4096u;
        const u32 dof = (s * TM + sub * 8u + (u32)(lane >> 5)) * (u32)(NN * 4) + ((u32)(p * NC) + t * TN) * 4u + (u32)(lane & 31) * 16u;
        v4u v[4], a[4];
        #pragma unroll
        for (int j = 0; j < 4; ++j) {
          if (ok) v[j] = __builtin_amdgcn_raw_buffer_load_b128(rs_rp, (int)(so + j * 1024 + l16), 0, AUX_SYS);
          else    v[j] = (v4u){0x7fc00000u, 0x7fc00000u, 0x7fc00000u, 0x7fc00000u};
        }
        #pragma unroll
        for (int j = 0; j < 4; ++j)
          a[j] = __builtin_amdgcn_raw_buffer_load_b128(rs_out, (int)(dof + (u32)(j * 2) * (u32)(NN * 4)), 0, AUX_PLD);
        if (lane == 0) tk = __hip_atomic_fetch_add(uctr, 1u, __ATOMIC_RELAXED, __HIP_MEMORY_SCOPE_AGENT);
        WAIT_VM(0);
        u = __builtin_amdgcn_readfirstlane(tk) - ubase;
        #pragma unroll
        for (int j = 0; j < 4; ++j) {
          const v4f o = __builtin_bit_cast(v4f, v[j]) + __builtin_bit_cast(v4f, a[j]);
          __builtin_amdgcn_raw_buffer_store_b128(__builtin_bit_cast(v4u, o), rs_out, (int)(dof + (u32)(j * 2) * (u32)(NN * 4)), 0, AUX_OUT);
        }
      }
    }
    STAMP(10);
    return;
  }

  const int g = bid - NPW;
  u32 call;
  const float scale = P.sx[0] * P.sw[0];
  const int wm = wave & 1, wn = wave >> 1;
  float* const img = (float*)smem;
  const int r0 = wave * 2 + (lane >> 5), c4 = (lane & 31) * 4;
  auto put_image = [&](const v16f (&acc)[2][2], bool ok) {
    const float nanf_ = __builtin_nanf("");
    #pragma unroll
    for (int i = 0; i < 2; ++i)
      #pragma unroll
      for (int j = 0; j < 2; ++j)
        #pragma unroll
        for (int q = 0; q < 4; ++q) {
          v4f o;
          #pragma unroll
          for (int e = 0; e < 4; ++e) o[e] = ok ? acc[i][j][4 * q + e] * scale : nanf_;
          const int row = wm * 64 + j * 32 + (lane & 31);
          const int col = wn * 64 + i * 32 + 8 * q + 4 * (lane >> 5);
          *(v4f*)(img + row * EPI_LD + col) = o;
        }
  };
  auto put_packed = [&](auto wtag, const v4u (&v)[16], u32 base, u32 rawbase) {
    constexpr bool W = decltype(wtag)::value;
    constexpr u32 RB = W ? RBW : RBX;
    lds_u32* const area = (lds_u32*)(smem + 65536) + wave * 128;
    const lds_u8* const raw = (const lds_u8*)smem + (W ? SM_W : 0) + (u32)(wave * 128 + (lane ^ ((lane >> 3) & 3))) * 16u;
    area[lane] = 0u; area[64 + lane] = 0u;
    asm volatile("" ::: "memory");
    #pragma unroll 1
    for (int a = 0; a < 4; ++a) {
      v4u w[4];
      #pragma unroll
      for (int k = 0; k < 4; ++k) w[k] = a == 0 ? v[k] : a == 1 ? v[4 + k] : a == 2 ? v[8 + k] : v[12 + k];
      u32 pk[14], escw;
      const bool rawb = pack_block<W, 6, 8192u>(w, raw + a * 16384, pk, area + a * 32, lane, escw);
      if (W && rawb) {
        #pragma unroll
        for (int k = 0; k < 4; ++k)
          __builtin_amdgcn_raw_buffer_store_b128(w[k], rs_my, (int)(rawbase + (u32)(a * 16384 + (k >> 1) * 8192 + (k & 1) * 1024 + (wave * 128 + lane) * 16)), 0, AUX_SYS);
      }
      const u32 o = base + (u32)(a * 4 + wave) * RB;
      #pragma unroll
      for (int k = 0; k < 3; ++k)
        __builtin_amdgcn_raw_buffer_store_b128((v4u){pk[4 * k], pk[4 * k + 1], pk[4 * k + 2], pk[4 * k + 3]}, rs_my, (int)(o + (u32)(k * 1024 + lane * 16)), 0, AUX_SYS);
      if (!W) __builtin_amdgcn_raw_buffer_store_b64((v2u){pk[12], pk[13]}, rs_my, (int)(o + 3072u + (u32)lane * 8u), 0, AUX_SYS);
      if (lane < 32) __builtin_amdgcn_raw_buffer_store_b32(escw, rs_my, (int)(o + RB - 128u + (u32)lane * 4u), 0, AUX_SYS);
    }
  };
  if (g < NXI) {
    const int s = g >> 3, m = g & 7;
    const unsigned char* src = P.x + (size_t)(m * MR + s * TM + wave * 32 + (lane & 31)) * KS + 32 * (lane >> 5);
    v4u v[16];
    #pragma unroll
    for (int j = 0; j < 16; ++j) v[j] = *(const v4u*)(src + (j >> 1) * 64 + (j & 1) * 16);
    __syncthreads();
    call = sm_word[0];
    const u32 dof = OFF_XT + ((call & 1u) * NXI + (u32)g) * ITEM + (u32)(wave * 2 * 64 + lane) * 16u;
    #pragma unroll
    for (int j = 0; j < 16; ++j)
      __builtin_amdgcn_raw_buffer_store_b128(v[j], rs_my, (int)(dof + (u32)((j >> 1) * 8192 + (j & 1) * 1024)), 0, AUX_SYS);
    if (s >= 2) {
      #pragma unroll
      for (int j = 0; j < 16; ++j) {
        const int ch = (j >> 1) * 512 + wave * 128 + (j & 1) * 64 + lane;
        *(v4u*)(smem + (ch ^ ((ch >> 3) & 3)) * 16) = v[j];
      }
      put_packed(std::false_type{}, v, OFF_XC + ((call & 1u) * NXI + (u32)g) * 16u * RBX, 0u);
    }
    WAIT_VM(0);
    __syncthreads();
    if (tid < NR && tid < P.nranks) {
      if (tid == me) __hip_atomic_fetch_add((u32*)(my + OFF_CNTX + g * 128), 4u, __ATOMIC_RELAXED, __HIP_MEMORY_SCOPE_AGENT);
      else __hip_atomic_fetch_add((u32*)(P.ws[tid] + OFF_INFLAG + me * 128 + 8), 1u, __ATOMIC_RELAXED, __HIP_MEMORY_SCOPE_SYSTEM);
    }
  } else if (g >= GW0) {
    const int j = (g - GW0) >> 4, t = (g - GW0) & 15;
    const int q = lane & 31;
    const unsigned char* src = P.w + (size_t)(j * NC + t * TN + q * 4);
    u32 r[4][16];
    #pragma unroll
    for (int a = 0; a < 4; ++a) {
      const int kg = a * 8 + wave * 2 + (lane >> 5);
      #pragma unroll
      for (int i = 0; i < 16; ++i) r[a][i] = *(const u32*)(src + (size_t)(kg * 16 + i) * NN);
    }
    #pragma unroll
    for (int a = 0; a < 4; ++a) {
      const int kg = a * 8 + wave * 2 + (lane >> 5);
      u32 o[4][4];
      #pragma unroll
      for (int b = 0; b < 4; ++b) {
        const u32 r0 = r[a][4 * b], r1 = r[a][4 * b + 1], r2 = r[a][4 * b + 2], r3 = r[a][4 * b + 3];
        const u32 t0 = __builtin_amdgcn_perm(r1, r0, 0x05010400u), t1 = __builtin_amdgcn_perm(r1, r0, 0x07030602u);
        const u32 t2 = __builtin_amdgcn_perm(r3, r2, 0x05010400u), t3 = __builtin_amdgcn_perm(r3, r2, 0x07030602u);
        o[0][b] = __builtin_amdgcn_perm(t2, t0, 0x05040100u);
        o[1][b] = __builtin_amdgcn_perm(t2, t0, 0x07060302u);
        o[2][b] = __builtin_amdgcn_perm(t3, t1, 0x05040100u);
        o[3][b] = __builtin_amdgcn_perm(t3, t1, 0x07060302u);
      }
      const int kb = kg >> 2, half = (kg >> 1) & 1, part = kg & 1, blk = q >> 3;
      const int c0 = ((kb * 4 + blk) * 2 + part) * 64 + half * 32 + (q & 7) * 4;
      #pragma unroll
      for (int c = 0; c < 4; ++c) {
        const int ch = c0 + c;
        const v4u ov = {o[c][0], o[c][1], o[c][2], o[c][3]};
        *(v4u*)(smem + SM_W + (ch ^ ((ch >> 3) & 3)) * 16) = ov;
      }
    }
    __syncthreads();
    STAMP(11);
    call = sm_word[0];
    const u32 dof = OFF_WT + (((call & 1u) * NR + (u32)j) * NT + (u32)t) * ITEM;
    {
      v4u v[16];
      #pragma unroll
      for (int i = 0; i < 16; ++i) {
        const int ch = (i >> 1) * 512 + wave * 128 + (i & 1) * 64 + lane;
        v[i] = *(const v4u*)(smem + SM_W + (ch ^ ((ch >> 3) & 3)) * 16);
        if (j == me) __builtin_amdgcn_raw_buffer_store_b128(v[i], rs_my, (int)(dof + (u32)ch * 16u), 0, AUX_SYS);
      }
      if (j != me) put_packed(std::true_type{}, v, OFF_WC + (((call & 1u) * NR + (u32)j) * NT + (u32)t) * 16u * RBW, dof);
    }
    STAMP(12);
    WAIT_VM(0);
    __syncthreads();
    if (tid == 0 && j < P.nranks) {
      if (j == me) __hip_atomic_fetch_add((u32*)(my + OFF_CNTW + t * 128), 4u, __ATOMIC_RELAXED, __HIP_MEMORY_SCOPE_AGENT);
      else __hip_atomic_fetch_add((u32*)(P.ws[j] + OFF_INFLAG + me * 128 + 4), 1u, __ATOMIC_RELAXED, __HIP_MEMORY_SCOPE_SYSTEM);
    }
  } else if (g < NXI + 7 * NPUSH) {
    const int p  = (me + 1 + (g - NXI) % 7) & (NR - 1), pw = (g - NXI) / 7;
    const int pm = (me + 1 + pw) & (NR - 1);
    const unsigned char* src = P.x + (size_t)(pm * MR + wave * 32 + (lane & 31)) * KS + 32 * (lane >> 5);
    v4u v[16];
    #pragma unroll
    for (int j = 0; j < 16; ++j) v[j] = *(const v4u*)(src + (j >> 1) * 64 + (j & 1) * 16);
    __syncthreads();
    call = sm_word[0];
    if (p < P.nranks) {
      const __amdgpu_buffer_rsrc_t rs_rp = mk_rsrc(P.ws[p], WS_USED);
      const u32 dof = OFF_XP + (((call & 1u) * NR + (u32)me) * NR + (u32)pw) * ITEM + (u32)(wave * 2 * 64 + lane) * 16u;
      #pragma unroll
      for (int j = 0; j < 16; ++j)
        __builtin_amdgcn_raw_buffer_store_b128(v[j], rs_rp, (int)(dof + (u32)((j >> 1) * 8192 + (j & 1) * 1024)), 0, AUX_SYS);
      WAIT_VM(0);
      if (lane == 0) __hip_atomic_fetch_add((u32*)(P.ws[p] + OFF_CNTP + ((call & 1u) * NR + (u32)pm) * 128u), 1u, __ATOMIC_RELAXED, __HIP_MEMORY_SCOPE_SYSTEM);
      USTAMP((g - NXI) % 7, 1000 + pw * 4 + wave);
    }
  } else {
    __syncthreads();
    call = sm_word[0];
  }
  STAMP(1);
  const u32 par = call & 1u;

  const int grp = bid & 7;
  const int gfirst = NPW + ((grp - NPW) & 7);
  const int gcnt = (GRID - 1 - gfirst) / 8 + 1;
  int ph = g >= GW0 && (g - GW0) >> 4 != me ? 0 : 2;
  for (int T = (bid - gfirst) >> 3;; T += gcnt) {
    if (ph < 2) {
      const int j = (g - GW0) >> 4, t = (g - GW0) & 15;
      const unsigned char* xs = P.x + (size_t)(me * MR + ph * 2 * TM + wm * 64 + (lane & 31)) * KS + 32 * (lane >> 5);
      v4u xf[2][8][2];
      auto xload = [&](int s) {
        #pragma unroll
        for (int kb = 0; kb < 8; ++kb)
          #pragma unroll
          for (int b = 0; b < 2; ++b)
            #pragma unroll
            for (int h = 0; h < 2; ++h) xf[b][kb][h] = *(const v4u*)(xs + (size_t)(s * TM + b * 32) * KS + kb * 64 + h * 16);
      };
      xload(0);
      const __amdgpu_buffer_rsrc_t rs_out = mk_rsrc(P.out, (u32)MR * NN * 4u);
      const unsigned char* const fw = smem + SM_W + (wn * 256 + (lane ^ ((lane >> 3) & 3))) * 16;
      #pragma unroll 1
      for (int s = 0; s < 2; ++s) {
        v16f acc[2][2];
        #pragma unroll
        for (int i = 0; i < 2; ++i)
          #pragma unroll
          for (int k = 0; k < 2; ++k)
            #pragma unroll
            for (int e = 0; e < 16; ++e) acc[i][k][e] = 0.f;
        #pragma unroll
        for (int kb = 0; kb < 8; ++kb) {
          v8i X[2], W[2];
          #pragma unroll
          for (int b = 0; b < 2; ++b) {
            const v4u w0 = *(const v4u*)(fw + kb * 8192 + b * 2048), w1 = *(const v4u*)(fw + kb * 8192 + b * 2048 + 1024);
            const v4u x0 = xf[b][kb][0], x1 = xf[b][kb][1];
            X[b] = (v8i){(int)x0[0], (int)x0[1], (int)x0[2], (int)x0[3], (int)x1[0], (int)x1[1], (int)x1[2], (int)x1[3]};
            W[b] = (v8i){(int)w0[0], (int)w0[1], (int)w0[2], (int)w0[3], (int)w1[0], (int)w1[1], (int)w1[2], (int)w1[3]};
          }
          #pragma unroll
          for (int i = 0; i < 2; ++i)
            #pragma unroll
            for (int k = 0; k < 2; ++k)
              acc[i][k] = __builtin_amdgcn_mfma_scale_f32_32x32x64_f8f6f4(W[i], X[k], acc[i][k], 1, 0, 0, 0x7F7F7F7F, 0, 0x7F7F7F7F);
        }
        SCHED_FENCE;
        if (s == 0) xload(1);
        WAIT_LGKM0; RAW_BAR;
        put_image(acc, true);
        WAIT_LGKM0; RAW_BAR;
        const u32 odof = ((u32)((ph * 2 + s) * TM + r0) * (u32)NN + (u32)(j * NC + t * TN + c4)) * 4u;
        #pragma unroll
        for (int i = 0; i < 16; ++i) {
          const v4f o = *(const v4f*)(img + (r0 + i * 8) * EPI_LD + c4);
          __builtin_amdgcn_raw_buffer_store_b128(__builtin_bit_cast(v4u, o), rs_out, (int)(odof + (u32)(i * 8) * (u32)(NN * 4)), 0, AUX_PST);
        }
      }
      WAIT_VM(0);
      __syncthreads();
      if (tid == 0) __hip_atomic_fetch_add((u32*)(my + OFF_PCNT + ph * 128), 1u, __ATOMIC_RELAXED, __HIP_MEMORY_SCOPE_AGENT);
      if (ph == 0) STAMP(13);
      ++ph;
    }
    if (T >= NTILE / 8) { if (ph < 2) continue; break; }
    const int s = T >> 4, t = (grp >> 1) * 4 + ((T >> 2) & 3), m = (grp & 1) * 4 + (T & 3);
    const int xi = s * 8 + m;
    __syncthreads();
    if (tid == 0) {
      const u32 np = s != 0 ? 0u : (u32)NPUSH - (((u32)(m - me - 1) & (NR - 1)) < (u32)NPUSH ? 1u : 0u);
      bool ok = wait_ge(my + OFF_CNTW + t * 128, CNT_PER_CALL * call, tmo, call);
      if (ok) ok = wait_ge(my + OFF_CNTX + xi * 128, 4u * ((m == me ? NR : NR - 1) - np) * call, tmo, call);
      if (ok) ok = wait_ge(my + OFF_CNTP + (par * NR + (u32)m) * 128u, 4u * np * ((call + 1u) >> 1), tmo, call);
      if (s == 0) __builtin_amdgcn_fence(__ATOMIC_ACQUIRE, "");
      else        __builtin_amdgcn_fence(__ATOMIC_ACQUIRE, "agent");
      WAIT_VM(0);
      if (ok) ok = (ld_agent(tmo) != call);
      sm_word[1] = ok ? 1u : 0u;
    }
    __syncthreads();
    const bool ok = sm_word[1] != 0u;
    STAMP(2 + 3 * (T / gcnt));

    v16f acc[2][2];
    #pragma unroll
    for (int i = 0; i < 2; ++i)
      #pragma unroll
      for (int j = 0; j < 2; ++j)
        #pragma unroll
        for (int e = 0; e < 16; ++e) acc[i][j][e] = 0.f;

    if (ok) {
      const unsigned char* xo = my + OFF_XT + (par * NXI + (u32)xi) * ITEM + tid * 16;
      const unsigned char* xl = my + OFF_XL + (u32)xi * ITEM + tid * 16;
      const unsigned char* xp = my + OFF_XP + par * NR * NR * ITEM + tid * 16;
      const unsigned char* wo = my + OFF_WT + ((par * NR + (u32)me) * NT + (u32)t) * ITEM + tid * 16;
      const int skip = m == me ? NR : m;
      const int nst = m == me ? 64 : 56;
      const unsigned char* wl = my + OFF_WL + (u32)t * ITEM + tid * 16;
      unsigned char* const ldst = smem + wave * 1024;
      auto issue = [&](int ks) {
        int sr = ks >> 3; sr += sr >= skip ? 1 : 0;
        const int kb = ks & 7;
        const u32 d = (u32)(m - sr - 1) & (NR - 1);
        const unsigned char* xa = (sr == me ? xo : (s == 0 && d < (u32)NPUSH) ? xp + (size_t)(((u32)sr * NR + d) * ITEM)
                                                                            : xl + (size_t)sr * (NXI * ITEM)) + kb * 8192;
        const unsigned char* wa = (sr == me ? wo : wl + (size_t)sr * (NT * ITEM)) + kb * 8192;
        unsigned char* l = ldst + (ks & 3) * 16384;
        __builtin_amdgcn_global_load_lds((const __attribute__((address_space(1))) void*)(xa),        (__attribute__((address_space(3))) void*)(l),         16, 0, 0);
        __builtin_amdgcn_global_load_lds((const __attribute__((address_space(1))) void*)(xa + 4096), (__attribute__((address_space(3))) void*)(l + 4096),  16, 0, 0);
        __builtin_amdgcn_global_load_lds((const __attribute__((address_space(1))) void*)(wa),        (__attribute__((address_space(3))) void*)(l + 8192),  16, 0, 0);
        __builtin_amdgcn_global_load_lds((const __attribute__((address_space(1))) void*)(wa + 4096), (__attribute__((address_space(3))) void*)(l + 12288), 16, 0, 0);
      };
      const unsigned char* const fx = smem + (wm * 2) * 2048 + lane * 16;
      const unsigned char* const fw = smem + 8192 + (wn * 2) * 2048 + lane * 16;
      auto rdfrag = [&](int ks, v8i (&X)[2], v8i (&W)[2]) {
        const int so = (ks & 3) * 16384;
        #pragma unroll
        for (int b = 0; b < 2; ++b) {
          const v4u x0 = *(const v4u*)(fx + so + b * 2048), x1 = *(const v4u*)(fx + so + b * 2048 + 1024);
          const v4u w0 = *(const v4u*)(fw + so + b * 2048), w1 = *(const v4u*)(fw + so + b * 2048 + 1024);
          X[b] = (v8i){(int)x0[0], (int)x0[1], (int)x0[2], (int)x0[3], (int)x1[0], (int)x1[1], (int)x1[2], (int)x1[3]};
          W[b] = (v8i){(int)w0[0], (int)w0[1], (int)w0[2], (int)w0[3], (int)w1[0], (int)w1[1], (int)w1[2], (int)w1[3]};
        }
      };
      auto mma = [&](v8i (&X)[2], v8i (&W)[2]) {
        #pragma unroll
        for (int i = 0; i < 2; ++i)
          #pragma unroll
          for (int j = 0; j < 2; ++j)
            acc[i][j] = __builtin_amdgcn_mfma_scale_f32_32x32x64_f8f6f4(W[i], X[j], acc[i][j], 1, 0, 0, 0x7F7F7F7F, 0, 0x7F7F7F7F);
      };
      v8i XA[2], WA[2], XB[2], WB[2];
      issue(0); issue(1); issue(2);
      WAIT_VM(8); RAW_BAR;
      issue(3);
      rdfrag(0, XA, WA);
      for (int ks = 0; ks < nst - 4; ks += 2) {
        WAIT_VM(8); WAIT_LGKM0; RAW_BAR;
        issue(ks + 4);
        rdfrag(ks + 1, XB, WB);
        SCHED_FENCE;
        mma(XA, WA);
        SCHED_FENCE;
        WAIT_VM(8); WAIT_LGKM0; RAW_BAR;
        issue(ks + 5);
        rdfrag(ks + 2, XA, WA);
        SCHED_FENCE;
        mma(XB, WB);
        SCHED_FENCE;
      }
      WAIT_VM(8); WAIT_LGKM0; RAW_BAR;
      rdfrag(61, XB, WB); SCHED_FENCE; mma(XA, WA); SCHED_FENCE;
      WAIT_VM(4); WAIT_LGKM0; RAW_BAR;
      rdfrag(62, XA, WA); SCHED_FENCE; mma(XB, WB); SCHED_FENCE;
      WAIT_VM(0); WAIT_LGKM0; RAW_BAR;
      rdfrag(63, XB, WB); SCHED_FENCE; mma(XA, WA); SCHED_FENCE;
      mma(XB, WB);
    }
    STAMP(3 + 3 * (T / gcnt));
    __syncthreads();
    {
      put_image(acc, ok);
      __syncthreads();
      if (m == me) {
        float* dst = P.out + (size_t)(s * TM + r0) * NN + me * NC + t * TN + c4;
        #pragma unroll
        for (int i = 0; i < 16; ++i) {
          const v4f o = *(const v4f*)(img + (r0 + i * 8) * EPI_LD + c4);
          *(v4f*)(dst + (size_t)(i * 8) * NN) = o;
        }
      } else {
        const u32 dof = OFF_OUT + ((par * NR + (u32)m) * 64u + (u32)(s * NT + t)) * ITEM + (u32)(r0 * 512 + c4 * 4);
        #pragma unroll
        for (int i = 0; i < 16; ++i) {
          const v4f o = *(const v4f*)(img + (r0 + i * 8) * EPI_LD + c4);
          __builtin_amdgcn_raw_buffer_store_b128(__builtin_bit_cast(v4u, o), rs_my, (int)(dof + (u32)(i * 8 * 512)), 0, AUX_SYS);
        }
        WAIT_VM(0);
        __syncthreads();
        if (tid == 0 && m < P.nranks)
          __hip_atomic_store((u32*)(P.ws[m] + OFF_OFLAG + (u32)(me * 64 + s * NT + t) * 128u), call, __ATOMIC_RELAXED, __HIP_MEMORY_SCOPE_SYSTEM);
      }
    }
    STAMP(4 + 3 * (T / gcnt));
  }
}

extern "C" void kernel_launch(int rank, int nranks, ncclComm_t comm,
                              hipStream_t stream,
                              void* const* d_in, const int* in_sizes, int n_in,
                              void* d_out, int out_size,
                              const int* in_dtypes, int out_dtype,
                              void* const* d_out_all,
                              void* const* d_ws_all, size_t ws_bytes) {
  (void)comm; (void)in_sizes; (void)in_dtypes; (void)out_dtype; (void)d_out_all;
  if (nranks != NR || n_in < 4 || ws_bytes < (size_t)WS_USED || out_size != MR * NN || rank < 0 || rank >= NR) {
    fprintf(stderr, "unexpected configuration\n");
    return;
  }
  Params P;
  P.x  = (const unsigned char*)d_in[0];
  P.w  = (const unsigned char*)d_in[1];
  P.sx = (const float*)d_in[2];
  P.sw = (const float*)d_in[3];
  P.out = (float*)d_out;
  for (int p = 0; p < NR; ++p) P.ws[p] = (unsigned char*)d_ws_all[p];
  P.rank = rank;
  P.nranks = nranks;
  fused_gemm_rs<<<GRID, TPB, 0, stream>>>(P);
  HIP_CHECK(hipGetLastError());
}
```

```cpp
#include <hip/hip_runtime.h>
#include <rccl/rccl.h>
#include <cstdio>
#include <cstdint>
#include <type_traits>

#define HIP_CHECK(expr) do { hipError_t _e = (expr); if (_e != hipSuccess) { \
    fprintf(stderr, "HIP error %s at %s:%d\n", hipGetErrorString(_e), __FILE__, __LINE__); \
    return; } } while (0)
#define NCCL_CHECK(expr) do { ncclResult_t _r = (expr); if (_r != ncclSuccess) { \
    fprintf(stderr, "RCCL error %s at %s:%d\n", ncclGetErrorString(_r), __FILE__, __LINE__); \
    return; } } while (0)

typedef unsigned u32;
typedef unsigned long long u64;
typedef int      v8i  __attribute__((ext_vector_type(8)));
typedef unsigned v4u  __attribute__((ext_vector_type(4)));
typedef unsigned v3u  __attribute__((ext_vector_type(3)));
typedef unsigned v2u  __attribute__((ext_vector_type(2)));
typedef float    v4f  __attribute__((ext_vector_type(4)));
typedef float    v16f __attribute__((ext_vector_type(16)));

constexpr int NR    = 8;
constexpr int MM    = 4096;
constexpr int KK    = 4096;
constexpr int NN    = 16384;
constexpr int KS    = KK / NR;
constexpr int NC    = NN / NR;
constexpr int MR    = MM / NR;
constexpr int TM    = 128;
constexpr int TN    = 128;
constexpr int NSL   = MR / TM;
constexpr int NT    = NC / TN;
constexpr int NXI   = NSL * NR;
constexpr int NTILE = NXI * NT;
constexpr u32 ITEM  = 65536;
constexpr u32 UNIT  = 16384;

constexpr int GRID  = 256;
constexpr int TPB   = 256;
#define PWG 6
constexpr int NPW   = 7 * PWG;
constexpr int NGW   = GRID - NPW;
constexpr int GW0   = NGW - NR * NT;

constexpr u32 OFF_TICKET = 0;
constexpr u32 OFF_TMO    = 2048;
constexpr u32 OFF_UCTR   = 3072;
constexpr u32 OFF_INFLAG = 4096;
constexpr u32 OFF_CNTX   = 8192;
constexpr u32 OFF_CNTW   = 16384;
constexpr u32 OFF_CNTP   = 24576;
constexpr u32 OFF_PCNT   = 32768;
constexpr u32 OFF_OFLAG  = 65536;
constexpr u32 OFF_XT     = 1u << 20;
constexpr u32 OFF_WT     = OFF_XT + 2u * NXI * ITEM;
constexpr u32 OFF_XL     = OFF_WT + 2u * NR * NT * ITEM;
constexpr u32 OFF_WL     = OFF_XL + (u32)NR * NXI * ITEM;
constexpr u32 OFF_OUT    = OFF_WL + (u32)NR * NT * ITEM;
#define NPUSH 4
#define TMAP 1
#define TAILT 4
constexpr u32 OFF_XP     = OFF_OUT + 2u * NR * (NSL * NT) * ITEM;
constexpr u32 RBW        = 3072 + 128;
constexpr u32 RBX        = 3584 + 128;
constexpr u32 OFF_WC     = OFF_XP + 2u * NR * NR * ITEM;
constexpr u32 OFF_XC     = OFF_WC + 2u * NR * NT * 16u * RBW;
constexpr u32 WS_USED    = OFF_XC + 2u * NXI * 16u * RBX;
static_assert(NPUSH <= NR - 1 && NXI + 7 * NPUSH <= GW0, "pushed x items: row blocks of slice 0, one per otherwise idle workgroup");

constexpr u32 CNT_PER_CALL = NR * 4;
constexpr u64 DEADLINE_TICKS = 100000000ull;

constexpr int EPI_LD   = TN + 4;
constexpr int SM_BYTES = TM * EPI_LD * 4;
constexpr int SM_W     = SM_BYTES + 2048;
constexpr int SM_TOTAL = SM_W + 65536;

constexpr int AUX_SYS   = 17;
constexpr int AUX_AGENT = 16;
#define AUX_OUT 16
#define AUX_PLD 16
#define AUX_PST 16
constexpr int NUX1  = (2 * (NR - 1) - NPUSH) * 4;
#define NUA 20
static_assert(NUA % 4 == 0 && NUA <= NUX1 && NUA >= (NR - 1 - NPUSH) * 4, "x units pulled ahead of w");
constexpr int NUW   = NUA + NT * 4;
constexpr int NUIN  = (NT + NXI - NSL - NPUSH) * 4;
constexpr int NUBIG = NUIN + (NSL * NT - TAILT) * 4;
constexpr int NUNIT = NUBIG + TAILT * 16;
static_assert(TAILT >= 1 && TAILT <= NT, "tail tiles: the last column tiles of the last slice");
static_assert(NPW >= NUX1, "the first x units are packed by the pullers, one unit per workgroup");

__device__ __forceinline__ u32 x_item(u32 z, u32 p) {
  if (z < (u32)(NR - 1 - NPUSH)) return (p + 1u + NPUSH + z) & (NR - 1);
  const u32 y = z - (u32)(NR - 1 - NPUSH), i = y % 7u;
  return (1u + y / 7u) * 8u + i + (i >= p ? 1u : 0u);
}

#define STAMP(slot) do {} while (0)
#define USTAMP(g, u) do {} while (0)

struct Params {
  const unsigned char* x;
  const unsigned char* w;
  const float* sx;
  const float* sw;
  float* out;
  unsigned char* ws[NR];
  int rank;
  int nranks;
};

#define WAIT_VM(n)  asm volatile("s_waitcnt vmcnt(" #n ")" ::: "memory")
#define WAIT_VM4    do { __builtin_amdgcn_s_waitcnt(0x0F74); asm volatile("" ::: "memory"); } while (0)
#define WAIT_VM0    do { __builtin_amdgcn_s_waitcnt(0x0F70); asm volatile("" ::: "memory"); } while (0)
#define WAIT_LGKM0  do { __builtin_amdgcn_s_waitcnt(0xC07F); asm volatile("" ::: "memory"); } while (0)
#define RAW_BAR     __builtin_amdgcn_s_barrier()
#define SCHED_FENCE __builtin_amdgcn_sched_barrier(0)

__device__ __forceinline__ u32 ld_sys(const unsigned char* p) {
  return __hip_atomic_load((const u32*)p, __ATOMIC_RELAXED, __HIP_MEMORY_SCOPE_SYSTEM);
}
__device__ __forceinline__ u32 ld_sys1(const unsigned char* p) {
  return __builtin_amdgcn_readfirstlane(__hip_atomic_load((const u32*)p, __ATOMIC_RELAXED, __HIP_MEMORY_SCOPE_SYSTEM));
}
__device__ __forceinline__ u32 ld_agent(const unsigned char* p) {
  return __builtin_amdgcn_readfirstlane(__hip_atomic_load((const u32*)p, __ATOMIC_RELAXED, __HIP_MEMORY_SCOPE_AGENT));
}

__device__ __forceinline__ bool wait_ge(const unsigned char* flag, u32 target, unsigned char* tmo, u32 call) {
  if ((int)(ld_sys1(flag) - target) >= 0) return true;
  const u64 t0 = __builtin_amdgcn_s_memrealtime();
  for (;;) {
    if ((int)(ld_sys1(flag) - target) >= 0) return true;
    if (ld_agent(tmo) == call || __builtin_amdgcn_s_memrealtime() - t0 > DEADLINE_TICKS) {
      __hip_atomic_store((u32*)tmo, call, __ATOMIC_RELAXED, __HIP_MEMORY_SCOPE_AGENT);
      return false;
    }
    __builtin_amdgcn_s_sleep(2);
  }
}

__device__ __forceinline__ bool wait_flags(const unsigned char* base, int n, int lane, u32 target, unsigned char* tmo, u32 call) {
  const u64 t0 = __builtin_amdgcn_s_memrealtime();
  for (;;) {
    const u32 f = lane < n ? ld_sys(base + lane * 128) : target;
    if (__builtin_amdgcn_ballot_w64((int)(f - target) < 0) == 0ull) return true;
    if (ld_agent(tmo) == call || __builtin_amdgcn_s_memrealtime() - t0 > DEADLINE_TICKS) {
      __hip_atomic_store((u32*)tmo, call, __ATOMIC_RELAXED, __HIP_MEMORY_SCOPE_AGENT);
      return false;
    }
    __builtin_amdgcn_s_sleep(2);
  }
}

__device__ __forceinline__ __amdgpu_buffer_rsrc_t mk_rsrc(const void* p, u32 bytes) {
  return __builtin_amdgcn_make_buffer_rsrc((void*)p, (short)0, (int)bytes, 0x00020000);
}

typedef __attribute__((address_space(3))) unsigned char lds_u8;
typedef __attribute__((address_space(3))) u32 lds_u32;
typedef __attribute__((address_space(3))) v4u lds_v4u;

__device__ __forceinline__ u32 lane_prefix(u32 n) {
  u32 off = 0;
  #pragma unroll
  for (int b = 0; b < 5; ++b) {
    const u64 m = __builtin_amdgcn_ballot_w64(((n >> b) & 1u) != 0u);
    off += __builtin_amdgcn_mbcnt_hi((u32)(m >> 32), __builtin_amdgcn_mbcnt_lo((u32)m, 0u)) << b;
  }
  return off;
}
__device__ __forceinline__ u32 flag_mask(const u32* e) {
  u32 t[4];
  #pragma unroll
  for (int k = 0; k < 4; ++k) t[k] = __builtin_amdgcn_udot4(e[2 * k + 1], 0x80402010u, __builtin_amdgcn_udot4(e[2 * k], 0x08040201u, 0u, false), false);
  return (t[0] >> 7) | (t[1] << 1) | (t[2] << 9) | (t[3] << 17);
}

template <bool W, int S, u32 H>
__device__ __forceinline__ bool pack_block(const v4u* v, const lds_u8* raw, u32 (&pk)[14], lds_u32* area, int lane, u32& escw) {
  u32 c[16], e[16];
  #pragma unroll
  for (int i = 0; i < 16; ++i) {
    const u32 d = v[i >> 2][i & 3];
    const u32 y = (((d & (W ? 0x7C7C7C7Cu : 0x78787878u)) | 0x80808080u) - (W ? 0x28282828u : 0x10101010u)) & 0x7F7F7F7Fu;
    e[i] = (y + (W ? 0x64646464u : 0x48484848u)) & 0x80808080u;
    c[i] = W ? (d & 0x83838383u) | ((y | ((e[i] >> 2) - (e[i] >> 5))) & 0x1C1C1C1Cu)
             : (d & 0x87878787u) | ((y | ((e[i] >> 1) - (e[i] >> 4))) & 0x38383838u);
  }
  if (W) {
    #pragma unroll
    for (int k = 0; k < 4; ++k) {
      const u32 z = c[4 * k + 3];
      pk[3 * k]     = c[4 * k]     | ((z << 5) & 0x60606060u);
      pk[3 * k + 1] = c[4 * k + 1] | ((z << 3) & 0x60606060u);
      pk[3 * k + 2] = c[4 * k + 2] | ((z << 1) & 0x20202020u) | ((z >> 1) & 0x40404040u);
    }
    pk[12] = 0; pk[13] = 0;
  } else {
    #pragma unroll
    for (int k = 0; k < 2; ++k) {
      const u32 z = c[8 * k + 7];
      #pragma unroll
      for (int i = 0; i < 6; ++i) pk[7 * k + i] = c[8 * k + i] | ((z << (6 - i)) & 0x40404040u);
      pk[7 * k + 6] = c[8 * k + 6] | ((z >> 1) & 0x40404040u);
    }
  }
  u32 m0 = flag_mask(e), m1 = flag_mask(e + 8);
  const u32 n0 = (u32)__builtin_popcount(m0), n1 = (u32)__builtin_popcount(m1);
  u32 a0 = 0, a1 = 0, top = 0;
  bool rawb = __builtin_amdgcn_ballot_w64(n0 > 8u || n1 > 8u) != 0ull;
  if (!rawb) {
    for (u32 sh = 0; __builtin_amdgcn_ballot_w64((m0 | m1) != 0u) != 0ull; sh += 4u) {
      const u32 p0 = (u32)__builtin_ctz(m0 | 0x80000000u), p1 = (u32)__builtin_ctz(m1 | 0x80000000u);
      const u32 b0 = raw[((p0 & 16u) << S) + (p0 & 15u)], b1 = raw[H + ((p1 & 16u) << S) + (p1 & 15u)];
      u32 f0 = W ? (b0 >> 2) & 31u : (b0 >> 3) & 15u, f1 = W ? (b1 >> 2) & 31u : (b1 >> 3) & 15u;
      if (W) { f0 = f0 < 10u ? f0 : f0 - 7u; f1 = f1 < 10u ? f1 : f1 - 7u; }
      f0 = m0 != 0u ? f0 : 0u; f1 = m1 != 0u ? f1 : 0u;
      top |= f0 | f1;
      a0 |= f0 << sh; a1 |= f1 << sh;
      m0 &= m0 - 1u; m1 &= m1 - 1u;
    }
  }
  const u32 n = rawb ? 0u : n0 + n1;
  const u32 off = lane_prefix(n);
  rawb = rawb || __builtin_amdgcn_ballot_w64(top > 15u) != 0ull || __builtin_amdgcn_readlane(off + n, 63) > 254u;
  if (!rawb && n != 0u) {
    const u64 a = (u64)a0 | ((u64)a1 << (4u * n0));
    const u32 sh = (off & 7u) * 4u, l = (u32)a, h = (u32)(a >> 32);
    lds_u32* q = area + (off >> 3);
    __hip_atomic_fetch_or(q,     l << sh, __ATOMIC_RELAXED, __HIP_MEMORY_SCOPE_WORKGROUP);
    __hip_atomic_fetch_or(q + 1, (u32)(a >> (32u - sh)), __ATOMIC_RELAXED, __HIP_MEMORY_SCOPE_WORKGROUP);
    __hip_atomic_fetch_or(q + 2, (u32)((u64)h >> (32u - sh)), __ATOMIC_RELAXED, __HIP_MEMORY_SCOPE_WORKGROUP);
  }
  asm volatile("" ::: "memory");
  escw = __hip_atomic_load(area + (lane & 31), __ATOMIC_RELAXED, __HIP_MEMORY_SCOPE_WORKGROUP);
  if (rawb && (lane & 31) == 31) escw = 0xFF000000u;
  return rawb;
}

template <bool W>
__device__ __forceinline__ void unpack_block(const u32 (&pk)[14], u32 escw, lds_u8* st, v4u (&o)[4]) {
  u32 c[16], e[16];
  if (W) {
    #pragma unroll
    for (int k = 0; k < 4; ++k) {
      c[4 * k] = pk[3 * k]; c[4 * k + 1] = pk[3 * k + 1]; c[4 * k + 2] = pk[3 * k + 2];
      c[4 * k + 3] = ((pk[3 * k] >> 5) & 0x03030303u) | ((pk[3 * k + 1] >> 3) & 0x0C0C0C0Cu) |
                     ((pk[3 * k + 2] >> 1) & 0x10101010u) | ((pk[3 * k + 2] << 1) & 0x80808080u);
    }
  } else {
    #pragma unroll
    for (int k = 0; k < 2; ++k) {
      u32 t = (pk[7 * k + 6] << 1) & 0x80808080u;
      #pragma unroll
      for (int i = 0; i < 6; ++i) t |= (pk[7 * k + i] >> (6 - i)) & (0x01010101u << i);
      #pragma unroll
      for (int i = 0; i < 7; ++i) c[8 * k + i] = pk[7 * k + i];
      c[8 * k + 7] = t;
    }
  }
  #pragma unroll
  for (int i = 0; i < 16; ++i) {
    e[i] = W ? ((c[i] & 0x1F1F1F1Fu) + 0x64646464u) & 0x80808080u : ((c[i] & 0x3F3F3F3Fu) + 0x48484848u) & 0x80808080u;
    c[i] = W ? (c[i] & 0x9F9F9F9Fu) + 0x28282828u : (c[i] & 0xBFBFBFBFu) + 0x10101010u;
  }
  #pragma unroll
  for (int k = 0; k < 4; ++k) *(lds_v4u*)(st + k * 16) = (v4u){c[4 * k], c[4 * k + 1], c[4 * k + 2], c[4 * k + 3]};
  u32 m0 = flag_mask(e), m1 = flag_mask(e + 8);
  const u32 n0 = (u32)__builtin_popcount(m0);
  const u32 off = lane_prefix(n0 + (u32)__builtin_popcount(m1));
  const int ix = (int)(off >> 3) * 4;
  const u32 w0 = (u32)__builtin_amdgcn_ds_bpermute(ix, (int)escw), w1 = (u32)__builtin_amdgcn_ds_bpermute(ix + 4, (int)escw),
            w2 = (u32)__builtin_amdgcn_ds_bpermute(ix + 8, (int)escw);
  const u32 s0 = (off & 7u) * 4u;
  const u32 a0 = __builtin_amdgcn_alignbit(w1, w0, s0);
  const u32 a1 = (u32)((((u64)__builtin_amdgcn_alignbit(w2, w1, s0) << 32) | a0) >> (4u * n0));
  asm volatile("" ::: "memory");
  for (u32 sh = 0; __builtin_amdgcn_ballot_w64((m0 | m1) != 0u) != 0ull; sh += 4u) {
    const u32 p0 = (u32)__builtin_ctz(m0 | 0x80000000u), p1 = (u32)__builtin_ctz(m1 | 0x80000000u);
    u32 f0 = (a0 >> sh) & 15u, f1 = (a1 >> sh) & 15u;
    if (W) { f0 += ((f0 + 6u) >> 4) * 7u; f1 += ((f1 + 6u) >> 4) * 7u; }
    f0 = m0 != 0u ? f0 - (W ? 17u : 9u) : 0u; f1 = m1 != 0u ? f1 - (W ? 17u : 9u) : 0u;
    __hip_atomic_fetch_add((lds_u32*)(st + (p0 & 28u)),       f0 << ((p0 * 8u + (W ? 2u : 3u)) & 31u), __ATOMIC_RELAXED, __HIP_MEMORY_SCOPE_WORKGROUP);
    __hip_atomic_fetch_add((lds_u32*)(st + 32u + (p1 & 28u)), f1 << ((p1 * 8u + (W ? 2u : 3u)) & 31u), __ATOMIC_RELAXED, __HIP_MEMORY_SCOPE_WORKGROUP);
    m0 &= m0 - 1u; m1 &= m1 - 1u;
  }
  asm volatile("" ::: "memory");
  #pragma unroll
  for (int k = 0; k < 4; ++k) o[k] = *(volatile lds_v4u*)(st + k * 16);
}

__global__ void __launch_bounds__(TPB, 1) fused_gemm_rs(Params P) {
  __shared__ __attribute__((aligned(16))) unsigned char smem[SM_TOTAL];
  const int tid  = threadIdx.x;
  const int lane = tid & 63;
  const int wave = tid >> 6;
  const int me   = P.rank;
  const int bid  = blockIdx.x;
  unsigned char* const my = P.ws[me];
  unsigned char* const tmo = my + OFF_TMO;
  volatile u32* const sm_word = (volatile u32*)(smem + SM_BYTES);
  const __amdgpu_buffer_rsrc_t rs_my = mk_rsrc(my, WS_USED);

  STAMP(0);
  if (tid == 0) {
    const u32 ticket = __hip_atomic_fetch_add((u32*)(my + OFF_TICKET + (bid & 15) * 128), 1u, __ATOMIC_RELAXED, __HIP_MEMORY_SCOPE_AGENT);
    sm_word[0] = ticket / (u32)(GRID / 16) + 1u;
  }

  if (bid < NPW) {
    const int p    = (me + 1 + bid % 7) & (NR - 1);
    const bool enc = bid < NUX1;
    const int ei   = (int)x_item((u32)bid >> 2, (u32)me);
    v4u ev[4];
    if (enc) {
      const unsigned char* src = P.x + (size_t)((ei & 7) * MR + (ei >> 3) * TM + wave * 32 + (lane & 31)) * KS + 32 * (lane >> 5) + (bid & 3) * 128;
      #pragma unroll
      for (int j = 0; j < 4; ++j) ev[j] = *(const v4u*)(src + (j >> 1) * 64 + (j & 1) * 16);
    }
    __syncthreads();
    const u32 call = __builtin_amdgcn_readfirstlane(sm_word[0]);
    const u32 par  = call & 1u;
    if (enc) {
      lds_u32* const area = (lds_u32*)(smem + 65536) + wave * 32;
      lds_u8* const st = (lds_u8*)smem + (u32)tid * 80u;
      area[lane & 31] = 0u;
      #pragma unroll
      for (int j = 0; j < 4; ++j) *(lds_v4u*)(st + j * 16) = ev[j];
      asm volatile("" ::: "memory");
      u32 pk[14], escw;
      pack_block<false, 0, 32u>(ev, st, pk, area, lane, escw);
      const u32 o = OFF_XC + ((par * NXI + (u32)ei) * 16u + (u32)((bid & 3) * 4 + wave)) * RBX;
      #pragma unroll
      for (int k = 0; k < 3; ++k)
        __builtin_amdgcn_raw_buffer_store_b128((v4u){pk[4 * k], pk[4 * k + 1], pk[4 * k + 2], pk[4 * k + 3]}, rs_my, (int)(o + (u32)(k * 1024 + lane * 16)), 0, AUX_SYS);
      __builtin_amdgcn_raw_buffer_store_b64((v2u){pk[12], pk[13]}, rs_my, (int)(o + 3072u + (u32)lane * 8u), 0, AUX_SYS);
      if (lane < 32) __builtin_amdgcn_raw_buffer_store_b32(escw, rs_my, (int)(o + RBX - 128u + (u32)lane * 4u), 0, AUX_SYS);
      WAIT_VM(0);
      __syncthreads();
      if (tid < NR && tid < P.nranks && tid != me)
        __hip_atomic_fetch_add((u32*)(P.ws[tid] + OFF_INFLAG + me * 128), 1u, __ATOMIC_RELAXED, __HIP_MEMORY_SCOPE_SYSTEM);
      STAMP(8);
    }
    if ((unsigned)p >= (unsigned)P.nranks) return;
    const __amdgpu_buffer_rsrc_t rs_rp  = mk_rsrc(P.ws[p], WS_USED);
    const __amdgpu_buffer_rsrc_t rs_out = mk_rsrc(P.out, (u32)MR * NN * 4u);
    const u32 l16 = (u32)lane * 16u;

    u32 zoff = 0;
    asm volatile("" : "+v"(zoff));
    u32* const uctr = (u32*)(my + OFF_UCTR + p * 128 + zoff);
    const u32 ubase = (call - 1u) * (u32)(NUNIT + 4 * PWG);
    const u32 fl_base = OFF_OFLAG + (u32)p * 64u * 128u;
    u32 tk = 0;
    if (lane == 0) tk = __hip_atomic_fetch_add(uctr, 1u, __ATOMIC_RELAXED, __HIP_MEMORY_SCOPE_AGENT);
    bool ok = true;
    u32 u = __builtin_amdgcn_readfirstlane(tk) - ubase;
    unsigned char* pend = nullptr;
    u32 fl = 0;
    bool pf = false;
    u32 iph = 99u;
    const u32 ftl = (u32)(NSL * NT - TAILT) + (u32)(lane < TAILT ? lane : 0);
    u32 pc = 0, pct = 0;
    const u32 pc_base = OFF_PCNT + (u32)p * 16u * 128u;
    bool pre = false;
    v4u a0, a1, a2, a3;
    const u32 tlw = lane < (int)((RBW - 3072u) / 16u) ? 3072u + l16 : 0x7FFFFFF0u;
    const u32 tlx = lane < (int)((RBX - 3072u) / 16u) ? 3072u + l16 : 0x7FFFFFF0u;
    lds_u8* const ta = (lds_u8*)smem + 32768 + wave * 1024;
    auto fetch = [&](u32 o, u32 tl) {
      a0 = __builtin_amdgcn_raw_buffer_load_b128(rs_rp, (int)(o + l16), 0, AUX_SYS);
      a1 = __builtin_amdgcn_raw_buffer_load_b128(rs_rp, (int)(o + 1024u + l16), 0, AUX_SYS);
      a2 = __builtin_amdgcn_raw_buffer_load_b128(rs_rp, (int)(o + 2048u + l16), 0, AUX_SYS);
      a3 = __builtin_amdgcn_raw_buffer_load_b128(rs_rp, (int)(o + tl), 0, AUX_SYS);
    };
    auto out_flags = [&](u32 z) {
      pf = z >= (u32)NUBIG;
      fl = __builtin_amdgcn_raw_buffer_load_b32(rs_my, (int)(fl_base + (pf ? ftl : ((z - NUIN) >> 2)) * 128u), 0, AUX_SYS);
      pc = __builtin_amdgcn_raw_buffer_load_b32(rs_my, (int)(pc_base + ((pf ? ftl : ((z - NUIN) >> 2)) & 15u) * 128u), 0, AUX_SYS);
      pct = 4u * call - 3u + (pf ? 3u : (z - NUIN) >> 6);
    };
    bool flok = false, pcok = false;
    auto eval_flags = [&]() {
      flok = __builtin_amdgcn_ballot_w64((int)(fl - call) < 0) == 0ull;
      pcok = __builtin_amdgcn_ballot_w64((int)(pc - pct) < 0) == 0ull;
    };
    u32 un = 0;
    while (u < (u32)NUIN) {
      {
        u32 ph = u < (u32)NUA ? 0u : u < (u32)NUW ? 1u : 2u;
        if (ok && ph != iph) {
          if (pend != nullptr) {
            WAIT_VM(0);
            if (lane == 0) __hip_atomic_fetch_add((u32*)pend, 1u, __ATOMIC_RELAXED, __HIP_MEMORY_SCOPE_AGENT);
            pend = nullptr;
          }
          ok = wait_ge(my + OFF_INFLAG + p * 128 + ph * 4u, (ph == 0u ? (u32)NUX1 : ph == 1u ? (u32)NT : (u32)NXI) * call, tmo, call);
          if (ok && ph == 2u) ok = wait_ge(my + OFF_INFLAG + p * 128, (u32)NUX1 * call, tmo, call);
          iph = ph;
          if (ph == 0u) STAMP(1);
          if (ph == 1u) STAMP(11);
          if (ph == 2u) STAMP(12);
        }
        u32 so, sc, dof; unsigned char* cnt;
        auto place = [&](u32 z) {
          const u32 q = z & 3u;
          if (ph == 1u) {
            const u32 t = (z - NUA) >> 2;
            so  = OFF_WT + ((par * NR + (u32)me) * NT + t) * ITEM + q * UNIT;
            sc  = OFF_WC + (((par * NR + (u32)me) * NT + t) * 16u + q * 4u) * RBW;
            dof = OFF_WL + ((u32)p * NT + t) * ITEM + q * UNIT;
            cnt = my + OFF_CNTW + t * 128u;
          } else {
            const u32 it = x_item((ph == 2u ? z - 64u : z) >> 2, (u32)p);
            so  = OFF_XT + (par * NXI + it) * ITEM + q * UNIT;
            sc  = OFF_XC + ((par * NXI + it) * 16u + q * 4u) * RBX;
            dof = OFF_XL + ((u32)p * NXI + it) * ITEM + q * UNIT;
            cnt = my + OFF_CNTX + it * 128u;
          }
        };
        place(u);
        if (ok) {
          v3u inf = {0u, 0u, 0u};
          fetch(sc, ph == 1u ? tlw : tlx);
          WAIT_VM0;
          #pragma unroll 1
          for (u32 r = 0;; r = (r + 1u) & 3u) {
            WAIT_VM4;
            const u32 bph = ph;
            *(volatile lds_v4u*)(ta + l16) = a3;
            asm volatile("" ::: "memory");
            const v2u t3 = *(volatile __attribute__((address_space(3))) v2u*)(ta + (l16 >> 1));
            const u32 escw = *(volatile lds_u32*)(ta + (bph == 1u ? RBW - 3200u : RBX - 3200u) + (u32)(lane & 31) * 4u);
            const u32 pk[14] = {a0[0], a0[1], a0[2], a0[3], a1[0], a1[1], a1[2], a1[3], a2[0], a2[1], a2[2], a2[3], t3[0], t3[1]};
            const u32 ro = r * 2048u + l16, bso = so + ro, bdof = dof + ro;
            asm volatile("" : "+v"(a0), "+v"(a1), "+v"(a2), "+v"(a3));
            if (r == 1u) {
              un = __builtin_amdgcn_readfirstlane(tk) - ubase;
              if (pend != nullptr && lane == 0) __hip_atomic_fetch_add((u32*)pend, 1u, __ATOMIC_RELAXED, __HIP_MEMORY_SCOPE_AGENT);
              pend = cnt;
            }
            bool last = false;
            u32 tl = bph == 1u ? tlw : tlx;
            sc += bph == 1u ? RBW : RBX;
            if (r == 3u) {
              if (un < (u32)NUIN) {
                const u32 nph = un < (u32)NUA ? 0u : un < (u32)NUW ? 1u : 2u;
                if (nph != ph) {
                  const u32 i0 = __builtin_amdgcn_readfirstlane(inf[0]), i1 = __builtin_amdgcn_readfirstlane(inf[1]), i2 = __builtin_amdgcn_readfirstlane(inf[2]);
                  last = nph == 1u ? (int)(i1 - (u32)NT * call) < 0 : ((int)(i2 - (u32)NXI * call) < 0 || (int)(i0 - (u32)NUX1 * call) < 0);
                }
                if (!last) {
                  USTAMP(bid % 7, u);
                  u = un; ph = nph; iph = nph; place(u);
                  tl = nph == 1u ? tlw : tlx;
                }
              } else {
                last = true;
                eval_flags();
                pre = un < (u32)NUBIG && flok && pcok;
                sc = OFF_OUT + ((par * NR + (u32)me) * 64u) * ITEM + (un - NUIN) * UNIT;
                tl = 3072u + l16;
              }
            }
            if (!last || pre) fetch(sc, tl);
            if (r == 0u && lane == 0) tk = __hip_atomic_fetch_add(uctr, 1u, __ATOMIC_RELAXED, __HIP_MEMORY_SCOPE_AGENT);
            if (r == 2u) {
              inf = __builtin_amdgcn_raw_buffer_load_b96(rs_my, (int)(OFF_INFLAG + (u32)p * 128u), 0, AUX_SYS);
              out_flags(un < (u32)NUIN ? (u32)NUIN : un);
            }
            v4u o[4];
            if (__builtin_amdgcn_readlane(escw, 31) >> 24) {
              if (bph == 0u && ok) ok = wait_ge(my + OFF_INFLAG + p * 128 + 8u, (u32)NXI * call, tmo, call);
              #pragma unroll
              for (int j = 0; j < 4; ++j)
                o[j] = __builtin_amdgcn_raw_buffer_load_b128(rs_rp, (int)(bso + (u32)((j >> 1) * 8192 + (j & 1) * 1024)), 0, AUX_SYS);
              WAIT_VM0;
            } else if (bph == 1u) {
              unpack_block<true>(pk, escw, (lds_u8*)smem + (u32)tid * 80u, o);
            } else {
              unpack_block<false>(pk, escw, (lds_u8*)smem + (u32)tid * 80u, o);
            }
            #pragma unroll
            for (int j = 0; j < 4; ++j)
              __builtin_amdgcn_raw_buffer_store_b128(o[j], rs_my, (int)(bdof + (u32)((j >> 1) * 8192 + (j & 1) * 1024)), 0, AUX_AGENT);
            if (last) break;
          }
        } else {
          if (lane == 0) tk = __hip_atomic_fetch_add(uctr, 1u, __ATOMIC_RELAXED, __HIP_MEMORY_SCOPE_AGENT);
          un = __builtin_amdgcn_readfirstlane(tk) - ubase;
        }
      }
      USTAMP(bid % 7, u);
      u = un;
    }
    const u32 obase = OFF_OUT + ((par * NR + (u32)me) * 64u) * ITEM;
    const u32 dbase = (u32)(lane >> 5) * (u32)(NN * 4) + (u32)(p * NC * 4) + (u32)(lane & 31) * 16u;
    auto blk0 = [&](u32 z) { return z < (u32)NUBIG ? (z - NUIN) * 4u : z - NUBIG + (u32)(NSL * NT - TAILT) * 16u; };
    auto dofs = [&](u32 b) { return dbase + ((b >> 8) * TM + (b & 15u) * 8u) * (u32)(NN * 4) + ((b >> 4) & 15u) * (u32)(TN * 4); };
    v4u c0, c1, c2, c3;
    auto fetch_own = [&](u32 d) {
      c0 = __builtin_amdgcn_raw_buffer_load_b128(rs_out, (int)d, 0, AUX_PLD);
      c1 = __builtin_amdgcn_raw_buffer_load_b128(rs_out, (int)(d + 2u * (u32)(NN * 4)), 0, AUX_PLD);
      c2 = __builtin_amdgcn_raw_buffer_load_b128(rs_out, (int)(d + 4u * (u32)(NN * 4)), 0, AUX_PLD);
      c3 = __builtin_amdgcn_raw_buffer_load_b128(rs_out, (int)(d + 6u * (u32)(NN * 4)), 0, AUX_PLD);
    };
    while (u < (u32)NUNIT) {
      u32 b = blk0(u);
      if (ok && !(flok && pcok)) {
        if (pend != nullptr) {
          WAIT_VM(0);
          if (lane == 0) __hip_atomic_fetch_add((u32*)pend, 1u, __ATOMIC_RELAXED, __HIP_MEMORY_SCOPE_AGENT);
          pend = nullptr;
        }
        if (!pcok)
          ok = u < (u32)NUBIG ? wait_ge(my + pc_base + ((b >> 4) & 15u) * 128u, 4u * call - 3u + (b >> 8), tmo, call)
                              : wait_flags(my + pc_base + (u32)(NT - TAILT) * 128u, TAILT, lane, 4u * call, tmo, call);
        if (ok && !flok)
          ok = u < (u32)NUBIG ? wait_ge(my + fl_base + (b >> 4) * 128u, call, tmo, call)
                              : wait_flags(my + fl_base + (u32)(NSL * NT - TAILT) * 128u, TAILT, lane, call, tmo, call);
        flok = true; pcok = true;
      }
      if (!ok) {
        if (lane == 0) tk = __hip_atomic_fetch_add(uctr, 1u, __ATOMIC_RELAXED, __HIP_MEMORY_SCOPE_AGENT);
        const v4u nanv = {0x7fc00000u, 0x7fc00000u, 0x7fc00000u, 0x7fc00000u};
        for (u32 r = 0; r < (u < (u32)NUBIG ? 16u : 4u); ++r)
          __builtin_amdgcn_raw_buffer_store_b128(nanv, rs_out, (int)(dofs(b) + r * 2u * (u32)(NN * 4)), 0, 0);
        u = __builtin_amdgcn_readfirstlane(tk) - ubase;
        continue;
      }
      if (!pre) fetch(obase + b * 4096u, 3072u + l16);
      fetch_own(dofs(b));
      if (lane == 0) tk = __hip_atomic_fetch_add(uctr, 1u, __ATOMIC_RELAXED, __HIP_MEMORY_SCOPE_AGENT);
      pre = false;
      WAIT_VM0;
      if (pend != nullptr && lane == 0) __hip_atomic_fetch_add((u32*)pend, 1u, __ATOMIC_RELAXED, __HIP_MEMORY_SCOPE_AGENT);
      pend = nullptr;
      #pragma unroll 1
      for (;;) {
        WAIT_VM4;
        const v4f o0 = __builtin_bit_cast(v4f, a0) + __builtin_bit_cast(v4f, c0), o1 = __builtin_bit_cast(v4f, a1) + __builtin_bit_cast(v4f, c1),
                  o2 = __builtin_bit_cast(v4f, a2) + __builtin_bit_cast(v4f, c2), o3 = __builtin_bit_cast(v4f, a3) + __builtin_bit_cast(v4f, c3);
        const u32 d = dofs(b), r = u < (u32)NUBIG ? b & 3u : 4u;
        if ((r & 3u) == 0u) un = __builtin_amdgcn_readfirstlane(tk) - ubase;
        bool go = true;
        ++b;
        if (r >= 3u) {
          if (r == 3u) eval_flags();
          go = un < (u32)NUNIT && flok && pcok;
          USTAMP(bid % 7, u);
          u = un; b = blk0(u);
        }
        if (go) {
          fetch(obase + b * 4096u, 3072u + l16);
          fetch_own(dofs(b));
          if (r >= 3u && lane == 0) tk = __hip_atomic_fetch_add(uctr, 1u, __ATOMIC_RELAXED, __HIP_MEMORY_SCOPE_AGENT);
          if (r == 2u) out_flags(un);
        }
        __builtin_amdgcn_raw_buffer_store_b128(__builtin_bit_cast(v4u, o0), rs_out, (int)d, 0, AUX_OUT);
        __builtin_amdgcn_raw_buffer_store_b128(__builtin_bit_cast(v4u, o1), rs_out, (int)(d + 2u * (u32)(NN * 4)), 0, AUX_OUT);
        __builtin_amdgcn_raw_buffer_store_b128(__builtin_bit_cast(v4u, o2), rs_out, (int)(d + 4u * (u32)(NN * 4)), 0, AUX_OUT);
        __builtin_amdgcn_raw_buffer_store_b128(__builtin_bit_cast(v4u, o3), rs_out, (int)(d + 6u * (u32)(NN * 4)), 0, AUX_OUT);
        if (!go) break;
      }
    }
    if (pend != nullptr) {
      WAIT_VM(0);
      if (lane == 0) __hip_atomic_fetch_add((u32*)pend, 1u, __ATOMIC_RELAXED, __HIP_MEMORY_SCOPE_AGENT);
    }
    STAMP(10);
    return;
  }

  const int g = bid - NPW;
  u32 call;
  const float scale = P.sx[0] * P.sw[0];
  const int wm = wave & 1, wn = wave >> 1;
  float* const img = (float*)smem;
  const int r0 = wave * 2 + (lane >> 5), c4 = (lane & 31) * 4;
  auto put_image = [&](const v16f (&acc)[2][2], bool ok) {
    const float nanf_ = __builtin_nanf("");
    #pragma unroll
    for (int i = 0; i < 2; ++i)
      #pragma unroll
      for (int j = 0; j < 2; ++j)
        #pragma unroll
        for (int q = 0; q < 4; ++q) {
          v4f o;
          #pragma unroll
          for (int e = 0; e < 4; ++e) o[e] = ok ? acc[i][j][4 * q + e] * scale : nanf_;
          const int row = wm * 64 + j * 32 + (lane & 31);
          const int col = wn * 64 + i * 32 + 8 * q + 4 * (lane >> 5);
          *(v4f*)(img + row * EPI_LD + col) = o;
        }
  };
  auto put_packed = [&](auto wtag, const v4u (&v)[16], u32 base, u32 rawbase) {
    constexpr bool W = decltype(wtag)::value;
    constexpr u32 RB = W ? RBW : RBX;
    lds_u32* const area = (lds_u32*)(smem + 65536) + wave * 128;
    const lds_u8* const raw = (const lds_u8*)smem + (W ? SM_W : 0) + (u32)(wave * 128 + (lane ^ ((lane >> 3) & 3))) * 16u;
    area[lane] = 0u; area[64 + lane] = 0u;
    asm volatile("" ::: "memory");
    #pragma unroll 1
    for (int a = 0; a < 4; ++a) {
      v4u w[4];
      #pragma unroll
      for (int k = 0; k < 4; ++k) w[k] = a == 0 ? v[k] : a == 1 ? v[4 + k] : a == 2 ? v[8 + k] : v[12 + k];
      u32 pk[14], escw;
      const bool rawb = pack_block<W, 6, 8192u>(w, raw + a * 16384, pk, area + a * 32, lane, escw);
      if (W && rawb) {
        #pragma unroll
        for (int k = 0; k < 4; ++k)
          __builtin_amdgcn_raw_buffer_store_b128(w[k], rs_my, (int)(rawbase + (u32)(a * 16384 + (k >> 1) * 8192 + (k & 1) * 1024 + (wave * 128 + lane) * 16)), 0, AUX_SYS);
      }
      const u32 o = base + (u32)(a * 4 + wave) * RB;
      #pragma unroll
      for (int k = 0; k < 3; ++k)
        __builtin_amdgcn_raw_buffer_store_b128((v4u){pk[4 * k], pk[4 * k + 1], pk[4 * k + 2], pk[4 * k + 3]}, rs_my, (int)(o + (u32)(k * 1024 + lane * 16)), 0, AUX_SYS);
      if (!W) __builtin_amdgcn_raw_buffer_store_b64((v2u){pk[12], pk[13]}, rs_my, (int)(o + 3072u + (u32)lane * 8u), 0, AUX_SYS);
      if (lane < 32) __builtin_amdgcn_raw_buffer_store_b32(escw, rs_my, (int)(o + RB - 128u + (u32)lane * 4u), 0, AUX_SYS);
    }
  };
  if (g < NXI) {
    const int s = g >> 3, m = g & 7;
    const unsigned char* src = P.x + (size_t)(m * MR + s * TM + wave * 32 + (lane & 31)) * KS + 32 * (lane >> 5);
    v4u v[16];
    #pragma unroll
    for (int j = 0; j < 16; ++j) v[j] = *(const v4u*)(src + (j >> 1) * 64 + (j & 1) * 16);
    __syncthreads();
    call = sm_word[0];
    const u32 dof = OFF_XT + ((call & 1u) * NXI + (u32)g) * ITEM + (u32)(wave * 2 * 64 + lane) * 16u;
    #pragma unroll
    for (int j = 0; j < 16; ++j)
      __builtin_amdgcn_raw_buffer_store_b128(v[j], rs_my, (int)(dof + (u32)((j >> 1) * 8192 + (j & 1) * 1024)), 0, AUX_SYS);
    if (s >= 2) {
      #pragma unroll
      for (int j = 0; j < 16; ++j) {
        const int ch = (j >> 1) * 512 + wave * 128 + (j & 1) * 64 + lane;
        *(v4u*)(smem + (ch ^ ((ch >> 3) & 3)) * 16) = v[j];
      }
      put_packed(std::false_type{}, v, OFF_XC + ((call & 1u) * NXI + (u32)g) * 16u * RBX, 0u);
    }
    WAIT_VM(0);
    __syncthreads();
    if (tid < NR && tid < P.nranks) {
      if (tid == me) __hip_atomic_fetch_add((u32*)(my + OFF_CNTX + g * 128), 4u, __ATOMIC_RELAXED, __HIP_MEMORY_SCOPE_AGENT);
      else __hip_atomic_fetch_add((u32*)(P.ws[tid] + OFF_INFLAG + me * 128 + 8), 1u, __ATOMIC_RELAXED, __HIP_MEMORY_SCOPE_SYSTEM);
    }
  } else if (g >= GW0) {
    const int j = (g - GW0) >> 4, t = (g - GW0) & 15;
    const int q = lane & 31;
    const unsigned char* src = P.w + (size_t)(j * NC + t * TN + q * 4);
    u32 r[4][16];
    #pragma unroll
    for (int a = 0; a < 4; ++a) {
      const int kg = a * 8 + wave * 2 + (lane >> 5);
      #pragma unroll
      for (int i = 0; i < 16; ++i) r[a][i] = *(const u32*)(src + (size_t)(kg * 16 + i) * NN);
    }
    #pragma unroll
    for (int a = 0; a < 4; ++a) {
      const int kg = a * 8 + wave * 2 + (lane >> 5);
      u32 o[4][4];
      #pragma unroll
      for (int b = 0; b < 4; ++b) {
        const u32 r0 = r[a][4 * b], r1 = r[a][4 * b + 1], r2 = r[a][4 * b + 2], r3 = r[a][4 * b + 3];
        const u32 t0 = __builtin_amdgcn_perm(r1, r0, 0x05010400u), t1 = __builtin_amdgcn_perm(r1, r0, 0x07030602u);
        const u32 t2 = __builtin_amdgcn_perm(r3, r2, 0x05010400u), t3 = __builtin_amdgcn_perm(r3, r2, 0x07030602u);
        o[0][b] = __builtin_amdgcn_perm(t2, t0, 0x05040100u);
        o[1][b] = __builtin_amdgcn_perm(t2, t0, 0x07060302u);
        o[2][b] = __builtin_amdgcn_perm(t3, t1, 0x05040100u);
        o[3][b] = __builtin_amdgcn_perm(t3, t1, 0x07060302u);
      }
      const int kb = kg >> 2, half = (kg >> 1) & 1, part = kg & 1, blk = q >> 3;
      const int c0 = ((kb * 4 + blk) * 2 + part) * 64 + half * 32 + (q & 7) * 4;
      #pragma unroll
      for (int c = 0; c < 4; ++c) {
        const int ch = c0 + c;
        const v4u ov = {o[c][0], o[c][1], o[c][2], o[c][3]};
        *(v4u*)(smem + SM_W + (ch ^ ((ch >> 3) & 3)) * 16) = ov;
      }
    }
    __syncthreads();
    STAMP(11);
    call = sm_word[0];
    const u32 dof = OFF_WT + (((call & 1u) * NR + (u32)j) * NT + (u32)t) * ITEM;
    {
      v4u v[16];
      #pragma unroll
      for (int i = 0; i < 16; ++i) {
        const int ch = (i >> 1) * 512 + wave * 128 + (i & 1) * 64 + lane;
        v[i] = *(const v4u*)(smem + SM_W + (ch ^ ((ch >> 3) & 3)) * 16);
        if (j == me) __builtin_amdgcn_raw_buffer_store_b128(v[i], rs_my, (int)(dof + (u32)ch * 16u), 0, AUX_SYS);
      }
      if (j != me) put_packed(std::true_type{}, v, OFF_WC + (((call & 1u) * NR + (u32)j) * NT + (u32)t) * 16u * RBW, dof);
    }
    STAMP(12);
    WAIT_VM(0);
    __syncthreads();
    if (tid == 0 && j < P.nranks) {
      if (j == me) __hip_atomic_fetch_add((u32*)(my + OFF_CNTW + t * 128), 4u, __ATOMIC_RELAXED, __HIP_MEMORY_SCOPE_AGENT);
      else __hip_atomic_fetch_add((u32*)(P.ws[j] + OFF_INFLAG + me * 128 + 4), 1u, __ATOMIC_RELAXED, __HIP_MEMORY_SCOPE_SYSTEM);
    }
  } else if (g < NXI + 7 * NPUSH) {
    const int p  = (me + 1 + (g - NXI) % 7) & (NR - 1), pw = (g - NXI) / 7;
    const int pm = (me + 1 + pw) & (NR - 1);
    const unsigned char* src = P.x + (size_t)(pm * MR + wave * 32 + (lane & 31)) * KS + 32 * (lane >> 5);
    v4u v[16];
    #pragma unroll
    for (int j = 0; j < 16; ++j) v[j] = *(const v4u*)(src + (j >> 1) * 64 + (j & 1) * 16);
    __syncthreads();
    call = sm_word[0];
    if (p < P.nranks) {
      const __amdgpu_buffer_rsrc_t rs_rp = mk_rsrc(P.ws[p], WS_USED);
      const u32 dof = OFF_XP + (((call & 1u) * NR + (u32)me) * NR + (u32)pw) * ITEM + (u32)(wave * 2 * 64 + lane) * 16u;
      #pragma unroll
      for (int j = 0; j < 16; ++j)
        __builtin_amdgcn_raw_buffer_store_b128(v[j], rs_rp, (int)(dof + (u32)((j >> 1) * 8192 + (j & 1) * 1024)), 0, AUX_SYS);
      WAIT_VM(0);
      if (lane == 0) __hip_atomic_fetch_add((u32*)(P.ws[p] + OFF_CNTP + ((call & 1u) * NR + (u32)pm) * 128u), 1u, __ATOMIC_RELAXED, __HIP_MEMORY_SCOPE_SYSTEM);
      USTAMP((g - NXI) % 7, 1000 + pw * 4 + wave);
    }
  } else {
    __syncthreads();
    call = sm_word[0];
  }
  STAMP(1);
  const u32 par = call & 1u;

  const int grp = bid & 7;
  const int gfirst = NPW + ((grp - NPW) & 7);
  const int gcnt = (GRID - 1 - gfirst) / 8 + 1;
  int ps = g >= GW0 && (g - GW0) >> 4 != me ? 0 : NSL;
  const int T0 = (bid - gfirst) >> 3;
  for (int T = T0;; T += gcnt) {
    const int s = T >> 4, t = ((T >> 2) & 3) * 4 + (grp >> 1), m = (grp & 1) * 4 + (T & 3);
    const int xi = s * 8 + m;
    const u32 np = s != 0 ? 0u : (u32)NPUSH - (((u32)(m - me - 1) & (NR - 1)) < (u32)NPUSH ? 1u : 0u);
    const u32 tgw = CNT_PER_CALL * call, tgx = 4u * ((m == me ? NR : NR - 1) - np) * call, tgp = 4u * np * ((call + 1u) >> 1);
    if (ps < NSL) {
      const int j = (g - GW0) >> 4, jt = (g - GW0) & 15;
      const unsigned char* xs = P.x + (size_t)(me * MR + wm * 64 + (lane & 31)) * KS + 32 * (lane >> 5);
      v4u xf[2][8][2];
      auto xload = [&](int z) {
        #pragma unroll
        for (int kb = 0; kb < 8; ++kb)
          #pragma unroll
          for (int b = 0; b < 2; ++b)
            #pragma unroll
            for (int h = 0; h < 2; ++h) xf[b][kb][h] = *(const v4u*)(xs + (size_t)(z * TM + b * 32) * KS + kb * 64 + h * 16);
      };
      xload(ps);
      const __amdgpu_buffer_rsrc_t rs_out = mk_rsrc(P.out, (u32)MR * NN * 4u);
      const unsigned char* const fw = smem + SM_W + (wn * 256 + (lane ^ ((lane >> 3) & 3))) * 16;
      u32* const pcnt = (u32*)(my + OFF_PCNT + (g - GW0) * 128);
      const int ps0 = ps;
      #pragma unroll 1
      for (;;) {
        v16f acc[2][2];
        #pragma unroll
        for (int i = 0; i < 2; ++i)
          #pragma unroll
          for (int k = 0; k < 2; ++k)
            #pragma unroll
            for (int e = 0; e < 16; ++e) acc[i][k][e] = 0.f;
        #pragma unroll
        for (int kb = 0; kb < 8; ++kb) {
          v8i X[2], W[2];
          #pragma unroll
          for (int b = 0; b < 2; ++b) {
            const v4u w0 = *(const v4u*)(fw + kb * 8192 + b * 2048), w1 = *(const v4u*)(fw + kb * 8192 + b * 2048 + 1024);
            const v4u x0 = xf[b][kb][0], x1 = xf[b][kb][1];
            X[b] = (v8i){(int)x0[0], (int)x0[1], (int)x0[2], (int)x0[3], (int)x1[0], (int)x1[1], (int)x1[2], (int)x1[3]};
            W[b] = (v8i){(int)w0[0], (int)w0[1], (int)w0[2], (int)w0[3], (int)w1[0], (int)w1[1], (int)w1[2], (int)w1[3]};
          }
          #pragma unroll
          for (int i = 0; i < 2; ++i)
            #pragma unroll
            for (int k = 0; k < 2; ++k)
              acc[i][k] = __builtin_amdgcn_mfma_scale_f32_32x32x64_f8f6f4(W[i], X[k], acc[i][k], 1, 0, 0, 0x7F7F7F7F, 0, 0x7F7F7F7F);
        }
        SCHED_FENCE;
        WAIT_VM(16);
        if (ps < NSL - 1) xload(ps + 1);
        WAIT_LGKM0; RAW_BAR;
        if (tid == 0 && ps >= ps0 + 2) __hip_atomic_store(pcnt, 4u * call - 5u + (u32)ps, __ATOMIC_RELAXED, __HIP_MEMORY_SCOPE_AGENT);
        put_image(acc, true);
        WAIT_LGKM0; RAW_BAR;
        const bool chk = T == T0 && ps >= 1 && ps < NSL - 1;
        u32 fw_ = 0, fx_ = 0, fp_ = 0;
        if (chk && tid == 0) {
          fw_ = ld_sys(my + OFF_CNTW + t * 128); fx_ = ld_sys(my + OFF_CNTX + xi * 128); fp_ = ld_sys(my + OFF_CNTP + (par * NR + (u32)m) * 128u);
        }
        const u32 odof = ((u32)(ps * TM + r0) * (u32)NN + (u32)(j * NC + jt * TN + c4)) * 4u;
        #pragma unroll
        for (int i = 0; i < 16; ++i) {
          const v4f o = *(const v4f*)(img + (r0 + i * 8) * EPI_LD + c4);
          __builtin_amdgcn_raw_buffer_store_b128(__builtin_bit_cast(v4u, o), rs_out, (int)(odof + (u32)(i * 8) * (u32)(NN * 4)), 0, AUX_PST);
        }
        if (++ps == NSL) break;
        if (chk) {
          if (tid == 0) sm_word[2] = (int)(fw_ - tgw) >= 0 && (int)(fx_ - tgx) >= 0 && (int)(fp_ - tgp) >= 0 ? 1u : 0u;
          WAIT_LGKM0; RAW_BAR;
          if (__builtin_amdgcn_readfirstlane(sm_word[2]) != 0u) break;
        }
      }
      WAIT_VM(0);
      __syncthreads();
      if (tid == 0) __hip_atomic_store(pcnt, 4u * call - 4u + (u32)ps, __ATOMIC_RELAXED, __HIP_MEMORY_SCOPE_AGENT);
      if (ps0 == 0) STAMP(13);
      if (ps == NSL) STAMP(14);
    }
    if (T >= NTILE / 8) break;
    __syncthreads();
    if (tid == 0) {
      bool ok = wait_ge(my + OFF_CNTW + t * 128, tgw, tmo, call);
      if (ok) ok = wait_ge(my + OFF_CNTX + xi * 128, tgx, tmo, call);
      if (ok) ok = wait_ge(my + OFF_CNTP + (par * NR + (u32)m) * 128u, tgp, tmo, call);
      if (s == 0) __builtin_amdgcn_fence(__ATOMIC_ACQUIRE, "");
      else        __builtin_amdgcn_fence(__ATOMIC_ACQUIRE, "agent");
      WAIT_VM(0);
      if (ok) ok = (ld_agent(tmo) != call);
      sm_word[1] = ok ? 1u : 0u;
    }
    __syncthreads();
    const bool ok = sm_word[1] != 0u;
    STAMP(2 + 3 * (T / gcnt));

    v16f acc[2][2];
    #pragma unroll
    for (int i = 0; i < 2; ++i)
      #pragma unroll
      for (int j = 0; j < 2; ++j)
        #pragma unroll
        for (int e = 0; e < 16; ++e) acc[i][j][e] = 0.f;

    if (ok) {
      const unsigned char* xo = my + OFF_XT + (par * NXI + (u32)xi) * ITEM + tid * 16;
      const unsigned char* xl = my + OFF_XL + (u32)xi * ITEM + tid * 16;
      const unsigned char* xp = my + OFF_XP + par * NR * NR * ITEM + tid * 16;
      const unsigned char* wo = my + OFF_WT + ((par * NR + (u32)me) * NT + (u32)t) * ITEM + tid * 16;
      const int skip = m == me ? NR : m;
      const int nst = m == me ? 64 : 56;
      const unsigned char* wl = my + OFF_WL + (u32)t * ITEM + tid * 16;
      unsigned char* const ldst = smem + wave * 1024;
      auto issue = [&](int ks) {
        int sr = ks >> 3; sr += sr >= skip ? 1 : 0;
        const int kb = ks & 7;
        const u32 d = (u32)(m - sr - 1) & (NR - 1);
        const unsigned char* xa = (sr == me ? xo : (s == 0 && d < (u32)NPUSH) ? xp + (size_t)(((u32)sr * NR + d) * ITEM)
                                                                            : xl + (size_t)sr * (NXI * ITEM)) + kb * 8192;
        const unsigned char* wa = (sr == me ? wo : wl + (size_t)sr * (NT * ITEM)) + kb * 8192;
        unsigned char* l = ldst + (ks & 3) * 16384;
        __builtin_amdgcn_global_load_lds((const __attribute__((address_space(1))) void*)(xa),        (__attribute__((address_space(3))) void*)(l),         16, 0, 0);
        __builtin_amdgcn_global_load_lds((const __attribute__((address_space(1))) void*)(xa + 4096), (__attribute__((address_space(3))) void*)(l + 4096),  16, 0, 0);
        __builtin_amdgcn_global_load_lds((const __attribute__((address_space(1))) void*)(wa),        (__attribute__((address_space(3))) void*)(l + 8192),  16, 0, 0);
        __builtin_amdgcn_global_load_lds((const __attribute__((address_space(1))) void*)(wa + 4096), (__attribute__((address_space(3))) void*)(l + 12288), 16, 0, 0);
      };
      const unsigned char* const fx = smem + (wm * 2) * 2048 + lane * 16;
      const unsigned char* const fw = smem + 8192 + (wn * 2) * 2048 + lane * 16;
      auto rdfrag = [&](int ks, v8i (&X)[2], v8i (&W)[2]) {
        const int so = (ks & 3) * 16384;
        #pragma unroll
        for (int b = 0; b < 2; ++b) {
          const v4u x0 = *(const v4u*)(fx + so + b * 2048), x1 = *(const v4u*)(fx + so + b * 2048 + 1024);
          const v4u w0 = *(const v4u*)(fw + so + b * 2048), w1 = *(const v4u*)(fw + so + b * 2048 + 1024);
          X[b] = (v8i){(int)x0[0], (int)x0[1], (int)x0[2], (int)x0[3], (int)x1[0], (int)x1[1], (int)x1[2], (int)x1[3]};
          W[b] = (v8i){(int)w0[0], (int)w0[1], (int)w0[2], (int)w0[3], (int)w1[0], (int)w1[1], (int)w1[2], (int)w1[3]};
        }
      };
      auto mma = [&](v8i (&X)[2], v8i (&W)[2]) {
        #pragma unroll
        for (int i = 0; i < 2; ++i)
          #pragma unroll
          for (int j = 0; j < 2; ++j)
            acc[i][j] = __builtin_amdgcn_mfma_scale_f32_32x32x64_f8f6f4(W[i], X[j], acc[i][j], 1, 0, 0, 0x7F7F7F7F, 0, 0x7F7F7F7F);
      };
      v8i XA[2], WA[2], XB[2], WB[2];
      issue(0); issue(1); issue(2);
      WAIT_VM(8); RAW_BAR;
      issue(3);
      rdfrag(0, XA, WA);
      for (int ks = 0; ks < nst - 4; ks += 2) {
        WAIT_VM(8); WAIT_LGKM0; RAW_BAR;
        issue(ks + 4);
        rdfrag(ks + 1, XB, WB);
        SCHED_FENCE;
        mma(XA, WA);
        SCHED_FENCE;
        WAIT_VM(8); WAIT_LGKM0; RAW_BAR;
        issue(ks + 5);
        rdfrag(ks + 2, XA, WA);
        SCHED_FENCE;
        mma(XB, WB);
        SCHED_FENCE;
      }
      WAIT_VM(8); WAIT_LGKM0; RAW_BAR;
      rdfrag(61, XB, WB); SCHED_FENCE; mma(XA, WA); SCHED_FENCE;
      WAIT_VM(4); WAIT_LGKM0; RAW_BAR;
      rdfrag(62, XA, WA); SCHED_FENCE; mma(XB, WB); SCHED_FENCE;
      WAIT_VM(0); WAIT_LGKM0; RAW_BAR;
      rdfrag(63, XB, WB); SCHED_FENCE; mma(XA, WA); SCHED_FENCE;
      mma(XB, WB);
    }
    STAMP(3 + 3 * (T / gcnt));
    __syncthreads();
    {
      put_image(acc, ok);
      __syncthreads();
      if (m == me) {
        float* dst = P.out + (size_t)(s * TM + r0) * NN + me * NC + t * TN + c4;
        #pragma unroll
        for (int i = 0; i < 16; ++i) {
          const v4f o = *(const v4f*)(img + (r0 + i * 8) * EPI_LD + c4);
          *(v4f*)(dst + (size_t)(i * 8) * NN) = o;
        }
      } else {
        const u32 dof = OFF_OUT + ((par * NR + (u32)m) * 64u + (u32)(s * NT + t)) * ITEM + (u32)(r0 * 512 + c4 * 4);
        #pragma unroll
        for (int i = 0; i < 16; ++i) {
          const v4f o = *(const v4f*)(img + (r0 + i * 8) * EPI_LD + c4);
          __builtin_amdgcn_raw_buffer_store_b128(__builtin_bit_cast(v4u, o), rs_my, (int)(dof + (u32)(i * 8 * 512)), 0, AUX_SYS);
        }
        WAIT_VM(0);
        __syncthreads();
        if (tid == 0 && m < P.nranks)
          __hip_atomic_store((u32*)(P.ws[m] + OFF_OFLAG + (u32)(me * 64 + s * NT + t) * 128u), call, __ATOMIC_RELAXED, __HIP_MEMORY_SCOPE_SYSTEM);
      }
    }
    STAMP(4 + 3 * (T / gcnt));
  }
}

extern "C" void kernel_launch(int rank, int nranks, ncclComm_t comm,
                              hipStream_t stream,
                              void* const* d_in, const int* in_sizes, int n_in,
                              void* d_out, int out_size,
                              const int* in_dtypes, int out_dtype,
                              void* const* d_out_all,
                              void* const* d_ws_all, size_t ws_bytes) {
  (void)comm; (void)in_sizes; (void)in_dtypes; (void)out_dtype; (void)d_out_all;
  if (nranks != NR || n_in < 4 || ws_bytes < (size_t)WS_USED || out_size != MR * NN || rank < 0 || rank >= NR) {
    fprintf(stderr, "unexpected configuration\n");
    return;
  }
  Params P;
  P.x  = (const unsigned char*)d_in[0];
  P.w  = (const unsigned char*)d_in[1];
  P.sx = (const float*)d_in[2];
  P.sw = (const float*)d_in[3];
  P.out = (float*)d_out;
  for (int p = 0; p < NR; ++p) P.ws[p] = (unsigned char*)d_ws_all[p];
  P.rank = rank;
  P.nranks = nranks;
  fused_gemm_rs<<<GRID, TPB, 0, stream>>>(P);
  HIP_CHECK(hipGetLastError());
}
```

```cpp
#include <hip/hip_runtime.h>
#include <rccl/rccl.h>
#include <cstdio>
#include <cstdint>

#define HIP_CHECK(expr) do { hipError_t _e = (expr); if (_e != hipSuccess) { \
    fprintf(stderr, "HIP error %s at %s:%d\n", hipGetErrorString(_e), __FILE__, __LINE__); \
    return; } } while (0)

typedef unsigned u32;
typedef unsigned long long u64;
typedef int      v8i  __attribute__((ext_vector_type(8)));
typedef unsigned v4u  __attribute__((ext_vector_type(4)));
typedef unsigned v3u  __attribute__((ext_vector_type(3)));
typedef float    v16f __attribute__((ext_vector_type(16)));

constexpr int NR    = 8;
constexpr int MM    = 4096;
constexpr int KK    = 4096;
constexpr int NN    = 16384;
constexpr int KS    = KK / NR;
constexpr int MR    = MM / NR;
constexpr int GRID  = 256;
constexpr int TPB   = 1024;
constexpr int NSTRIP = 256;
constexpr int G0     = 32;
constexpr int G1     = 224;
#define PW 5

#define DEL 16
#define ND 0
#define DLT 32
constexpr u32 NT = 1920u - ND;

constexpr u32 RCAP  = 3072;
constexpr u32 NMAX  = (RCAP - 384u) / 2u;
constexpr u32 BSZ   = 2816;
constexpr u32 STRIP = 4u * RCAP + 4u * BSZ;

constexpr u32 OFF_TICKET = 0;
constexpr u32 OFF_TMO    = 2048;
constexpr u32 OFF_LC     = 3072;
constexpr u32 OFF_CX     = 4096;
constexpr u32 OFF_FB     = 49152;
constexpr u32 OFF_FL     = 65536;
constexpr u32 OFF_AR     = 131072;
constexpr u32 OFF_XA     = 1u << 20;
constexpr u32 OFF_ENC    = OFF_XA + 2u * (u32)(MR * KK);
constexpr u32 OFF_RAW    = OFF_ENC + 2u * NSTRIP * STRIP;
constexpr u32 OFF_CP     = OFF_RAW + 2u * NSTRIP * 32768u;
constexpr u32 OFF_EB     = OFF_CP + 7u * NSTRIP * STRIP;
constexpr u32 WS_USED    = OFF_EB + 2u * 7u * G0 * 4u * BSZ;

#define DL 100000000ull
constexpr u64 DEADLINE_TICKS = DL;

constexpr u32 NS     = 4;
constexpr u32 CHS    = 1040;
constexpr u32 SLOT   = 8u * CHS;
constexpr u32 L_DEC  = 0;
constexpr u32 L_ENC  = 16384;
constexpr u32 L_B    = 16384;
constexpr u32 L_R    = L_B + 7u * BSZ;
constexpr u32 L_SLOT = L_R + 7u * 4u * RCAP;
constexpr u32 L_CTL  = L_SLOT + NS * SLOT;
constexpr u32 SM_TOTAL = L_CTL + 256u;
enum { C_CALL = 0, C_BAD = 1, C_XOK = 2, C_PUSH = 3, C_ENCD = 4, C_N = 8, C_RDY = 16, C_CONS = 24, C_ARR = 32 };

constexpr int AUX_SYS = 17;
#define POLLS 2
#define AUX_OUT 16

constexpr unsigned char FREQ[18 * 18] = {
    1,  1,  1,  1,  1,  1,  1,  1,  1,  1,  1,  1,  1,  1,  1,  1,  1,  1,
    1,  1,  1,  1,  1,  1,  1,  1,  1,  1,  1,  1,  1,  1,  1,  1,  1,  1,
    1,  1,  1,  1,  1,  1,  1,  1,  1,  1,  1,  1,  1,  1,  1,  1,  1,  1,
    1,  1,  1,  1,  1,  1,  1,  1,  1,  1,  1,  1,  1,  1,  1,  1,  1,  1,
    1,  1,  1,  1,  1,  1,  1,  1,  1,  1,  1,  1,  1,  1,  1,  1,  1,  1,
    1,  1,  1,  1,  1,  1,  1,  1,  1,  1,  1,  1,  1,  1,  1,  1,  1,  1,
    1,  1,  1,  1,  1,  1,  1,  1,  1,  1,  1,  1,  1,  1,  2,  2,  1,  1,
    1,  1,  1,  1,  1,  1,  1,  1,  1,  1,  1,  1,  1,  2,  4,  4,  1,  1,
    1,  1,  1,  1,  1,  1,  1,  1,  1,  1,  1,  1,  2,  4,  7,  7,  2,  1,
    1,  1,  1,  1,  1,  1,  1,  1,  1,  1,  1,  2,  5,  9, 15, 14,  3,  1,
    1,  1,  1,  1,  1,  1,  1,  1,  1,  1,  2,  5,  9, 18, 29, 29,  6,  1,
    1,  1,  1,  1,  1,  1,  1,  1,  1,  2,  5,  9, 18, 35, 58, 58, 12,  1,
    1,  1,  1,  1,  1,  1,  1,  1,  2,  5,  9, 18, 36, 70,115,114, 24,  1,
    1,  1,  1,  1,  1,  1,  1,  2,  4,  9, 18, 35, 70,133,220,218, 46,  1,
    1,  1,  1,  1,  1,  1,  2,  4,  7, 15, 29, 58,115,220,255,255, 76,  1,
    1,  1,  1,  1,  1,  1,  2,  4,  7, 14, 29, 58,114,217,255,255, 75,  1,
    1,  1,  1,  1,  1,  1,  1,  1,  2,  3,  6, 12, 24, 46, 76, 75, 16,  1,
    1,  1,  1,  1,  1,  1,  1,  1,  1,  1,  1,  1,  1,  1,  1,  1,  1,  1,
};
struct Tabs { u32 t[4096 + 4096]; };
constexpr Tabs make_tabs() {
  Tabs T{};
  u32 c = 0;
  for (u32 e2 = 0; e2 < 18; ++e2)
    for (u32 e1 = 0; e1 < 18; ++e1) {
      const u32 f = FREQ[e2 * 18 + e1];
      for (u32 i = 0; i < f; ++i) T.t[c + i] = f << 24 | i << 16 | e2 << 10 | e1 << 2;
      u32 s = 0;
      while ((2u << s) < f) ++s;
      u32* E = &T.t[4096 + (e2 * 32 + e1) * 4];
      E[0] = f == 1 ? 0xFFFFFFFFu : (u32)(((1ull << (32 + s)) + f - 1) / f);
      E[1] = f << 20;
      E[2] = (f == 1 ? c + 4095u : c) << 16 | s;
      E[3] = 4096u - f;
      c += f;
    }
  return T;
}
__device__ const Tabs g_tabs __attribute__((aligned(64))) = make_tabs();

#define STAMP(slot) do {} while (0)

struct Params {
  const unsigned char* x;
  const unsigned char* w;
  const float* sx;
  const float* sw;
  float* out;
  unsigned char* ws[NR];
  int rank;
  int nranks;
};

#define WAIT_VM(n)  asm volatile("s_waitcnt vmcnt(" #n ")" ::: "memory")
#define WAIT_LGKM0  do { __builtin_amdgcn_s_waitcnt(0xC07F); asm volatile("" ::: "memory"); } while (0)
#define SCHED_FENCE __builtin_amdgcn_sched_barrier(0)

typedef __attribute__((address_space(3))) unsigned char  lds_u8;
typedef __attribute__((address_space(3))) unsigned short lds_u16;
typedef __attribute__((address_space(3))) u32 lds_u32;
typedef __attribute__((address_space(3))) v4u lds_v4u;

__device__ __forceinline__ u32 ld_sys(const unsigned char* p) {
  return __hip_atomic_load((const u32*)p, __ATOMIC_RELAXED, __HIP_MEMORY_SCOPE_SYSTEM);
}
__device__ __forceinline__ u32 ld_sys1(const unsigned char* p) {
  return __builtin_amdgcn_readfirstlane(__hip_atomic_load((const u32*)p, __ATOMIC_RELAXED, __HIP_MEMORY_SCOPE_SYSTEM));
}
__device__ __forceinline__ u32 ld_agent(const unsigned char* p) {
  return __builtin_amdgcn_readfirstlane(__hip_atomic_load((const u32*)p, __ATOMIC_RELAXED, __HIP_MEMORY_SCOPE_AGENT));
}

__device__ __forceinline__ bool wait_ge(const unsigned char* flag, u32 target, unsigned char* tmo, u32 call) {
  if ((int)(ld_sys1(flag) - target) >= 0) return true;
  const u64 t0 = __builtin_amdgcn_s_memrealtime();
  for (;;) {
    if ((int)(ld_sys1(flag) - target) >= 0) return true;
    if (ld_agent(tmo) == call || __builtin_amdgcn_s_memrealtime() - t0 > DEADLINE_TICKS) {
      __hip_atomic_store((u32*)tmo, call, __ATOMIC_RELAXED, __HIP_MEMORY_SCOPE_AGENT);
      return false;
    }
    __builtin_amdgcn_s_sleep(4);
  }
}
__device__ __forceinline__ bool wait_flags(const unsigned char* base, int n, int lane, u32 target, unsigned char* tmo, u32 call) {
  const u64 t0 = __builtin_amdgcn_s_memrealtime();
  for (;;) {
    const u32 f = lane < n ? ld_sys(base + lane * 128) : target;
    if (__builtin_amdgcn_ballot_w64((int)(f - target) < 0) == 0ull) return true;
    if (ld_agent(tmo) == call || __builtin_amdgcn_s_memrealtime() - t0 > DEADLINE_TICKS) {
      __hip_atomic_store((u32*)tmo, call, __ATOMIC_RELAXED, __HIP_MEMORY_SCOPE_AGENT);
      return false;
    }
    __builtin_amdgcn_s_sleep(2);
  }
}
__device__ __forceinline__ void wait_lds(volatile lds_u32* ctl, int idx, u32 target, unsigned char* tmo, u32 call) {
  if (__builtin_amdgcn_readfirstlane(ctl[idx]) >= target) return;
  const u64 t0 = __builtin_amdgcn_s_memrealtime();
  for (;;) {
    __builtin_amdgcn_s_sleep(1);
    if (__builtin_amdgcn_readfirstlane(ctl[idx]) >= target) return;
    if (__builtin_amdgcn_readfirstlane(ctl[C_BAD]) != 0u) return;
    if (__builtin_amdgcn_s_memrealtime() - t0 > 2ull * DEADLINE_TICKS) {
      ctl[C_BAD] = 1u;
      __hip_atomic_store((u32*)tmo, call, __ATOMIC_RELAXED, __HIP_MEMORY_SCOPE_AGENT);
      return;
    }
  }
}

__device__ __forceinline__ __amdgpu_buffer_rsrc_t mk_rsrc(const void* p, u32 bytes) {
  return __builtin_amdgcn_make_buffer_rsrc((void*)p, (short)0, (int)bytes, 0x00020000);
}
__device__ __forceinline__ v4u mk_srd(const unsigned char* p, u32 bytes) {
  const u64 a = (u64)p;
  return (v4u){(u32)__builtin_amdgcn_readfirstlane((u32)a), (u32)__builtin_amdgcn_readfirstlane((u32)(a >> 32)) & 0xFFFFu,
               (u32)__builtin_amdgcn_readfirstlane(bytes), 0x00020000u};
}
__device__ __forceinline__ u32 bfi(u32 m, u32 a, u32 b) { return (a & m) | (b & ~m); }
__device__ __forceinline__ u32 ror(u32 a, u32 s) { return __builtin_amdgcn_alignbit(a, a, s); }

#define SDWA_B3 " dst_sel:DWORD dst_unused:UNUSED_PAD src0_sel:DWORD src1_sel:BYTE_3\n"
#define SDWA_B2 " dst_sel:DWORD dst_unused:UNUSED_PAD src0_sel:DWORD src1_sel:BYTE_2\n"
#define DEC_STEP(E, EN, FILL) \
  "s_waitcnt lgkmcnt(0)\n" \
  "v_mul_u32_u24_sdwa %[x], %[q], %[" E "]" SDWA_B3 \
  "v_add_u32_sdwa %[x], %[x], %[" E "]" SDWA_B2 \
  "v_cmp_gt_u32 vcc, 0x10000, %[x]\n" \
  "v_lshl_or_b32 %[t], %[x], 16, %[wn]\n" \
  FILL \
  "v_cndmask_b32 %[x], %[x], %[t], vcc\n" \
  "v_and_b32 %[t], 0xfff, %[x]\n" \
  "v_lshlrev_b32 %[t], 2, %[t]\n" \
  "ds_read_b32 %[" EN "], %[t]\n" \
  "v_mbcnt_lo_u32_b32 %[r], vcc_lo, 0\n" \
  "v_mbcnt_hi_u32_b32 %[r], vcc_hi, %[r]\n" \
  "s_bcnt1_i32_b64 %[n], vcc\n" \
  "v_lshl_add_u32 %[r], %[r], 1, %[sp]\n" \
  "s_mov_b64 exec, vcc\n" \
  "ds_read_u16 %[wn], %[r]\n" \
  "s_mov_b64 exec, -1\n" \
  "s_lshl1_add_u32 %[sp], %[n], %[sp]\n" \
  "v_lshrrev_b32 %[q], 12, %[x]\n"

__device__ __forceinline__ void load_unit(const unsigned char* w, u32 strip, u32 unit, int lane, u32 (&d)[32]) {
  const unsigned char* src = w + (size_t)(unit * 128u + (u32)(lane >> 4) * 32u) * NN + strip * 64u + (u32)(lane & 15) * 4u;
  #pragma unroll
  for (int i = 0; i < 32; ++i) d[i] = *(const u32*)(src + (size_t)i * NN);
}
__device__ __forceinline__ void turn_unit(u32 (&d)[32]) {
  u32 o[32];
  #pragma unroll
  for (int b = 0; b < 8; ++b) {
    const u32 r0 = d[4 * b], r1 = d[4 * b + 1], r2 = d[4 * b + 2], r3 = d[4 * b + 3];
    const u32 t0 = __builtin_amdgcn_perm(r1, r0, 0x05010400u), t1 = __builtin_amdgcn_perm(r1, r0, 0x07030602u);
    const u32 t2 = __builtin_amdgcn_perm(r3, r2, 0x05010400u), t3 = __builtin_amdgcn_perm(r3, r2, 0x07030602u);
    o[b]      = __builtin_amdgcn_perm(t2, t0, 0x05040100u);
    o[8 + b]  = __builtin_amdgcn_perm(t2, t0, 0x07060302u);
    o[16 + b] = __builtin_amdgcn_perm(t3, t1, 0x05040100u);
    o[24 + b] = __builtin_amdgcn_perm(t3, t1, 0x07060302u);
  }
  #pragma unroll
  for (int i = 0; i < 32; ++i) d[i] = o[i];
}
__device__ __forceinline__ u32 store_bits(const u32 (&d)[32], __amdgpu_buffer_rsrc_t rs, u32 bo, int lane) {
  u32 rb[12];
  #pragma unroll
  for (int g = 0; g < 4; ++g) {
    const u32* e = d + 8 * g;
    rb[3 * g]     = (e[0] & 0x83838383u) | ror(e[1] & 0x83838383u, 29) | ((e[6] & 0x03030303u) << 5);
    rb[3 * g + 1] = (e[2] & 0x83838383u) | ror(e[3] & 0x83838383u, 29) | ((e[7] & 0x03030303u) << 5);
    rb[3 * g + 2] = (e[4] & 0x83838383u) | ror(e[5] & 0x83838383u, 29) | ((e[6] & 0x80808080u) >> 1) | ((e[7] & 0x80808080u) >> 2);
  }
  __builtin_amdgcn_raw_buffer_store_b128((v4u){rb[0], rb[1], rb[2], rb[3]}, rs, (int)(bo + (u32)lane * 16u), 0, AUX_SYS);
  __builtin_amdgcn_raw_buffer_store_b128((v4u){rb[4], rb[5], rb[6], rb[7]}, rs, (int)(bo + 1024u + (u32)lane * 16u), 0, AUX_SYS);
  #pragma unroll
  for (u32 i = 0; i < 3; ++i)
    __builtin_amdgcn_raw_buffer_store_b32(rb[8 + i], rs, (int)(bo + 2048u + i * 256u + (u32)lane * 4u), 0, AUX_SYS);
  return rb[11];
}
__device__ __forceinline__ bool wait_fl(const unsigned char* fp, u64& fl, unsigned char* tmo, u32 call) {
  const u64 t0 = __builtin_amdgcn_s_memrealtime();
  for (;;) {
    const u64 f = __hip_atomic_load((const u64*)fp, __ATOMIC_RELAXED, __HIP_MEMORY_SCOPE_SYSTEM);
    const u32 f0 = __builtin_amdgcn_readfirstlane((u32)f), f1 = __builtin_amdgcn_readfirstlane((u32)(f >> 32));
    if ((f1 >> 16) == (call & 0xFFFFu)) { fl = (u64)f1 << 32 | f0; return true; }
    if (ld_agent(tmo) == call || __builtin_amdgcn_s_memrealtime() - t0 > DEADLINE_TICKS) {
      __hip_atomic_store((u32*)tmo, call, __ATOMIC_RELAXED, __HIP_MEMORY_SCOPE_AGENT);
      fl = 0;
      return false;
    }
    __builtin_amdgcn_s_sleep(4);
  }
}
__device__ __forceinline__ u32 stream_bytes(u64 fl) {
  u32 tot = 0;
  #pragma unroll
  for (int i = 0; i < 4; ++i) { const u32 ni = (u32)(fl >> (12 * i)) & 4095u; if (ni != 4095u) tot += 384u + ((2u * ni + 15u) & ~15u); }
  return (tot + 63u) & ~63u;
}

__global__ void __launch_bounds__(TPB) fused_gemm_rs(const Params P) {
  __shared__ __attribute__((aligned(1024))) unsigned char smem[SM_TOTAL];
  const int tid  = threadIdx.x;
  const int lane = tid & 63;
  const int wave = __builtin_amdgcn_readfirstlane(tid >> 6);
  const int me   = P.rank;
  const int bid  = blockIdx.x;
  const int role = bid >> 3;
  unsigned char* const my = P.ws[me];
  unsigned char* const tmo = my + OFF_TMO;
  lds_u8* const sm = (lds_u8*)smem;
  volatile lds_u32* const ctl = (volatile lds_u32*)(sm + L_CTL);
  const u32 l16 = (u32)lane * 16u;
  const __amdgpu_buffer_rsrc_t rs_my = mk_rsrc(my, WS_USED);

  STAMP(0);
  if (tid == 960) {
    const u32 ticket = __hip_atomic_fetch_add((u32*)(my + OFF_TICKET + (bid & 15) * 128), 1u, __ATOMIC_RELAXED, __HIP_MEMORY_SCOPE_AGENT);
    ctl[C_CALL] = ticket / (u32)(GRID / 16) + 1u;
  }
  if (tid > 896 && tid < 960) ctl[tid - 896] = 0u;

  if (role >= 28) {
    volatile lds_u32* const tt = (volatile lds_u32*)sm;
    if (tid < 256) tt[tid] = 0u;
    WAIT_LGKM0; __builtin_amdgcn_s_barrier();
    const u32 call = __builtin_amdgcn_readfirstlane(ctl[C_CALL]);
    const u32 par = call & 1u;
    const int pi = bid - 224, l = pi % 7, p = (me + 1 + l) & (NR - 1);
    if (pi >= 28 || p >= P.nranks) return;
    if (wave >= 12) {
      const u32 e = (u32)(wave - 12) * 64u + (u32)lane;
      const u64* const fp = (const u64*)(my + OFF_FL + ((par * NR + (u32)p) * NSTRIP + e) * 8u);
      const u64 t0 = __builtin_amdgcn_s_memrealtime();
      for (;;) {
        const u64 f = __hip_atomic_load(fp, __ATOMIC_RELAXED, __HIP_MEMORY_SCOPE_SYSTEM);
        const u32 b = ld_agent(tmo);
        const bool got = (u32)(f >> 48) == (call & 0xFFFFu);
        if (got) tt[e] = stream_bytes(f) | 0x80000000u;
        if (__builtin_amdgcn_ballot_w64(got) == ~0ull) break;
        if (b == call || __builtin_amdgcn_s_memrealtime() - t0 > DEADLINE_TICKS) {
          __hip_atomic_store((u32*)tmo, call, __ATOMIC_RELAXED, __HIP_MEMORY_SCOPE_AGENT);
          ctl[C_BAD] = 1u;
          break;
        }
        __builtin_amdgcn_s_sleep(1);
      }
      return;
    }
    if (wave >= PW) return;
    const __amdgpu_buffer_rsrc_t rs_p = mk_rsrc(P.ws[p], WS_USED);
    constexpr u32 NONE = 0x80000000u;
    const u32 tko = lane == 0 ? OFF_LC + (u32)l * 128u : NONE;
    const u32 tkb = (call - 1u) * (NT + 8u * PW);
    u32 fa1 = NONE, fa2 = NONE;
    v4u VA[3], VB[3]; u32 oa[3], ob[3];
    #pragma unroll
    for (int i = 0; i < 3; ++i) { VA[i] = VB[i] = (v4u){0u, 0u, 0u, 0u}; oa[i] = ob[i] = NONE; }
    u32 ta = (u32)__builtin_amdgcn_raw_ptr_buffer_atomic_add_i32(1, rs_my, (int)tko, 0, 16);
    u32 tb = (u32)__builtin_amdgcn_raw_ptr_buffer_atomic_add_i32(1, rs_my, (int)tko, 0, 16);
    asm volatile("" : "+v"(ta), "+v"(tb));
    u32 tn = __builtin_amdgcn_readfirstlane(ta) - tkb;
    auto step = [&](u32& tk, u32& tk1, v4u (&C)[3], u32 (&oc)[3], const v4u (&V)[3], u32 (&ov)[3]) {
      const u32 t = tn;
      const u32 z = (t - 128u) & 4095u, kz = z / 224u;
      const u32 sz = z - kz * 224u + 224u - (u32)l * DEL;
      const u32 k = t < 128u ? 4u + (t >> 5) : kz, st = t < 128u ? t & 31u : 32u + (sz < 224u ? sz : sz - 224u);
      const u32 pc = ((t < 128u ? 0x32107654u : 0x76352410u) >> (4u * (k & 7u))) & 15u;
      const u32 ti = st;
      if (t >= NT || __builtin_amdgcn_readfirstlane(tt[ti]) == 0u) {
        WAIT_VM(0);
        __builtin_amdgcn_raw_buffer_store_b32(call, rs_my, (int)(lane == 0 ? fa2 : NONE), 0, AUX_SYS);
        #pragma unroll
        for (int i = 0; i < 3; ++i) { __builtin_amdgcn_raw_buffer_store_b128(V[i], rs_my, (int)ov[i], 0, AUX_SYS); ov[i] = NONE; }
        WAIT_VM(0);
        __builtin_amdgcn_raw_buffer_store_b32(call, rs_my, (int)(lane == 0 ? fa1 : NONE), 0, AUX_SYS);
        fa1 = fa2 = NONE;
        if (t >= NT) asm volatile("s_endpgm" ::: "memory");
        wait_lds(ctl, (int)ti - (int)(L_CTL / 4u), 1u, tmo, call);
        if (__builtin_amdgcn_readfirstlane(ctl[C_BAD]) != 0u) asm volatile("s_endpgm" ::: "memory");
      }
      const u32 tot = __builtin_amdgcn_readfirstlane(tt[st]) & 0x7FFFFFFFu;
      const u32 src = OFF_ENC + (par * NSTRIP + st) * STRIP, dst = OFF_CP + ((u32)l * NSTRIP + st) * STRIP;
      const u32 b0 = pc < 4u ? pc * RCAP : 4u * RCAP + (pc - 4u) * BSZ;
      const u32 b1 = pc < 4u ? (tot < b0 + RCAP ? tot : b0 + RCAP) : b0 + BSZ;
      #pragma unroll
      for (u32 i = 0; i < 3; ++i) {
        u32 o = b0 + i * 1024u + l16;
        o = o < b1 ? o : NONE;
        C[i] = __builtin_amdgcn_raw_buffer_load_b128(rs_p, (int)(src + o), 0, AUX_SYS);
        oc[i] = dst + o;
      }
      asm volatile("" ::: "memory");
      tk = (u32)__builtin_amdgcn_raw_ptr_buffer_atomic_add_i32(1, rs_my, (int)tko, 0, 16);
      WAIT_VM(4);
      asm volatile("" : "+v"(tk1));
      tn = __builtin_amdgcn_readfirstlane(tk1) - tkb;
      const u32 gz = t - (NT - DLT);
      const u32 go = OFF_AR + (255u - ND + 2u * gz + (u32)lane - (u32)l * DEL) * 256u + ((u32)l * 8u + 7u) * 4u;
      __builtin_amdgcn_raw_buffer_store_b32(call, rs_my, (int)(lane == 0 ? fa2 : lane < 3 && gz < ND / 2 ? go : NONE), 0, AUX_SYS);
      #pragma unroll
      for (int i = 0; i < 3; ++i) __builtin_amdgcn_raw_buffer_store_b128(V[i], rs_my, (int)ov[i], 0, AUX_SYS);
      fa2 = fa1;
      fa1 = OFF_AR + st * 256u + ((u32)l * 8u + k) * 4u;
    };
    #pragma unroll 1
    for (u32 n = 0; n < 2048u; ++n) {
      step(ta, tb, VA, oa, VB, ob);
      step(tb, ta, VB, ob, VA, oa);
    }
    return;
  }

  const u32 cw = (u32)(bid - 64);
  u32 call;
  auto prologue = [&]() {
    const v4u t1 = *(const v4u*)((const unsigned char*)g_tabs.t + 16384 + tid * 16);
    const v4u t0 = *(const v4u*)((const unsigned char*)g_tabs.t + tid * 16);
    *(lds_v4u*)(sm + L_ENC + (u32)tid * 16u) = t1;
    *(lds_v4u*)(sm + L_DEC + (u32)tid * 16u) = t0;
    WAIT_LGKM0; __builtin_amdgcn_s_barrier();
    call = __builtin_amdgcn_readfirstlane(ctl[C_CALL]);
    STAMP(1);
  };

  if (role < 8) {
    const int pd = (me + 1 + bid) & (NR - 1);
    const unsigned char* src = P.x + (size_t)(pd * MR + role * 64 + (lane & 31)) * KS + (wave >> 1) * 64 + (lane >> 5) * 32 + (wave & 1) * 16;
    const v4u xv0 = *(const v4u*)src, xv1 = *(const v4u*)(src + 32 * KS);
    prologue();
    if (pd < P.nranks) {
      const __amdgpu_buffer_rsrc_t rs = mk_rsrc(P.ws[pd], WS_USED);
      const u32 dof = OFF_XA + (call & 1u) * (u32)(MR * KK) + (u32)(((me * 8 + (wave >> 1)) * 16 + role * 2) * 2 + (wave & 1)) * 1024u + l16;
      __builtin_amdgcn_raw_buffer_store_b128(xv0, rs, (int)dof, 0, AUX_SYS);
      __builtin_amdgcn_raw_buffer_store_b128(xv1, rs, (int)(dof + 2048u), 0, AUX_SYS);
      WAIT_VM(0);
      if (lane == 0 && __hip_atomic_fetch_add((lds_u32*)(sm + L_CTL) + C_PUSH, 1u, __ATOMIC_RELAXED, __HIP_MEMORY_SCOPE_WORKGROUP) == 15u)
        __hip_atomic_fetch_add((u32*)(P.ws[pd] + OFF_CX + me * 128), 1u, __ATOMIC_RELAXED, __HIP_MEMORY_SCOPE_SYSTEM);
    }
    STAMP(2);
  } else if (role >= 24 && wave < 8) {
    prologue();
  } else if (role >= 24) {
    const u32 es = (u32)(bid - 192), eu = (u32)(wave & 3);
    const int i0 = wave & 4, rot = bid * 4 + (wave & 3);
    u32 d[32];
    load_unit(P.w, es, eu, lane, d);
    prologue();
    turn_unit(d);
    #pragma unroll
    for (int i = 0; i < 4; ++i) {
      const int li = (i0 + i + rot) % 7, pd = (me + 1 + li) & (NR - 1);
      if (i0 + i < 7 && pd < P.nranks)
        store_bits(d, mk_rsrc(P.ws[pd], WS_USED), OFF_EB + ((((call & 1u) * 7u + (u32)(6 - li)) * G0 + es) * 4u + eu) * BSZ, lane);
    }
    WAIT_VM(0);
    const int pf = (me + 1 + (i0 + lane + rot) % 7) & (NR - 1);
    if (lane < 4 && i0 + lane < 7 && pf < P.nranks)
      __hip_atomic_store((u32*)(P.ws[pf] + OFF_FB + (((call & 1u) * NR + (u32)me) * 128u + es * 4u + eu) * 4u), call, __ATOMIC_RELAXED, __HIP_MEMORY_SCOPE_SYSTEM);
    STAMP(2);
  } else if (wave >= 8) {
    prologue();
    __builtin_amdgcn_s_barrier();
  } else {
    const u32 es = cw * 2u + (u32)(wave >> 2), eu = (u32)(wave & 3);
    u32 d[32];
    load_unit(P.w, es, eu, lane, d);
    prologue();
    const u32 par = call & 1u;
    const u32 eb = OFF_ENC + (par * NSTRIP + es) * STRIP;
    turn_unit(d);
    const u32 pay = store_bits(d, rs_my, eb + 4u * RCAP + eu * BSZ, lane);
    u32 x = 0x10000u | (pay & 0xFFFFu), pend = pay >> 16, lim = 0xFFFFFFFFu;
    const u32 rend = L_R + (u32)(wave + 1) * 4u * RCAP;
    u32 top = rend;
    auto fetch = [&](int i, v4u (&E)[4]) {
      #pragma unroll
      for (int k = 1; k >= 0; --k) {
        const u32 a = ((d[i + k] << 2) & 0x01F001F0u) | ((d[i + k] >> 1) & 0x3E003E00u);
        E[2 * k + 1] = *(const lds_v4u*)(sm + L_ENC + (a >> 16));
        E[2 * k]     = *(const lds_v4u*)(sm + L_ENC + (a & 0xFFFFu));
      }
    };
    auto code = [&](const v4u (&E)[4]) {
      #pragma unroll
      for (int k = 3; k >= 0; --k) {
        u32 t, q, r, sn;
        asm volatile("v_cmp_ge_u32 vcc, %[x], %[e1]\n"
                     "v_lshrrev_b32 %[t], 16, %[x]\n"
                     "s_bcnt1_i32_b64 %[n], vcc\n"
                     "v_cndmask_b32 %[t], %[x], %[t], vcc\n"
                     "v_mul_hi_u32 %[q], %[t], %[e0]\n"
                     "v_mbcnt_lo_u32_b32 %[r], vcc_lo, 0\n"
                     "v_mbcnt_hi_u32_b32 %[r], vcc_hi, %[r]\n"
                     "s_lshl_b32 %[n], %[n], 1\n"
                     "s_sub_u32 %[top], %[top], %[n]\n"
                     "v_add_u32_sdwa %[t], %[t], %[e2] dst_sel:DWORD dst_unused:UNUSED_PAD src0_sel:DWORD src1_sel:WORD_1\n"
                     "v_lshrrev_b32 %[q], %[e2], %[q]\n"
                     "v_lshl_add_u32 %[r], %[r], 1, %[top]\n"
                     "s_mov_b64 exec, vcc\n"
                     "ds_write_b16 %[r], %[pend]\n"
                     "v_mov_b32 %[pend], %[x]\n"
                     "s_mov_b64 exec, -1\n"
                     "v_mad_u32_u24 %[x], %[q], %[e3], %[t]\n"
                     "v_min_u32 %[lim], %[lim], %[e1]\n"
                     : [x] "+v"(x), [pend] "+v"(pend), [lim] "+v"(lim), [top] "+s"(top), [t] "=&v"(t), [q] "=&v"(q), [r] "=&v"(r), [n] "=&s"(sn)
                     : [e0] "v"(E[k][0]), [e1] "v"(E[k][1]), [e2] "v"(E[k][2]), [e3] "v"(E[k][3]) : "vcc", "scc", "memory");
      }
    };
    v4u EA[4], EB[4];
    fetch(30, EA);
    #pragma unroll
    for (int i = 28; i >= 0; i -= 4) {
      fetch(i, EB);
      code(EA);
      if (i > 0) fetch(i - 2, EA);
      code(EB);
    }
    const u32 n = (rend - top) >> 1;
    const bool rawu = n > NMAX || __builtin_amdgcn_ballot_w64(lim == 0u) != 0ull;
    const u32 start = (top & ~15u) - 384u;
    *(lds_u32*)(sm + start + (u32)lane * 4u) = x;
    *(lds_u16*)(sm + start + 256u + (u32)lane * 2u) = (unsigned short)pend;
    ctl[C_N + wave] = rawu ? 4095u : n;
    if (rawu) {
      #pragma unroll
      for (int i = 0; i < 8; ++i)
        __builtin_amdgcn_raw_buffer_store_b128((v4u){d[4 * i], d[4 * i + 1], d[4 * i + 2], d[4 * i + 3]}, rs_my,
                                               (int)(OFF_RAW + ((par * NSTRIP + es) * 4u + eu) * 8192u + (u32)i * 1024u + l16), 0, AUX_SYS);
    }
    STAMP(2);
    WAIT_LGKM0; __builtin_amdgcn_s_barrier();
    u32 off = 0; u64 fl = (u64)(call & 0xFFFFu) << 48;
    #pragma unroll
    for (u32 i = 0; i < 4; ++i) {
      const u32 ni = __builtin_amdgcn_readfirstlane(ctl[C_N + (wave & 4) + i]);
      fl |= (u64)ni << (12u * i);
      if (i < eu && ni != 4095u) off += 384u + ((2u * ni + 15u) & ~15u);
    }
    if (!rawu) {
      #pragma unroll
      for (u32 i = 0; i < 3; ++i) {
        const u32 o = i * 1024u + l16;
        if (start + o < rend) {
          const v4u v = *(const lds_v4u*)(sm + start + o);
          __builtin_amdgcn_raw_buffer_store_b128(v, rs_my, (int)(eb + off + o), 0, AUX_SYS);
        }
      }
    }
    WAIT_VM(0);
    u32 last = 0;
    if (lane == 0) last = __hip_atomic_fetch_add((lds_u32*)(sm + L_CTL) + C_ENCD + (wave >> 2), 1u, __ATOMIC_RELAXED, __HIP_MEMORY_SCOPE_WORKGROUP);
    if (__builtin_amdgcn_readfirstlane(last) == 3u && lane < NR && lane < P.nranks && lane != me)
      __hip_atomic_store((u64*)(P.ws[lane] + OFF_FL + ((par * NR + (u32)me) * NSTRIP + es) * 8u), fl, __ATOMIC_RELAXED, __HIP_MEMORY_SCOPE_SYSTEM);
    STAMP(3);
  }
  const u32 par = call & 1u;
  const u32 it0 = role >= 24 ? 0u : 1u;
  const u32 l0 = bid + 6 * DEL >= G1 ? (u32)(G1 - bid + DEL - 1) / DEL : 0u;

  if (wave == 7) {
    bool ok = wait_flags(my + OFF_CX, NR, lane, 8u * call, tmo, call);
    __builtin_amdgcn_fence(__ATOMIC_ACQUIRE, "");
    WAIT_VM(0);
    if (!ok) ctl[C_BAD] = 1u;
    ctl[C_XOK] = 1u;
    STAMP(4);
    #pragma unroll 1
    for (u32 it = it0; it < 2; ++it) {
      const unsigned char* const ap = lane >= 56 ? tmo :
          it == 0u && (lane & 4) == 0 ? my + OFF_FB + ((par * NR + (u32)((me + 1 + (lane >> 3)) & (NR - 1))) * 128u + (u32)(bid - 192) * 4u + (u32)(lane & 3)) * 4u :
          my + OFF_AR + (u32)(it ? bid + G0 : bid - 192) * 256u + (u32)lane * 4u;
      const u64 t0 = __builtin_amdgcn_s_memrealtime();
      while (ok) {
        const u64 m = __builtin_amdgcn_ballot_w64(ld_sys(ap) == call);
        if (lane < 7) ctl[C_ARR + lane] = it * 8u + (u32)__builtin_ctz(~((u32)(m >> (8 * lane)) & 255u));
        if ((m & 0xFFFFFFFFFFFFFFull) == 0xFFFFFFFFFFFFFFull) break;
        if ((m >> 63) != 0ull || __builtin_amdgcn_s_memrealtime() - t0 > DEADLINE_TICKS) {
          __hip_atomic_store((u32*)tmo, call, __ATOMIC_RELAXED, __HIP_MEMORY_SCOPE_AGENT);
          ok = false;
        }
        __builtin_amdgcn_s_sleep(POLLS);
      }
    }
    if (!ok) {
      ctl[C_BAD] = 1u;
      if (lane < 7) ctl[C_ARR + lane] = 16u;
    }
    return;
  }

  if (wave < 7) {
    const int p0 = (me + 1 + wave) & (NR - 1), p = p0 < P.nranks ? p0 : me;
    const u32 lr = L_R + (u32)wave * 4u * RCAP;
    const u32 lb = L_B + (u32)wave * BSZ;
    u32 d[32];
    auto hand_over = [&](u32 q) {
      wait_lds(ctl, C_CONS + (int)(q % NS), 8u * (q / NS), tmo, call);
      #pragma unroll
      for (int i = 0; i < 8; ++i)
        *(volatile lds_v4u*)(sm + L_SLOT + (q % NS) * SLOT + (u32)i * CHS + l16) = (v4u){d[4 * i], d[4 * i + 1], d[4 * i + 2], d[4 * i + 3]};
      ctl[C_RDY + q % NS] = q + 1u;
    };
    #pragma unroll 1
    for (u32 it = it0; it < 2; ++it) {
      const u32 st = (u32)(it ? bid + G0 : bid - 192), q0 = (it - it0) * 32u;
      if (wave < 4) {
        load_unit(P.w, st, (u32)wave, lane, d);
        turn_unit(d);
        if (role >= 8 && role < 24) wait_lds(ctl, C_ENCD + 1, 4u, tmo, call);
        hand_over(q0 + (u32)wave);
      }
      u64 fl;
      if (!wait_fl(my + OFF_FL + ((par * NR + (u32)p0) * NSTRIP + st) * 8u, fl, tmo, call)) ctl[C_BAD] = 1u;
      const unsigned char* const rp = my + OFF_CP + ((u32)wave * NSTRIP + st) * STRIP;
      const v4u srd_r = mk_srd(rp, stream_bytes(fl));
      v4u srd_b = mk_srd(it ? rp + 4u * RCAP : my + OFF_EB + ((par * 7u + (u32)wave) * G0 + st) * 4u * BSZ, 4u * BSZ);
      auto load_r = [&](u32 j) {
        asm volatile("s_mov_b32 m0, %0\n s_nop 0\n"
                     "buffer_load_dwordx4 %1, %2, %3 offen sc0 sc1 lds\n"
                     "buffer_load_dwordx4 %1, %2, %3 offen offset:1024 sc0 sc1 lds\n"
                     "buffer_load_dwordx4 %1, %2, %3 offen offset:2048 sc0 sc1 lds\n"
                     :: "s"(lr + j * RCAP), "v"(l16), "s"(srd_r), "s"(j * RCAP) : "memory");
      };
      auto load_b = [&](u32 j) {
        asm volatile("s_mov_b32 m0, %0\n s_nop 0\n"
                     "buffer_load_dwordx4 %1, %2, %3 offen sc0 sc1 lds\n"
                     "buffer_load_dwordx4 %1, %2, %3 offen offset:1024 sc0 sc1 lds\n"
                     "s_mov_b32 exec_hi, 0xffff\n"
                     "buffer_load_dwordx4 %1, %2, %3 offen offset:2048 sc0 sc1 lds\n"
                     "s_mov_b32 exec_hi, -1\n"
                     :: "s"(lb), "v"(l16), "s"(srd_b), "s"(j * BSZ) : "memory");
      };
      if (it == 0) STAMP(4);
      wait_lds(ctl, C_ARR + wave, it ? 9u : 5u, tmo, call);
      if (it == 0) STAMP(5);
      load_r(0);
      WAIT_VM(0);
      u32 off = lr;
      #pragma unroll 1
      for (u32 u = 0; u < 4; ++u) {
        const u32 n = (u32)(fl >> (12u * u)) & 4095u;
        u32 pay = 0;
        if (n != 4095u) {
          u32 x, wn, q, ea, eb, t, r, sn, sp = off + 384u + ((0u - 2u * n) & 15u);
          asm volatile("ds_read_b32 %[x], %[a4]\n"
                       "ds_read_u16 %[wn], %[a2] offset:256\n"
                       "s_waitcnt lgkmcnt(1)\n"
                       "v_and_b32 %[t], 0xfff, %[x]\n"
                       "v_lshlrev_b32 %[t], 2, %[t]\n"
                       "ds_read_b32 %[ea], %[t]\n"
                       "v_lshrrev_b32 %[q], 12, %[x]\n"
                       : [x] "=&v"(x), [wn] "=&v"(wn), [q] "=&v"(q), [ea] "=&v"(ea), [t] "=&v"(t)
                       : [a4] "v"(off + (u32)lane * 4u), [a2] "v"(off + (u32)lane * 2u) : "memory");
          #pragma unroll
          for (int i = 0; i < 32; ++i)
            asm volatile(DEC_STEP("ea", "eb", "s_nop 0\n")
                         DEC_STEP("eb", "ea", "v_perm_b32 %[o], %[eb], %[ea], %[sel]\n")
                         : [x] "+v"(x), [wn] "+v"(wn), [q] "+v"(q), [ea] "+v"(ea), [sp] "+s"(sp),
                           [eb] "=&v"(eb), [t] "=&v"(t), [r] "=&v"(r), [n] "=&s"(sn), [o] "=&v"(d[i])
                         : [sel] "s"(0x05040100u) : "vcc", "scc", "memory");
          asm volatile("s_waitcnt lgkmcnt(0)" : "+v"(wn), "+v"(ea) :: "memory");
          pay = wn << 16 | (x & 0xFFFFu);
          off += 384u + ((2u * n + 15u) & ~15u);
        }
        if (it) STAMP(7 + 2 * u);
        if (u < 3) {
          wait_lds(ctl, C_ARR + wave, it ? 10u + 2u * u : 6u + u, tmo, call);
          load_r(u + 1u);
        }
        if (it) wait_lds(ctl, C_ARR + wave, u < 3u ? 2u * u + 11u : 16u, tmo, call);
        if (it && u == 3u && bid + wave * DEL >= G1 - ND && bid + wave * DEL < G1)
          srd_b = mk_srd(P.ws[p] + OFF_ENC + (par * NSTRIP + st) * STRIP + 4u * RCAP, 4u * BSZ);
        load_b(u);
        WAIT_VM(0);
        if (n != 4095u) {
          const v4u c0 = *(const volatile lds_v4u*)(sm + lb + l16), c1 = *(const volatile lds_v4u*)(sm + lb + 1024u + l16);
          const volatile lds_u32* const c2 = (const volatile lds_u32*)(sm + lb + 2048u) + lane;
          const u32 rb[12] = {c0[0], c0[1], c0[2], c0[3], c1[0], c1[1], c1[2], c1[3], c2[0], c2[64], c2[128], pay};
          #pragma unroll
          for (int g = 0; g < 4; ++g) {
            u32* e = d + 8 * g;
            const u32 r0 = rb[3 * g], r1 = rb[3 * g + 1], r2 = rb[3 * g + 2];
            e[0] = bfi(0x83838383u, r0, e[0]); e[1] = bfi(0x83838383u, ror(r0, 3), e[1]);
            e[2] = bfi(0x83838383u, r1, e[2]); e[3] = bfi(0x83838383u, ror(r1, 3), e[3]);
            e[4] = bfi(0x83838383u, r2, e[4]); e[5] = bfi(0x83838383u, ror(r2, 3), e[5]);
            e[6] = bfi(0x80808080u, r2 << 1, bfi(0x03030303u, r0 >> 5, e[6]));
            e[7] = bfi(0x80808080u, r2 << 2, bfi(0x03030303u, r1 >> 5, e[7]));
          }
        } else {
          const v4u srd_w = mk_srd(P.ws[p] + OFF_RAW + ((par * NSTRIP + st) * 4u + u) * 8192u, 8192u);
          #pragma unroll
          for (int h = 0; h < 2; ++h) {
            v4u w0, w1, w2, w3;
            asm volatile("buffer_load_dwordx4 %0, %4, %5, %6 offen sc0 sc1\n"
                         "buffer_load_dwordx4 %1, %4, %5, %6 offen offset:1024 sc0 sc1\n"
                         "buffer_load_dwordx4 %2, %4, %5, %6 offen offset:2048 sc0 sc1\n"
                         "buffer_load_dwordx4 %3, %4, %5, %6 offen offset:3072 sc0 sc1\n"
                         "s_waitcnt vmcnt(0)\n"
                         : "=&v"(w0), "=&v"(w1), "=&v"(w2), "=&v"(w3) : "v"(l16), "s"(srd_w), "s"(h * 4096) : "memory");
            #pragma unroll
            for (int k = 0; k < 4; ++k) { d[16 * h + k] = w0[k]; d[16 * h + 4 + k] = w1[k]; d[16 * h + 8 + k] = w2[k]; d[16 * h + 12 + k] = w3[k]; }
          }
        }
        hand_over(q0 + 4u + u * 7u + ((u32)wave + 7u - l0) % 7u);
        if (it) STAMP(8 + 2 * u);
      }
      if (it == 0) STAMP(6);
    }
    return;
  }

  const int mw = wave - 8;
  const unsigned char* const xa = my + OFF_XA + par * (u32)(MR * KK) + (u32)mw * 4096u + l16;
  const u32 bo = L_SLOT + (u32)(lane & 3) * 2u * CHS + (u32)((lane >> 5) * 16 + ((lane & 31) >> 2)) * 16u;
  v16f acc[2][2];
  auto kb_of = [&](u32 q) -> u32 {
    if (q < 4u) return (u32)me * 8u + q * 2u;
    const u32 z = q - 4u, u = z / 7u, l = (z + l0) % 7u;
    return (((u32)me + 1u + l) & (NR - 1)) * 8u + u * 2u;
  };
  auto load_a = [&](u32 kb, v4u (&A)[4]) {
    #pragma unroll
    for (int i = 0; i < 4; ++i) A[i] = *(const v4u*)(xa + kb * 32768u + i * 1024);
  };
  auto mma = [&](const v4u (&A)[4], u32 so) {
    v8i W[2];
    #pragma unroll
    for (int t = 0; t < 2; ++t) {
      const v4u w0 = *(const volatile lds_v4u*)(sm + so + t * 128), w1 = *(const volatile lds_v4u*)(sm + so + t * 128 + CHS);
      W[t] = (v8i){(int)w0[0], (int)w0[1], (int)w0[2], (int)w0[3], (int)w1[0], (int)w1[1], (int)w1[2], (int)w1[3]};
    }
    #pragma unroll
    for (int i = 0; i < 2; ++i) {
      const v8i X = (v8i){(int)A[2 * i][0], (int)A[2 * i][1], (int)A[2 * i][2], (int)A[2 * i][3],
                          (int)A[2 * i + 1][0], (int)A[2 * i + 1][1], (int)A[2 * i + 1][2], (int)A[2 * i + 1][3]};
      #pragma unroll
      for (int t = 0; t < 2; ++t)
        acc[i][t] = __builtin_amdgcn_mfma_scale_f32_32x32x64_f8f6f4(X, W[t], acc[i][t], 0, 1, 0, 0x7F7F7F7F, 0, 0x7F7F7F7F);
    }
    asm volatile("" : "+v"(acc[0][0]), "+v"(acc[0][1]), "+v"(acc[1][0]), "+v"(acc[1][1]));
  };
  v4u A0[4], A1[4];
  wait_lds(ctl, C_XOK, 1u, tmo, call);
  load_a(kb_of(0), A0);
  #pragma unroll 1
  for (u32 it = it0; it < 2; ++it) {
    #pragma unroll
    for (int i = 0; i < 2; ++i)
      #pragma unroll
      for (int j = 0; j < 2; ++j)
        #pragma unroll
        for (int e = 0; e < 16; ++e) acc[i][j][e] = 0.f;
    #pragma unroll 1
    for (u32 q = 0; q < 32u; ++q) {
      load_a(kb_of(q) + 1u, A1);
      wait_lds(ctl, C_RDY + (int)(q % NS), (it - it0) * 32u + q + 1u, tmo, call);
      const u32 so = bo + (q % NS) * SLOT;
      mma(A0, so);
      SCHED_FENCE;
      load_a(kb_of((q + 1u) & 31u), A0);
      mma(A1, so + 512u);
      SCHED_FENCE;
      if (lane == 0) __hip_atomic_fetch_add((lds_u32*)(sm + L_CTL) + C_CONS + q % NS, 1u, __ATOMIC_RELAXED, __HIP_MEMORY_SCOPE_WORKGROUP);
    }
    STAMP(it ? 8 : 3);
    const bool ok = __builtin_amdgcn_readfirstlane(ctl[C_BAD]) == 0u;
    const float scale = ok ? P.sx[0] * P.sw[0] : __builtin_nanf("");
    const __amdgpu_buffer_rsrc_t rs_out = mk_rsrc(P.out, (u32)MR * NN * 4u);
    const u32 ob = ((u32)(mw * 64 + (lane >> 5) * 4) * (u32)NN + (u32)(it ? bid + G0 : bid - 192) * 64u + (u32)(lane & 31)) * 4u;
    #pragma unroll
    for (int i = 0; i < 2; ++i)
      #pragma unroll
      for (int e = 0; e < 16; ++e)
        #pragma unroll
        for (int t = 0; t < 2; ++t)
          __builtin_amdgcn_raw_buffer_store_b32(__builtin_bit_cast(u32, acc[i][t][e] * scale), rs_out,
                                                (int)(ob + (u32)(i * 32 + (e >> 2) * 8 + (e & 3)) * (u32)(NN * 4) + (u32)t * 128u), 0, AUX_OUT);
    STAMP(it ? 9 : 4);
  }
}

extern "C" void kernel_launch(int rank, int nranks, ncclComm_t comm,
                              hipStream_t stream,
                              void* const* d_in, const int* in_sizes, int n_in,
                              void* d_out, int out_size,
                              const int* in_dtypes, int out_dtype,
                              void* const* d_out_all,
                              void* const* d_ws_all, size_t ws_bytes) {
  (void)comm; (void)in_sizes; (void)in_dtypes; (void)out_dtype; (void)d_out_all;
  if (nranks != NR || n_in < 4 || ws_bytes < (size_t)WS_USED || out_size != MR * NN || rank < 0 || rank >= NR) {
    fprintf(stderr, "unexpected configuration\n");
    return;
  }
  Params P;
  P.x  = (const unsigned char*)d_in[0];
  P.w  = (const unsigned char*)d_in[1];
  P.sx = (const float*)d_in[2];
  P.sw = (const float*)d_in[3];
  P.out = (float*)d_out;
  for (int p = 0; p < NR; ++p) P.ws[p] = (unsigned char*)d_ws_all[p];
  P.rank = rank;
  P.nranks = nranks;
  fused_gemm_rs<<<GRID, TPB, 0, stream>>>(P);
  HIP_CHECK(hipGetLastError());
}
```
